```python
import numpy as np

kernel_name = "gpu_gemm_ar_m4096_k4096_n16384_f32_qe4m3"


ABSMAX_RTOL = 0.09

COMPUTE_DTYPE = "float32"
EPILOGUE = "quant_e4m3"
IN_QUANT = None
OUT_QUANT = "e4m3"
IN_DTYPES = {"x": "float32", "w_mat": "float32"}
WORLD_SIZE = 8

def setup_inputs():

    x = np.random.randn(4096, 4096).astype(np.float32)
    w_mat = (np.random.randn(4096, 16384) / np.sqrt(4096)).astype(np.float32)
    return {"x": x, "w_mat": w_mat}

def _snap_fp8(x, mant_bits, max_abs, min_normal):
    a = np.abs(x.astype(np.float64))
    _m, _e = np.frexp(a)
    q = np.where(a >= min_normal, np.ldexp(1.0, _e - 1 - mant_bits),
                 min_normal * 2.0 ** -mant_bits)
    snapped = np.minimum(np.round(a / q) * q, max_abs)
    return (np.sign(x) * snapped).astype(np.float32)
def _quant_dequant(y, amax):
    scale = amax / 448.0
    q = _snap_fp8(y / scale, 3, 448.0, 2.0 ** -6)
    return (q * scale).astype(np.float32)
def _epilogue(y):
    return _quant_dequant(y, np.abs(y).max())

def reference(x, w_mat):
    y = (x @ w_mat).astype(np.float32)
    return _epilogue(y)
```

```cpp
#include <hip/hip_runtime.h>
#include <rccl/rccl.h>
#include <cstdio>
#include <cstdint>

#define HIP_CHECK(expr) do { hipError_t _e = (expr); if (_e != hipSuccess) { \
    fprintf(stderr, "HIP error %s at %s:%d\n", hipGetErrorString(_e), __FILE__, __LINE__); \
    return; } } while (0)

#define GAS __attribute__((address_space(1)))
#define LAS __attribute__((address_space(3)))
typedef unsigned v4u __attribute__((ext_vector_type(4)));
typedef unsigned v2u __attribute__((ext_vector_type(2)));
typedef float f32x4 __attribute__((ext_vector_type(4)));
typedef short bf16x8 __attribute__((ext_vector_type(8)));
typedef __bf16 bf2 __attribute__((ext_vector_type(2)));
typedef GAS unsigned gu32;
typedef GAS unsigned long long gu64;

namespace {

constexpr int NR = 8;
constexpr int MM = 4096, NN = 16384;
constexpr int KS = 512;
constexpr int NS = 2048;
constexpr int KBS = 16;
constexpr unsigned A_BYTES = 256u * 1024u;
constexpr unsigned B_BYTES = 128u * 1024u;
constexpr unsigned BLK_BYTES = A_BYTES + B_BYTES;
constexpr unsigned SRC_BYTES = KBS * BLK_BYTES;
constexpr size_t STAGE_OFF = 1u << 20;
constexpr size_t WS_NEED = STAGE_OFF + (size_t)NR * SRC_BYTES;
constexpr unsigned OUT_BYTES = (unsigned)MM * NN;

constexpr unsigned TICKP = 0x0000;
constexpr unsigned TICKS = 0x0400;
constexpr unsigned TMO   = 0x0800;
constexpr unsigned CNT1  = 0x1000;
constexpr unsigned AFLAG = 0x4000;
constexpr unsigned GMAX  = 0x5000;
constexpr unsigned RMAX  = 0x6000;
constexpr unsigned ODONE = 0x8000;
constexpr unsigned FIN   = 0x9000;
constexpr unsigned PROF  = 0x10000;

constexpr int NWG = 256, NTHREADS = 768;
constexpr int NSLOT = 6;
constexpr unsigned SLOT_BYTES = 24576;
constexpr unsigned L_FULL = NSLOT * SLOT_BYTES;
constexpr unsigned L_FREE = L_FULL + 16;
constexpr unsigned L_WMAX = L_FREE + 32;
constexpr unsigned L_MISC = L_WMAX + 32;
constexpr unsigned L_SCHED = L_MISC + 16;
constexpr unsigned LDS_BYTES = L_SCHED + 512;
constexpr int NSTEP = NR * KBS;
constexpr unsigned long long DEADLINE = 100000000ull;

#define WIN1 2
#define NPUSH 32
#define WIN2 2
#define ACQ_EACH 0
__device__ __forceinline__ int ch_end(int c) {
    const unsigned long long lo = 0x0F0E0C0A08060300ull;
    return c >= 7 ? 16 : (int)((lo >> (8 * (c + 1))) & 0xff);
}

#define VMCNT(n) asm volatile("s_waitcnt vmcnt(" #n ")" ::: "memory")
#define LGKM0() asm volatile("s_waitcnt lgkmcnt(0)" ::: "memory")
#define BAR() do { asm volatile("s_waitcnt lgkmcnt(0)" ::: "memory"); __builtin_amdgcn_s_barrier(); asm volatile("" ::: "memory"); } while (0)
#define RLX_AGENT __ATOMIC_RELAXED, __HIP_MEMORY_SCOPE_AGENT
#define RLX_SYS __ATOMIC_RELAXED, __HIP_MEMORY_SCOPE_SYSTEM

__device__ __forceinline__ unsigned long long rt() { return __builtin_amdgcn_s_memrealtime(); }
__device__ __forceinline__ unsigned rfl(unsigned v) { return (unsigned)__builtin_amdgcn_readfirstlane((int)v); }
__device__ __forceinline__ unsigned pk_bf16(float lo, float hi) {
    bf2 v; v.x = (__bf16)lo; v.y = (__bf16)hi; return __builtin_bit_cast(unsigned, v);
}
__device__ __forceinline__ void lds_st(unsigned addr, unsigned v) { asm volatile("ds_write_b32 %0, %1" :: "v"(addr), "v"(v) : "memory"); }
__device__ __forceinline__ unsigned lds_ld(unsigned addr) {
    unsigned v; asm volatile("ds_read_b32 %0, %1\n\ts_waitcnt lgkmcnt(0)" : "=&v"(v) : "v"(addr) : "memory"); return v;
}
__device__ __forceinline__ void glds4(unsigned voff, const char* sbase, unsigned m0v) {
    unsigned keep;
    asm volatile("s_mov_b32 %0, m0\n\ts_mov_b32 m0, %3\n\ts_nop 4\n\t"
                 "global_load_lds_dwordx4 %1, %2\n\t"
                 "global_load_lds_dwordx4 %1, %2 offset:1024\n\t"
                 "global_load_lds_dwordx4 %1, %2 offset:2048\n\t"
                 "global_load_lds_dwordx4 %1, %2 offset:3072\n\t"
                 "s_mov_b32 m0, %0"
                 : "=&s"(keep) : "v"(voff), "s"(sbase), "s"(m0v) : "memory");
}
__device__ __forceinline__ __amdgpu_buffer_rsrc_t mk_rsrc(void* base, unsigned bytes) {
    return __builtin_amdgcn_make_buffer_rsrc(base, 0, (int)bytes, 0x00020000);
}
__device__ __forceinline__ unsigned atomic_inc_async(const void* p) {
    unsigned old; unsigned long long sv;
    asm volatile("s_mov_b64 %1, exec\n\ts_mov_b64 exec, 1\n\t"
                 "global_atomic_add %0, %2, %3, %4 sc0\n\t"
                 "s_mov_b64 exec, %1"
                 : "=&v"(old), "=&s"(sv) : "v"(0u), "v"(1u), "s"(p) : "memory");
    return old;
}
#define ATOMIC_RESULT(dst, old, n) asm volatile("s_waitcnt vmcnt(" #n ")\n\tv_readfirstlane_b32 %0, %1" : "=s"(dst) : "v"(old) : "memory")
constexpr int AUX_SYS = 17;
constexpr int AUX_DEV = 16;

__device__ __forceinline__ bool spin_expired(gu32* tmo, unsigned call, unsigned long long t0) {
    if (rfl(__hip_atomic_load(tmo, RLX_AGENT)) == call || rt() - t0 > DEADLINE) {
        __hip_atomic_store(tmo, call, RLX_AGENT);
        return true;
    }
    return false;
}

#define STAMP(idx) do {} while (0)

}

__global__ void __launch_bounds__(NTHREADS, 3)
gemm_ar_q8(const float* __restrict__ x, const float* __restrict__ w, void* const* __restrict__ d_out_all,
           void* const* __restrict__ d_ws_all, int rank) {
    __shared__ __attribute__((aligned(1024))) unsigned char smem[LDS_BYTES];
    const unsigned lbase = (unsigned)(size_t)(LAS unsigned char*)smem;
    const int lane = threadIdx.x & 63;
    const int wave = (int)rfl(threadIdx.x >> 6);
    const int g = blockIdx.x & 7, gi = blockIdx.x >> 3;
    char* const ws = (char*)d_ws_all[rank];
    gu32* const tmo = (gu32*)(ws + TMO);
    const int tm = g * 4 + (gi >> 3), tn = gi & 7;

    if (wave < 8) {
        const int wm = wave >> 2, wn = wave & 3;
        f32x4 acc[4][4];
#pragma unroll
        for (int i = 0; i < 4; ++i)
#pragma unroll
            for (int j = 0; j < 4; ++j) acc[i][j] = (f32x4){0.f, 0.f, 0.f, 0.f};
        BAR();
        const LAS unsigned char* abase = (const LAS unsigned char*)smem + wm * 4096 + lane * 16;
        const LAS unsigned char* bbase = (const LAS unsigned char*)smem + 8192 + wn * 4096 + lane * 16;
        volatile LAS v2u* full = (volatile LAS v2u*)(smem + L_FULL);
        unsigned slot_off = 0;
        for (unsigned s = 0; s < (unsigned)NSTEP; ++s) {
            for (;;) {
                const v2u f = *full;
                if (rfl(f.x) > s && rfl(f.y) > s) break;
                __builtin_amdgcn_s_sleep(1);
            }
            asm volatile("" ::: "memory");
            v4u a[4], b[4];
#pragma unroll
            for (int i = 0; i < 4; ++i) b[i] = *(const LAS v4u*)(bbase + slot_off + i * 1024);
#pragma unroll
            for (int i = 0; i < 4; ++i) a[i] = *(const LAS v4u*)(abase + slot_off + i * 1024);
            LGKM0();
            lds_st(lbase + L_FREE + wave * 4, s + 1);
            asm volatile("" : "+v"(a[0]), "+v"(a[1]), "+v"(a[2]), "+v"(a[3]), "+v"(b[0]), "+v"(b[1]), "+v"(b[2]), "+v"(b[3]) :: "memory");
#pragma unroll
            for (int i = 0; i < 4; ++i)
#pragma unroll
                for (int j = 0; j < 4; ++j)
                    acc[i][j] = __builtin_amdgcn_mfma_f32_16x16x32_bf16(__builtin_bit_cast(bf16x8, b[j]), __builtin_bit_cast(bf16x8, a[i]), acc[i][j], 0, 0, 0);
            slot_off += SLOT_BYTES;
            if (slot_off == NSLOT * SLOT_BYTES) slot_off = 0;
        }
        if (wave == 0) STAMP(4);
        float mx = 0.f;
#pragma unroll
        for (int i = 0; i < 4; ++i)
#pragma unroll
            for (int j = 0; j < 4; ++j)
#pragma unroll
                for (int r = 0; r < 4; ++r) mx = fmaxf(mx, fabsf(acc[i][j][r]));
#pragma unroll
        for (int o = 32; o > 0; o >>= 1) mx = fmaxf(mx, __shfl_xor(mx, o));
        lds_st(lbase + L_WMAX + wave * 4, __builtin_bit_cast(unsigned, mx));
        BAR();
        BAR();
        const float inv = __builtin_bit_cast(float, lds_ld(lbase + L_MISC));
        const unsigned call = lds_ld(lbase + L_MISC + 4);
        const unsigned bad = lds_ld(lbase + L_MISC + 8);
        {
            const int r16 = lane & 15, q = lane >> 4;
#pragma unroll
            for (int i = 0; i < 4; ++i) {
                v4u o;
#pragma unroll
                for (int r = 0; r < 4; ++r) {
                    int d = __builtin_amdgcn_cvt_pk_fp8_f32(acc[i][0][r] * inv, acc[i][1][r] * inv, 0, false);
                    d = __builtin_amdgcn_cvt_pk_fp8_f32(acc[i][2][r] * inv, acc[i][3][r] * inv, d, true);
                    o[r] = bad ? 0x7f7f7f7fu : (unsigned)d;
                }
                const int row = wm * 64 + i * 16 + r16;
                const int ch = (wn * 4 + q) ^ r16;
                *(LAS v4u*)(smem + row * 256 + ch * 16) = o;
            }
        }
        BAR();
        {
            const __amdgpu_buffer_rsrc_t ro = mk_rsrc(d_out_all[rank], OUT_BYTES);
#pragma unroll
            for (int u = 0; u < 4; ++u) {
                const int row = wave * 16 + u * 4 + (lane >> 4), c = lane & 15;
                const v4u v = *(const LAS v4u*)(smem + row * 256 + ((c ^ (row & 15)) * 16));
                __builtin_amdgcn_raw_buffer_store_b128(v, ro, (tm * 128 + row) * NN + rank * NS + tn * 256 + c * 16, 0, AUX_SYS);
            }
            VMCNT(0);
            if (lane == 0) {
                const unsigned old = __hip_atomic_fetch_add((LAS unsigned*)(smem + L_MISC + 12), 1u, __ATOMIC_RELAXED, __HIP_MEMORY_SCOPE_WORKGROUP);
                if (old == 7) __hip_atomic_store((gu32*)(ws + ODONE) + blockIdx.x, call, RLX_AGENT);
            }
        }
        if (wave == 0) STAMP(7);
    } else if (wave < 10) {
        const int lw = wave - 8;
        BAR();
        const char* stage = ws + STAGE_OFF;
        const unsigned voff = lane * 16;
        unsigned slot_off = 0, pub = 0;
        const unsigned a_full = lbase + L_FULL + lw * 4;
        for (unsigned s = 0; s < (unsigned)NSTEP; ++s) {
            unsigned e = rfl(lds_ld(lbase + L_SCHED + s * 4));
            if (e == 0xffffffffu) {
                if (pub < s) { VMCNT(0); pub = s; lds_st(a_full, pub); }
                do { __builtin_amdgcn_s_sleep(1); e = rfl(lds_ld(lbase + L_SCHED + s * 4)); } while (e == 0xffffffffu);
            }
            if (s >= (unsigned)NSLOT) {
                const unsigned need = s - NSLOT + 1;
                if (!__all(lds_ld(lbase + L_FREE + (lane & 7) * 4) >= need)) {
                    if (pub < s) { VMCNT(0); pub = s; lds_st(a_full, pub); }
                    do { __builtin_amdgcn_s_sleep(1); } while (!__all(lds_ld(lbase + L_FREE + (lane & 7) * 4) >= need));
                }
            }
            const char* blk = stage + (size_t)(e & 127u) * BLK_BYTES;
            const unsigned dst = lbase + slot_off;
            if (lw == 0) {
                glds4(voff, blk + tm * 8192, dst);
                glds4(voff, blk + tm * 8192 + 4096, dst + 4096);
                glds4(voff, blk + A_BYTES + tn * 16384, dst + 8192);
            } else {
                glds4(voff, blk + A_BYTES + tn * 16384 + 4096, dst + 12288);
                glds4(voff, blk + A_BYTES + tn * 16384 + 8192, dst + 16384);
                glds4(voff, blk + A_BYTES + tn * 16384 + 12288, dst + 20480);
            }
            if (s + 1 - pub > 3) { VMCNT(36); pub = s - 2; lds_st(a_full, pub); }
            slot_off += SLOT_BYTES;
            if (slot_off == NSLOT * SLOT_BYTES) slot_off = 0;
        }
        if (pub < NSTEP - 2) { VMCNT(24); lds_st(a_full, NSTEP - 2); }
        if (pub < NSTEP - 1) { VMCNT(12); lds_st(a_full, NSTEP - 1); }
        VMCNT(0); lds_st(a_full, NSTEP);
        BAR();
        BAR();
        BAR();
    } else if (wave == 10) {
        unsigned tk = 0;
        if (lane == 0) tk = __hip_atomic_fetch_add((gu32*)(ws + TICKS + g * 128), 1u, RLX_AGENT);
        lds_st(lbase + L_SCHED + lane * 4, 0xffffffffu);
        lds_st(lbase + L_SCHED + 256 + lane * 4, 0xffffffffu);
        if (lane < 24) lds_st(lbase + L_FULL + lane * 4, 0u);
        BAR();
        STAMP(0);
        const unsigned call = (rfl(tk) >> 5) + 1;
        bool bad = false;
        __builtin_amdgcn_fence(__ATOMIC_ACQUIRE, "");
        VMCNT(0);
        {
            gu32* fl = (gu32*)(ws + AFLAG + g * 256) + lane;
            unsigned long long done = 0;
            unsigned cnt = 0, spins = 0;
            const unsigned long long t0 = rt();
            while (done != ~0ull) {
                unsigned long long ready = __ballot(__hip_atomic_load(fl, RLX_SYS) >= call);
                if (bad) ready = ~0ull;
                unsigned long long nw = ready & ~done;
                if (nw) {
                    done |= nw;
                    while (nw) {
                        const int bit = __builtin_ctzll(nw);
                        nw &= nw - 1;
                        const int c = bit >> 3, src = bit & 7;
                        const int k0 = ch_end(c - 1), k1 = ch_end(c);
                        if (lane < k1 - k0) lds_st(lbase + L_SCHED + (cnt + lane) * 4, (unsigned)(src * KBS + k0 + lane));
                        cnt += k1 - k0;
                    }
                } else {
                    __builtin_amdgcn_s_sleep(8);
                    if ((++spins & 63u) == 0u && spin_expired(tmo, call, t0)) bad = true;
                }
            }
        }
        STAMP(3);
        BAR();
        {
            unsigned m = lds_ld(lbase + L_WMAX + (lane & 7) * 4);
#pragma unroll
            for (int o = 4; o > 0; o >>= 1) m = max(m, (unsigned)__shfl_xor((int)m, o));
            if (lane == 0) __hip_atomic_store((gu64*)(ws + GMAX + g * 256) + gi, ((unsigned long long)call << 32) | m, RLX_AGENT);
            const unsigned long long t0 = rt();
            unsigned spins = 0;
            if (gi == 0) {
                unsigned long long v;
                for (;;) {
                    v = __hip_atomic_load((gu64*)(ws + GMAX + g * 256) + (lane & 31), RLX_AGENT);
                    if (__all((unsigned)(v >> 32) == call)) break;
                    __builtin_amdgcn_s_sleep(1);
                    if ((++spins & 63u) == 0u && spin_expired(tmo, call, t0)) { bad = true; break; }
                }
                m = (unsigned)v;
#pragma unroll
                for (int o = 16; o > 0; o >>= 1) m = max(m, (unsigned)__shfl_xor((int)m, o));
                char* pw = (char*)d_ws_all[lane >> 3];
                __hip_atomic_store((gu64*)(pw + RMAX + (lane & 7) * 512) + (rank * 8 + g), ((unsigned long long)call << 32) | m, RLX_SYS);
            }
            unsigned long long v;
            for (;;) {
                v = __hip_atomic_load((gu64*)(ws + RMAX + g * 512) + lane, RLX_SYS);
                if (__all((unsigned)(v >> 32) == call)) break;
                __builtin_amdgcn_s_sleep(1);
                if ((++spins & 63u) == 0u && spin_expired(tmo, call, t0)) { bad = true; break; }
            }
            m = (unsigned)v;
#pragma unroll
            for (int o = 32; o > 0; o >>= 1) m = max(m, (unsigned)__shfl_xor((int)m, o));
            if (rfl(__hip_atomic_load(tmo, RLX_AGENT)) == call) bad = true;
            const float amax = __builtin_bit_cast(float, m);
            const float scale = amax / 448.0f;
            const float inv = 1.0f / scale;
            if (lane == 0) {
                lds_st(lbase + L_MISC, __builtin_bit_cast(unsigned, inv));
                lds_st(lbase + L_MISC + 4, call);
                lds_st(lbase + L_MISC + 8, bad ? 1u : 0u);
            }
        }
        STAMP(5);
        BAR();
        BAR();
    } else {
        const unsigned p = (unsigned)(rank + 1 + g) & 7u;
        const unsigned tkv = atomic_inc_async(ws + TICKP + g * 128);
        char* const pws = (char*)d_ws_all[p];
        const __amdgpu_buffer_rsrc_t rs = mk_rsrc(pws + STAGE_OFF + (size_t)rank * SRC_BYTES, SRC_BYTES);
        const int r16 = lane & 15, q = lane >> 4;
        constexpr int RP = 32 / NPUSH;
        const int pi = gi / RP;
        const float* xa = x + (size_t)(pi * 64 + r16) * KS + 4 * q;
        const float* wb = w + (size_t)(4 * q) * NN + p * NS + pi * 64 + 4 * r16;
        f32x4 buf[8];
        v4u o[4];
#define LOAD_A(T, HALF) _Pragma("unroll") for (int e = 0; e < 8; ++e) \
            buf[e] = *(const f32x4*)(xa + (size_t)((HALF) * 2048 + ((T) % RP) * NPUSH * 64) * KS + ((T) / RP) * 32 + (e >> 1) * 16 * KS + (e & 1) * 16)
#define LOAD_B(T) _Pragma("unroll") for (int e = 0; e < 8; ++e) \
            buf[e] = *(const f32x4*)(wb + (size_t)(((T) / RP) * 32 + (e >> 2) * 16 + (e & 3)) * NN + ((T) % RP) * NPUSH * 64)
#define CVT_A() _Pragma("unroll") for (int f = 0; f < 4; ++f) { \
            o[f][0] = pk_bf16(buf[2*f][0], buf[2*f][1]); o[f][1] = pk_bf16(buf[2*f][2], buf[2*f][3]); \
            o[f][2] = pk_bf16(buf[2*f+1][0], buf[2*f+1][1]); o[f][3] = pk_bf16(buf[2*f+1][2], buf[2*f+1][3]); }
#define CVT_B() _Pragma("unroll") for (int j = 0; j < 4; ++j) { \
            o[j][0] = pk_bf16(buf[0][j], buf[1][j]); o[j][1] = pk_bf16(buf[2][j], buf[3][j]); \
            o[j][2] = pk_bf16(buf[4][j], buf[5][j]); o[j][3] = pk_bf16(buf[6][j], buf[7][j]); }
#define PIN_O() asm volatile("" : "+v"(o[0]), "+v"(o[1]), "+v"(o[2]), "+v"(o[3]) :: "memory")
#define ST1(F, OFF) __builtin_amdgcn_raw_buffer_store_b128(o[F], rs, lane * 16 + (F) * 1024 + (OFF), 0, AUX_SYS)
#define STORE4(OFF) do { VMCNT(9); ST1(0, OFF); VMCNT(9); ST1(1, OFF); VMCNT(1); ST1(2, OFF); VMCNT(1); ST1(3, OFF); } while (0)
#define TOFF(T) (((T) / RP) * BLK_BYTES + (pi + ((T) % RP) * NPUSH) * 4096)
        const bool pushing = (gi % RP) == 0;
        if (pushing) { LOAD_A(0, 0); }
        BAR();
        unsigned tk;
        ATOMIC_RESULT(tk, tkv, 0);
        const unsigned call = (tk >> 5) + 1;
        if (pushing) {
            CVT_A(); PIN_O();
            LOAD_A(0, 1);
            STORE4(TOFF(0));
            int c = 0;
            for (int t = 0; t < KBS * RP; ++t) {
                const unsigned boff = TOFF(t);
                CVT_A(); PIN_O();
                const bool sig = (t == ch_end(c) * RP);
                unsigned oldv = 0;
                if (sig) oldv = atomic_inc_async(ws + CNT1 + (g * 8 + c) * 128);
                LOAD_B(t);
                STORE4(boff + 32 * 4096);
                if (sig) {
                    unsigned old;
                    ATOMIC_RESULT(old, oldv, 12);
                    if (old + 1 == (unsigned)NPUSH * call && lane < 8)
                        __hip_atomic_store((gu32*)(pws + AFLAG + lane * 256) + (c * 8 + rank), call, RLX_SYS);
                    ++c;
                }
                CVT_B(); PIN_O();
                if (t + 1 < KBS * RP) {
                    LOAD_A(t + 1, 0);
                    STORE4(boff + A_BYTES);
                    CVT_A(); PIN_O();
                    LOAD_A(t + 1, 1);
                    STORE4(TOFF(t + 1));
                } else {
                    ST1(0, boff + A_BYTES); ST1(1, boff + A_BYTES); ST1(2, boff + A_BYTES); ST1(3, boff + A_BYTES);
                }
            }
            VMCNT(0);
            const unsigned oldv = atomic_inc_async(ws + CNT1 + (g * 8 + 7) * 128);
            unsigned old;
            ATOMIC_RESULT(old, oldv, 0);
            if (old + 1 == (unsigned)NPUSH * call && lane < 8)
                __hip_atomic_store((gu32*)(pws + AFLAG + lane * 256) + (7 * 8 + rank), call, RLX_SYS);
        }
        STAMP(2);
        BAR();
        BAR();
        BAR();
        STAMP(6);
        if (g != 7) {
            const __amdgpu_buffer_rsrc_t rd = mk_rsrc(d_out_all[p], OUT_BYTES);
            const __amdgpu_buffer_rsrc_t rl = mk_rsrc(d_out_all[rank], OUT_BYTES);
            for (int it = 0; it < 8; ++it) {
#pragma unroll
                for (int u = 0; u < 4; ++u) {
                    const int row = it * 16 + u * 4 + q;
                    o[u] = *(const LAS v4u*)(smem + row * 256 + ((r16 ^ (row & 15)) * 16));
                }
                PIN_O();
                VMCNT(4);
#pragma unroll
                for (int u = 0; u < 4; ++u)
                    __builtin_amdgcn_raw_buffer_store_b128(o[u], rd, (u * 4 + q) * NN + r16 * 16 + (tm * 128 + it * 16) * NN + rank * NS + tn * 256, 0, AUX_SYS);
            }
            bool bad = false;
            {
                const unsigned long long t0 = rt();
                unsigned spins = 0;
                const __amdgpu_buffer_rsrc_t rw = mk_rsrc(ws + ODONE, 1024);
                for (;;) {
                    const v4u f = __builtin_amdgcn_raw_buffer_load_b128(rw, lane * 16, 0, AUX_DEV);
                    if (__all(f[0] == call && f[1] == call && f[2] == call && f[3] == call)) break;
                    __builtin_amdgcn_s_sleep(2);
                    if ((++spins & 63u) == 0u && spin_expired(tmo, call, t0)) { bad = true; break; }
                    asm volatile("" ::: "memory");
                }
                asm volatile("" ::: "memory");
            }
            v4u ob[4];
            const unsigned vo = lane * 16;
#define ITEM_OFF(T) ((unsigned)(((((g + 1 + ((T) >> 3)) & 7) * 512 + ((T) & 7) * 64 + gi * 2)) * NN + rank * NS))
#define LOAD_O(DST, T) _Pragma("unroll") for (int u = 0; u < 4; ++u) \
            DST[u] = __builtin_amdgcn_raw_buffer_load_b128(rl, vo + (u >> 1) * NN + (u & 1) * 1024, ITEM_OFF(T), AUX_DEV)
#define STORE_O(SRC, T) _Pragma("unroll") for (int u = 0; u < 4; ++u) { \
            if (bad) SRC[u] = (v4u){0x7f7f7f7fu, 0x7f7f7f7fu, 0x7f7f7f7fu, 0x7f7f7f7fu}; \
            __builtin_amdgcn_raw_buffer_store_b128(SRC[u], rd, vo + (u >> 1) * NN + (u & 1) * 1024 + ITEM_OFF(T), 0, AUX_SYS); }
#define THROTTLE2() VMCNT(8)
            LOAD_O(o, 0);
            for (int t = 0; t < 56; t += 2) {
                LOAD_O(ob, t + 1);
                THROTTLE2();
                STORE_O(o, t);
                if (t + 2 < 56) { LOAD_O(o, t + 2); THROTTLE2(); }
                STORE_O(ob, t + 1);
            }
            VMCNT(0);
            if (lane == 0) __hip_atomic_store((gu32*)(pws + FIN + rank * 128) + gi, call, RLX_SYS);
            STAMP(8);
        } else if (gi == 0) {
            const unsigned long long t0 = rt();
            unsigned spins = 0;
            const __amdgpu_buffer_rsrc_t rw = mk_rsrc(ws + FIN, 1024);
            for (;;) {
                const v4u f = __builtin_amdgcn_raw_buffer_load_b128(rw, lane * 16, 0, AUX_SYS);
                if (__all((lane >> 3) == rank || (f[0] >= call && f[1] >= call && f[2] >= call && f[3] >= call))) break;
                __builtin_amdgcn_s_sleep(2);
                if ((++spins & 63u) == 0u && spin_expired(tmo, call, t0)) {
                    if (lane == 0) ((unsigned char*)d_out_all[rank])[(size_t)tm * 128 * NN + rank * NS + tn * 256] = 0x7f;
                    break;
                }
                asm volatile("" ::: "memory");
            }
            __builtin_amdgcn_fence(__ATOMIC_ACQUIRE, "");
            STAMP(9);
        }
    }
}

extern "C" void kernel_launch(int rank, int nranks, ncclComm_t comm, hipStream_t stream,
                              void* const* d_in, const int* in_sizes, int n_in,
                              void* d_out, int out_size,
                              const int* in_dtypes, int out_dtype,
                              void* const* d_out_all,
                              void* const* d_ws_all, size_t ws_bytes) {
    (void)comm; (void)d_out; (void)in_dtypes; (void)out_dtype;
    if (nranks != NR || (unsigned)rank >= (unsigned)NR || n_in < 2 || ws_bytes < WS_NEED ||
        in_sizes[0] != MM * KS || in_sizes[1] != KS * NN || out_size != MM * NN) {
        fprintf(stderr, "kernel_launch: unexpected problem shape\n");
        return;
    }
    hipLaunchKernelGGL(gemm_ar_q8, dim3(NWG), dim3(NTHREADS), 0, stream,
                       (const float*)d_in[0], (const float*)d_in[1], d_out_all, d_ws_all, rank);
    HIP_CHECK(hipGetLastError());
}
```

```cpp
#include <hip/hip_runtime.h>
#include <rccl/rccl.h>
#include <cstdio>
#include <cstdint>

#define HIP_CHECK(expr) do { hipError_t _e = (expr); if (_e != hipSuccess) { \
    fprintf(stderr, "HIP error %s at %s:%d\n", hipGetErrorString(_e), __FILE__, __LINE__); \
    return; } } while (0)

#define GAS __attribute__((address_space(1)))
#define LAS __attribute__((address_space(3)))
typedef unsigned v4u __attribute__((ext_vector_type(4)));
typedef unsigned v2u __attribute__((ext_vector_type(2)));
typedef float f32x4 __attribute__((ext_vector_type(4)));
typedef short bf16x8 __attribute__((ext_vector_type(8)));
typedef __bf16 bf2 __attribute__((ext_vector_type(2)));
typedef GAS unsigned gu32;
typedef GAS unsigned long long gu64;

namespace {

constexpr int NR = 8;
constexpr int MM = 4096, NN = 16384;
constexpr int KS = 512;
constexpr int NS = 2048;
constexpr int KBS = 16;
constexpr unsigned A_BYTES = 256u * 1024u;
constexpr unsigned B_BYTES = 128u * 1024u;
constexpr unsigned BLK_BYTES = A_BYTES + B_BYTES;
constexpr unsigned SRC_BYTES = KBS * BLK_BYTES;
constexpr size_t STAGE_OFF = 1u << 20;
constexpr size_t WS_NEED = STAGE_OFF + (size_t)NR * SRC_BYTES;
constexpr unsigned OUT_BYTES = (unsigned)MM * NN;

constexpr unsigned DRAW1 = 0x0000;
constexpr unsigned DRAW2 = 0x0C00;
constexpr unsigned PRIV  = 0x20000;
constexpr unsigned CLAIM = 0x24000;
constexpr unsigned TMO   = 0x0800;
constexpr unsigned CNT1  = 0xA000;
#define AFS 512
constexpr unsigned AFLAG = 0x40000;
constexpr unsigned MAILB = 0x4000;
constexpr unsigned GMAX  = 0x5000;
constexpr unsigned RMAX  = 0x6000;
constexpr unsigned RCNT  = 0x8000;
constexpr unsigned RDY   = 0x8400;
constexpr unsigned FIN   = 0x9000;
constexpr unsigned PROF  = 0x10000;

constexpr int NWG = 256, NTHREADS = 832;
constexpr int NSLOT = 6;
constexpr unsigned SLOT_BYTES = 24576;
constexpr unsigned L_FULL = NSLOT * SLOT_BYTES;
constexpr unsigned L_FREE = L_FULL + 16;
constexpr unsigned L_WMAX = L_FREE + 32;
constexpr unsigned L_MISC = L_WMAX + 32;
constexpr unsigned L_PTAG = L_MISC + 16;
constexpr unsigned L_PCALL = L_PTAG + 16;
constexpr unsigned L_SCHED = L_PCALL + 16;
constexpr unsigned L_PSLOT = L_SCHED + 512;
constexpr unsigned NPS = 3;
constexpr unsigned LDS_BYTES = L_PSLOT + NPS * 4096;
static_assert(LDS_BYTES <= 163840 && L_PSLOT % 16 == 0 && L_MISC % 16 == 0, "LDS layout");
constexpr int NSTEP = NR * KBS;
constexpr unsigned long long DEADLINE = 100000000ull;

#define WIN1 2
#define WIN2 2
#define ACQ_EACH 0
#define POLL_REP 1
#define FASTCNT2 124u
#define POLL_MID 32
#define FASTCNT 112u
#define MBOX 0
#define POLL_ALWAYS 127
#define POLL_JIT 0
#define POLL_SLEEP 16
#define FDEPTH 2
#define LDEPTH0 5
#define LSWITCH 112u
#define LDEPTH 5
constexpr unsigned NITEM1 = KBS * 96;
constexpr int SELFG = 3;
#define CPRIO 0
#define LDELAY 1
#define DPUSH 2
constexpr unsigned BLK_ITEMS = 64 - 8 * DPUSH;
constexpr unsigned NITEM2 = 7 * 256 + 4 * BLK_ITEMS;

__device__ __forceinline__ unsigned dest_of(int rank, int g) {
    const unsigned set = (0xB2u >> rank) & 1u;
    const unsigned pos = (0x33212100u >> (4 * rank)) & 3u;
    const unsigned members = ((g < 4) == (set == 0)) ? 0x6320u : 0x7541u;
    return (members >> (4 * ((pos + 1 + g) & 3))) & 7u;
}

#define VMCNT(n) asm volatile("s_waitcnt vmcnt(" #n ")" ::: "memory")
#define WAITW2() VMCNT(1)
#define LGKM0() asm volatile("s_waitcnt lgkmcnt(0)" ::: "memory")
#define BAR() do { asm volatile("s_waitcnt lgkmcnt(0)" ::: "memory"); __builtin_amdgcn_s_barrier(); asm volatile("" ::: "memory"); } while (0)
#define RLX_AGENT __ATOMIC_RELAXED, __HIP_MEMORY_SCOPE_AGENT
#define RLX_SYS __ATOMIC_RELAXED, __HIP_MEMORY_SCOPE_SYSTEM

__device__ __forceinline__ unsigned long long rt() { return __builtin_amdgcn_s_memrealtime(); }
__device__ __forceinline__ unsigned rfl(unsigned v) { return (unsigned)__builtin_amdgcn_readfirstlane((int)v); }
__device__ __forceinline__ unsigned pk_bf16(float lo, float hi) {
    bf2 v; v.x = (__bf16)lo; v.y = (__bf16)hi; return __builtin_bit_cast(unsigned, v);
}
__device__ __forceinline__ unsigned wave_umax(unsigned v) {
    v = max(v, (unsigned)__builtin_amdgcn_update_dpp(0, (int)v, 0x128, 0xf, 0xf, false));
    v = max(v, (unsigned)__builtin_amdgcn_update_dpp(0, (int)v, 0x124, 0xf, 0xf, false));
    v = max(v, (unsigned)__builtin_amdgcn_update_dpp(0, (int)v, 0x122, 0xf, 0xf, false));
    v = max(v, (unsigned)__builtin_amdgcn_update_dpp(0, (int)v, 0x121, 0xf, 0xf, false));
    const unsigned a = (unsigned)__builtin_amdgcn_readlane((int)v, 0), b = (unsigned)__builtin_amdgcn_readlane((int)v, 16);
    const unsigned c = (unsigned)__builtin_amdgcn_readlane((int)v, 32), d = (unsigned)__builtin_amdgcn_readlane((int)v, 48);
    return max(max(a, b), max(c, d));
}
__device__ __forceinline__ void lds_st(unsigned addr, unsigned v) { asm volatile("ds_write_b32 %0, %1" :: "v"(addr), "v"(v) : "memory"); }
__device__ __forceinline__ unsigned lds_ld(unsigned addr) {
    unsigned v; asm volatile("ds_read_b32 %0, %1\n\ts_waitcnt lgkmcnt(0)" : "=&v"(v) : "v"(addr) : "memory"); return v;
}
__device__ __forceinline__ void glds4(unsigned voff, const char* sbase, unsigned m0v) {
    unsigned keep;
#define GPOL ""
    asm volatile("s_mov_b32 %0, m0\n\ts_mov_b32 m0, %3\n\ts_nop 4\n\t"
                 "global_load_lds_dwordx4 %1, %2" GPOL "\n\t"
                 "global_load_lds_dwordx4 %1, %2 offset:1024" GPOL "\n\t"
                 "global_load_lds_dwordx4 %1, %2 offset:2048" GPOL "\n\t"
                 "global_load_lds_dwordx4 %1, %2 offset:3072" GPOL "\n\t"
                 "s_mov_b32 m0, %0"
                 : "=&s"(keep) : "v"(voff), "s"(sbase), "s"(m0v) : "memory");
}
__device__ __forceinline__ __amdgpu_buffer_rsrc_t mk_rsrc(void* base, unsigned bytes) {
    return __builtin_amdgcn_make_buffer_rsrc(base, 0, (int)bytes, 0x00020000);
}
__device__ __forceinline__ unsigned atomic_inc_async(const void* p) {
    unsigned old; unsigned long long sv;
    asm volatile("s_mov_b64 %1, exec\n\ts_mov_b64 exec, 1\n\ts_nop 3\n\t"
                 "global_atomic_add %0, %2, %3, %4 sc0\n\t"
                 "s_mov_b64 exec, %1"
                 : "=&v"(old), "=&s"(sv) : "v"(0u), "v"(1u), "s"(p) : "memory");
    return old;
}
__device__ __forceinline__ unsigned atomic_inc_masked_async(const void* p, unsigned mask, unsigned lane4) {
    unsigned old = 0; unsigned long long sv;
    asm volatile("s_mov_b64 %1, exec\n\ts_mov_b32 exec_lo, %5\n\ts_mov_b32 exec_hi, 0\n\ts_nop 2\n\t"
                 "global_atomic_add %0, %2, %3, %4 sc0\n\t"
                 "s_mov_b64 exec, %1"
                 : "+v"(old), "=&s"(sv) : "v"(lane4), "v"(1u), "s"(p), "s"(rfl(mask)) : "memory");
    return old;
}
#define ATOMIC_WAIT(old, n) asm volatile("s_waitcnt vmcnt(" #n ")" : "+v"(old) :: "memory")
#define ATOMIC_RESULT(dst, old, n) asm volatile("s_waitcnt vmcnt(" #n ")\n\tv_readfirstlane_b32 %0, %1\n\ts_nop 4" : "=s"(dst) : "v"(old) : "memory")
constexpr int AUX_SYS = 17;
constexpr int AUX_DEV = 16;
#define AUX_LOC 16

__device__ __forceinline__ bool spin_expired(gu32* tmo, unsigned call, unsigned long long t0) {
    if (rfl(__hip_atomic_load(tmo, RLX_AGENT)) == call || rt() - t0 > DEADLINE) {
        __hip_atomic_store(tmo, call, RLX_AGENT);
        return true;
    }
    return false;
}

#define STAMP(idx) do {} while (0)

}

__global__ void __launch_bounds__(NTHREADS, 4)
gemm_ar_q8(const float* __restrict__ x, const float* __restrict__ w, void* const* __restrict__ d_out_all,
           void* const* __restrict__ d_ws_all, int rank) {
    __shared__ __attribute__((aligned(1024))) unsigned char smem[LDS_BYTES];
    const unsigned lbase = (unsigned)(size_t)(LAS unsigned char*)smem;
    const int lane = threadIdx.x & 63;
    const int wave = (int)rfl(threadIdx.x >> 6);
    const int gi = blockIdx.x >> 3;
    char* const ws = (char*)d_ws_all[rank];
    gu32* const tmo = (gu32*)(ws + TMO);
    const int tn = gi & 7;
#define GET_GROUP() const v2u gc_ = *(volatile LAS v2u*)(smem + L_PCALL); \
    const unsigned call = rfl(gc_[0]); const int g = (int)(rfl(gc_[1]) & 7u); const int tm = g * 4 + (gi >> 3); (void)call; (void)tm

    if (wave < 8) {
        const int wm = wave >> 2, wn = wave & 3;
        f32x4 acc[4][4];
#pragma unroll
        for (int i = 0; i < 4; ++i)
#pragma unroll
            for (int j = 0; j < 4; ++j) acc[i][j] = (f32x4){0.f, 0.f, 0.f, 0.f};
        BAR();
        GET_GROUP();
        const LAS unsigned char* abase = (const LAS unsigned char*)smem + wm * 4096 + lane * 16;
        const LAS unsigned char* bbase = (const LAS unsigned char*)smem + 8192 + wn * 4096 + lane * 16;
        volatile LAS v2u* full = (volatile LAS v2u*)(smem + L_FULL);
        unsigned slot_off = 0;
        for (unsigned s = 0; s < (unsigned)NSTEP; ++s) {
            for (;;) {
                const v2u f = *full;
                if (rfl(f.x) > s && rfl(f.y) > s) break;
                __builtin_amdgcn_s_sleep(1);
            }
            asm volatile("" ::: "memory");
            v4u a[4], b[4];
#pragma unroll
            for (int i = 0; i < 4; ++i) b[i] = *(const LAS v4u*)(bbase + slot_off + i * 1024);
#pragma unroll
            for (int i = 0; i < 4; ++i) a[i] = *(const LAS v4u*)(abase + slot_off + i * 1024);
            LGKM0();
            lds_st(lbase + L_FREE + wave * 4, s + 1);
            asm volatile("" : "+v"(a[0]), "+v"(a[1]), "+v"(a[2]), "+v"(a[3]), "+v"(b[0]), "+v"(b[1]), "+v"(b[2]), "+v"(b[3]) :: "memory");
#pragma unroll
            for (int i = 0; i < 4; ++i)
#pragma unroll
                for (int j = 0; j < 4; ++j)
                    acc[i][j] = __builtin_amdgcn_mfma_f32_16x16x32_bf16(__builtin_bit_cast(bf16x8, b[j]), __builtin_bit_cast(bf16x8, a[i]), acc[i][j], 0, 0, 0);
            slot_off += SLOT_BYTES;
            if (slot_off == NSLOT * SLOT_BYTES) slot_off = 0;
        }
        if (wave == 0) STAMP(4);
        float mx = 0.f;
#pragma unroll
        for (int i = 0; i < 4; ++i)
#pragma unroll
            for (int j = 0; j < 4; ++j)
#pragma unroll
                for (int r = 0; r < 4; ++r) mx = fmaxf(mx, fabsf(acc[i][j][r]));
        lds_st(lbase + L_WMAX + wave * 4, wave_umax(__builtin_bit_cast(unsigned, mx)));
        BAR();
        BAR();
        const v4u misc = *(volatile LAS v4u*)(smem + L_MISC);
        const float inv = __builtin_bit_cast(float, misc[0]);
        const unsigned bad = misc[2];
        {
            const int r16 = lane & 15, q = lane >> 4;
#pragma unroll
            for (int i = 0; i < 4; ++i) {
                v4u o;
#pragma unroll
                for (int r = 0; r < 4; ++r) {
                    int d = __builtin_amdgcn_cvt_pk_fp8_f32(acc[i][0][r] * inv, acc[i][1][r] * inv, 0, false);
                    d = __builtin_amdgcn_cvt_pk_fp8_f32(acc[i][2][r] * inv, acc[i][3][r] * inv, d, true);
                    o[r] = bad ? 0x7f7f7f7fu : (unsigned)d;
                }
                const int row = wm * 64 + i * 16 + r16;
                const int ch = (wn * 4 + q) ^ r16;
                *(LAS v4u*)(smem + row * 256 + ch * 16) = o;
            }
        }
        char* pw = (char*)d_ws_all[lane & 7];
        BAR();
        if (g != SELFG) while (rfl(lds_ld(lbase + L_PCALL + 8)) == 0u) __builtin_amdgcn_s_sleep(4);
        {
            const __amdgpu_buffer_rsrc_t ro = mk_rsrc(d_out_all[rank], OUT_BYTES);
#pragma unroll
            for (int u = 0; u < 4; ++u) {
                const int row = wave * 16 + u * 4 + (lane >> 4), c = lane & 15;
                const v4u v = *(const LAS v4u*)(smem + row * 256 + ((c ^ (row & 15)) * 16));
                __builtin_amdgcn_raw_buffer_store_b128(v, ro, (tm * 128 + row) * NN + rank * NS + tn * 256 + c * 16, 0, AUX_SYS);
            }
            VMCNT(0);
            unsigned old = 0;
            if (lane == 0) old = __hip_atomic_fetch_add((LAS unsigned*)(smem + L_MISC + 12), 1u, __ATOMIC_RELAXED, __HIP_MEMORY_SCOPE_WORKGROUP);
            if (rfl(old) == 7) {
                const unsigned cv = atomic_inc_async(ws + RCNT + g * 128);
                unsigned oc;
                ATOMIC_RESULT(oc, cv, 0);
                if (oc + 1 == 32u * call && lane < 8) __hip_atomic_store((gu32*)(pw + RDY + rank * 128) + g, call, RLX_SYS);
                STAMP(7);
            }
        }
    } else if (wave < 10) {
        const int lw = wave - 8;
        BAR();
        GET_GROUP();
        const char* stage = ws + STAGE_OFF;
        const unsigned voff = lane * 16;
        unsigned slot_off = 0, pub = 0, s = 0;
        const unsigned a_full = lbase + L_FULL + lw * 4;
        while (pub < (unsigned)NSTEP) {
            bool go = false;
            unsigned e = 0;
            if (s < (unsigned)NSTEP && s - pub < (s < LSWITCH ? (unsigned)LDEPTH0 : (unsigned)LDEPTH)) {
                const unsigned v = lds_ld(lane < 8 ? lbase + L_FREE + lane * 4 : lbase + L_SCHED + s * 4);
                e = (unsigned)__builtin_amdgcn_readlane((int)v, 8);
                go = e != 0xffffffffu && __all(lane >= 8 || v + NSLOT > s);
            }
            if (go) {
                const char* blk = stage + (size_t)(e & 127u) * BLK_BYTES;
                const unsigned dst = lbase + slot_off;
                if (lw == 0) {
                    glds4(voff, blk + tm * 8192, dst);
                    glds4(voff, blk + tm * 8192 + 4096, dst + 4096);
                    glds4(voff, blk + A_BYTES + tn * 16384, dst + 8192);
                } else {
                    glds4(voff, blk + A_BYTES + tn * 16384 + 4096, dst + 12288);
                    glds4(voff, blk + A_BYTES + tn * 16384 + 8192, dst + 16384);
                    glds4(voff, blk + A_BYTES + tn * 16384 + 12288, dst + 20480);
                }
                ++s;
                slot_off += SLOT_BYTES;
                if (slot_off == NSLOT * SLOT_BYTES) slot_off = 0;
            } else if (pub < s) {
                const unsigned infl = s - pub;
                if (infl == 1) VMCNT(0); else if (infl == 2) VMCNT(12); else if (infl == 3) VMCNT(24);
                else if (infl == 4) VMCNT(36); else VMCNT(48);
                ++pub;
                lds_st(a_full, pub);
            } else {
                __builtin_amdgcn_s_sleep(1);
            }
        }
        BAR();
        BAR();
        BAR();
    } else if (wave == 10) {
        unsigned xcc;
        asm volatile("s_getreg_b32 %0, hwreg(HW_REG_XCC_ID)" : "=s"(xcc));
        int g = (int)(xcc & 7u);
        unsigned tk = 0;
        if (lane < 2) tk = __hip_atomic_fetch_add((gu32*)(ws + (lane == 0 ? PRIV + blockIdx.x * 64 : CLAIM + (gi * 8 + g) * 64)), 1u, RLX_AGENT);
        lds_st(lbase + L_SCHED + lane * 4, 0xffffffffu);
        lds_st(lbase + L_SCHED + 256 + lane * 4, 0xffffffffu);
        if (lane < 28) lds_st(lbase + L_FULL + lane * 4, 0u);
        const unsigned call = rfl(tk) + 1;
        unsigned got = (unsigned)__builtin_amdgcn_readlane((int)tk, 1) + 1;
        for (int tries = 0; got != call && tries < 7; ++tries) {
            unsigned o2 = 0;
            if (lane == 0) {
                __hip_atomic_fetch_add((gu32*)(ws + CLAIM + (gi * 8 + g) * 64), 0xffffffffu, RLX_AGENT);
                o2 = __hip_atomic_fetch_add((gu32*)(ws + CLAIM + (gi * 8 + ((g + 1) & 7)) * 64), 1u, RLX_AGENT);
            }
            g = (g + 1) & 7;
            got = rfl(o2) + 1;
        }
        if (lane < 3) lds_st(lbase + L_PCALL + lane * 4, lane == 0 ? call : lane == 1 ? (unsigned)g : 0u);
        BAR();
        STAMP(0);
        bool bad = false;
        __builtin_amdgcn_fence(__ATOMIC_ACQUIRE, "");
        VMCNT(0);
        {
            gu32* fl = (gu32*)(ws + AFLAG + g * AFS) + lane;
            unsigned long long done0 = 0, done1 = 0;
            unsigned cnt = 0, spins = 0;
            const unsigned ent = (lane & 7) * KBS + (lane >> 3);
            const unsigned long long t0 = rt();
            unsigned long long tlast = 0; (void)tlast;
            while ((done0 & done1) != ~0ull) {
                const unsigned f0 = __hip_atomic_load(fl + (done0 == ~0ull ? 64 : 0), RLX_SYS);
                const unsigned f1 = __hip_atomic_load(fl + 64, RLX_SYS);
                const unsigned long long n0 = __ballot(bad || f0 >= call) & ~done0;
                const unsigned long long n1 = __ballot(bad || f1 >= call) & ~done1;
                if (n0 | n1) {
                    const unsigned long long lt = (1ull << lane) - 1;
                    const unsigned c0 = __popcll(n0), c1 = __popcll(n1);
                    if ((n0 >> lane) & 1) lds_st(lbase + L_SCHED + (cnt + __popcll(n0 & lt)) * 4, ent);
                    if ((n1 >> lane) & 1) lds_st(lbase + L_SCHED + (cnt + c0 + __popcll(n1 & lt)) * 4, ent + 8);
                    cnt += c0 + c1;
                    done0 |= n0; done1 |= n1;
                    if (cnt < FASTCNT) for (int j = 0; j < POLL_REP; ++j) __builtin_amdgcn_s_sleep(POLL_ALWAYS);
                    else if (cnt < FASTCNT2) __builtin_amdgcn_s_sleep(POLL_MID);
                } else {
                    if (cnt < FASTCNT) __builtin_amdgcn_s_sleep(POLL_SLEEP); else if (cnt < FASTCNT2) __builtin_amdgcn_s_sleep(POLL_MID); else __builtin_amdgcn_s_sleep(1);
                    if ((++spins & 63u) == 0u && spin_expired(tmo, call, t0)) bad = true;
                }
            }
        }
        STAMP(3);
        char* pw = (char*)d_ws_all[lane >> 3];
        asm volatile("" : "+v"(pw));
        BAR();
        {
            unsigned m = wave_umax(lds_ld(lbase + L_WMAX + (lane & 7) * 4));
            if (lane == 0) __hip_atomic_store((gu64*)(ws + GMAX + g * 256) + gi, ((unsigned long long)call << 32) | m, RLX_AGENT);
            const unsigned long long t0 = rt();
            unsigned spins = 0;
            if (gi == 0) {
                unsigned long long v;
                for (;;) {
                    v = __hip_atomic_load((gu64*)(ws + GMAX + g * 256) + (lane & 31), RLX_AGENT);
                    if (__all((unsigned)(v >> 32) == call)) break;
                    if ((++spins & 63u) == 0u && spin_expired(tmo, call, t0)) { bad = true; break; }
                }
                m = wave_umax((unsigned)v);
                __hip_atomic_store((gu64*)(pw + RMAX + (lane & 7) * 512) + (rank * 8 + g), ((unsigned long long)call << 32) | m, RLX_SYS);
            }
            unsigned long long v;
            for (;;) {
                v = __hip_atomic_load((gu64*)(ws + RMAX + g * 512) + lane, RLX_SYS);
                if (rfl(__hip_atomic_load(tmo, RLX_AGENT)) == call) bad = true;
                if (bad || __all((unsigned)(v >> 32) == call)) break;
                if ((++spins & 63u) == 0u && spin_expired(tmo, call, t0)) { bad = true; break; }
            }
            m = wave_umax((unsigned)v);
            const float inv = 448.0f / __builtin_bit_cast(float, m);
            if (lane < 3) lds_st(lbase + L_MISC + lane * 4, lane == 0 ? __builtin_bit_cast(unsigned, inv) : lane == 1 ? call : (bad ? 1u : 0u));
        }
        STAMP(5);
        BAR();
        BAR();
    } else if (wave == 11) {
        const int r16 = lane & 15, q = lane >> 4;
        const float* xa = x + (size_t)r16 * KS + 4 * q;
        const float* wb = w;
        f32x4 buf[8];
        v4u o[4];
#define LOAD_ITEM(KB, U) do { if ((U) < 64u) { const float* a_ = xa + (size_t)(U) * 64 * KS + (KB) * 32; \
            _Pragma("unroll") for (int e = 0; e < 8; ++e) buf[e] = *(const f32x4*)(a_ + (e >> 1) * 16 * KS + (e & 1) * 16); \
        } else { const float* b_ = wb + (size_t)(KB) * 32 * NN + ((U) - 64u) * 64; \
            _Pragma("unroll") for (int e = 0; e < 8; ++e) buf[e] = *(const f32x4*)(b_ + (size_t)((e >> 2) * 16 + (e & 3)) * NN); } } while (0)
#define CVT_ITEM(U) do { if ((U) < 64u) { _Pragma("unroll") for (int f = 0; f < 4; ++f) { \
            o[f][0] = pk_bf16(buf[2*f][0], buf[2*f][1]); o[f][1] = pk_bf16(buf[2*f][2], buf[2*f][3]); \
            o[f][2] = pk_bf16(buf[2*f+1][0], buf[2*f+1][1]); o[f][3] = pk_bf16(buf[2*f+1][2], buf[2*f+1][3]); } \
        } else { _Pragma("unroll") for (int j = 0; j < 4; ++j) { \
            o[j][0] = pk_bf16(buf[0][j], buf[1][j]); o[j][1] = pk_bf16(buf[2][j], buf[3][j]); \
            o[j][2] = pk_bf16(buf[4][j], buf[5][j]); o[j][3] = pk_bf16(buf[6][j], buf[7][j]); } } } while (0)
#define PIN_O() asm volatile("" : "+v"(o[0]), "+v"(o[1]), "+v"(o[2]), "+v"(o[3]) :: "memory")
#define SLOT_WAIT() while (rfl(lds_ld(lbase + L_PTAG + 12)) + FDEPTH <= n) __builtin_amdgcn_s_sleep(2)
#define SLOT_TAG(T) do { lds_st(lbase + L_PTAG + sl * 4, ((n + 1) << 16) | (T)); ++n; sl = sl + 1 == NPS ? 0 : sl + 1; } while (0)
#define SLOT_PUT(T) do { SLOT_WAIT(); \
            _Pragma("unroll") for (int f = 0; f < 4; ++f) *(LAS v4u*)(smem + L_PSLOT + sl * 4096 + f * 1024 + lane * 16) = o[f]; \
            SLOT_TAG(T); } while (0)
        unsigned n = 0, sl = 0;
#define ITEM_Z(KB, U) (((KB) == 0u ? ((U) < 64u ? ((U) + rank * 8) & 63u : (U)) : ((U) < 32u ? (U) + 64u : ((U) - 32u + rank * 8) & 63u)))
        unsigned cur = ((gi & 31u) + rank * 8) & 63u;
        LOAD_ITEM(0u, cur);
        BAR();
        GET_GROUP();
        const unsigned p = dest_of(rank, g);
        wb = w + (size_t)(4 * q) * NN + p * NS + 4 * r16;
        const char* const draw1 = ws + DRAW1 + g * 128;
        const unsigned base1 = (call - 1) * (NITEM1 - 32) - 64;
        unsigned nxt = (gi & 31u) + 32, dv;
        CVT_ITEM(cur); PIN_O();
        while (nxt < NITEM1) {
            const unsigned nkb = nxt / 96, nu0 = nxt - nkb * 96, nu = ITEM_Z(nkb, nu0);
            LOAD_ITEM(nkb, nu);
            dv = atomic_inc_async(draw1);
            SLOT_PUT(cur);
            unsigned d;
            ATOMIC_RESULT(d, dv, 0);
            CVT_ITEM(nu); PIN_O();
            cur = nkb * 96 + nu; nxt = d - base1;
        }
        SLOT_PUT(cur);
        SLOT_WAIT(); SLOT_TAG(0xffffu);
        BAR();
        BAR();
        BAR();
        if (p != (unsigned)rank) {
            const __amdgpu_buffer_rsrc_t rp = mk_rsrc(d_out_all[p], OUT_BYTES);
            const char* const draw2 = ws + DRAW2 + g * 128;
            dv = atomic_inc_async(draw2);
            unsigned gp = 0;
            for (int j = 0; j < 8; ++j) if (dest_of((int)p, j) == (unsigned)rank) gp = j;
            bool bad = false;
            unsigned rdy = 0;
#define NEED_REGION(R) do { if (!((rdy >> (R)) & 1u)) { \
                const unsigned long long t0 = rt(); unsigned spins = 0; \
                for (;;) { \
                    const unsigned f = __hip_atomic_load((gu32*)(ws + RDY + p * 128) + (lane & 7), RLX_SYS); \
                    rdy = (unsigned)__ballot(f >= call) & 0xffu; \
                    if ((rdy >> (R)) & 1u) break; \
                    __builtin_amdgcn_s_sleep(2); \
                    if ((++spins & 63u) == 0u && spin_expired(tmo, call, t0)) { bad = true; rdy = 0xffu; break; } \
                } \
                asm volatile("" ::: "memory"); } } while (0)
            const unsigned vo = lane * 16 + p * NS;
            const unsigned base2 = (call - 1) * (NITEM2 + 32);
            const unsigned skip = ((gp - SELFG) & 7u) * 256;
#define PAIR_OF(T) ((T) < 1792u ? (SELFG * 256 + (T) + ((T) >= skip ? 256u : 0u)) & 2047u \
                                : gp * 256 + (((T) - 1792u) / BLK_ITEMS) * 64 + 8 * DPUSH + ((T) - 1792u) % BLK_ITEMS)
#define LOAD_H(H, PR) _Pragma("unroll") for (int u = 0; u < 2; ++u) \
            o[2 * (H) + u] = __builtin_amdgcn_raw_buffer_load_b128(rp, vo + (H) * NN + u * 1024 + (PR) * 2 * NN, 0, AUX_SYS)
#define PUT_H(H) _Pragma("unroll") for (int u = 0; u < 2; ++u) \
            *(LAS v4u*)(smem + L_PSLOT + sl * 4096 + (2 * (H) + u) * 1024 + lane * 16) = \
                bad ? (v4u){0x7f7f7f7fu, 0x7f7f7f7fu, 0x7f7f7f7fu, 0x7f7f7f7fu} : o[2 * (H) + u]
            unsigned d;
            ATOMIC_RESULT(d, dv, 0);
            cur = d - base2;
            if (cur < NITEM2) {
                cur = PAIR_OF(cur);
                NEED_REGION(cur >> 8);
                STAMP(18);
                LOAD_H(0, cur);
                for (;;) {
                    dv = atomic_inc_async(draw2);
                    SLOT_WAIT();
                    VMCNT(1);
                    LOAD_H(1, cur);
                    asm volatile("" ::: "memory");
                    PUT_H(0);
                    ATOMIC_RESULT(d, dv, 0);
                    nxt = d - base2;
                    if (nxt >= NITEM2) break;
                    nxt = PAIR_OF(nxt);
                    NEED_REGION(nxt >> 8);
                    LOAD_H(0, nxt);
                    asm volatile("" ::: "memory");
                    PUT_H(1);
                    SLOT_TAG(cur);
                    cur = nxt;
                }
                PUT_H(1);
                SLOT_TAG(cur);
            }
            SLOT_WAIT(); SLOT_TAG(0xffffu);
        }
    } else {
        char* pwl = (char*)d_ws_all[lane & 7];
        char* pol = (char*)d_out_all[lane & 7];
        asm volatile("" : "+v"(pwl), "+v"(pol));
        BAR();
        GET_GROUP();
        const unsigned p = dest_of(rank, g);
#define LANE_PTR(V, L) ((char*)(((unsigned long long)(unsigned)__builtin_amdgcn_readlane((int)((unsigned long long)(V) >> 32), (L)) << 32) | \
                                (unsigned)__builtin_amdgcn_readlane((int)(unsigned long long)(V), (L))))
        char* const pws = LANE_PTR(pwl, p);
        const __amdgpu_buffer_rsrc_t rs = mk_rsrc(pws + STAGE_OFF + (size_t)rank * SRC_BYTES, SRC_BYTES);
        const int r16 = lane & 15, q = lane >> 4;
        v4u o[4];
        unsigned n = 0, sl = 0;
        const bool wide = (WIN1 == 2) || (WIN1 == 15 && (gi & 1));
#define SLOT_GET() do { _Pragma("unroll") for (int f = 0; f < 4; ++f) o[f] = *(const LAS v4u*)(smem + L_PSLOT + sl * 4096 + f * 1024 + lane * 16); \
            LGKM0(); PIN_O(); ++n; lds_st(lbase + L_PTAG + 12, n); sl = sl + 1 == NPS ? 0 : sl + 1; } while (0)
#define ST1(F, OFF) __builtin_amdgcn_raw_buffer_store_b128(o[F], rs, lane * 16 + (F) * 1024 + (OFF), 0, AUX_SYS)
#define SIGNAL(OLD, KB) do { if ((OLD) + 1 == 40u * call) \
            __hip_atomic_store((gu32*)(pws + AFLAG + lane * AFS) + ((KB) * 8 + rank), call, RLX_SYS); } while (0)
#define COMPLETE_NOW() do { if (pkb < 16u) { VMCNT(0); unsigned cv_ = atomic_inc_masked_async(ws + CNT1 + (g * 17 + pkb) * 128, pmask, lane * 4); \
            ATOMIC_WAIT(cv_, 0); SIGNAL(cv_, pkb); pkb = 16; } } while (0)
        unsigned pmask = 1;
        unsigned pkb = 16;
        for (;;) {
            unsigned tg = rfl(lds_ld(lbase + L_PTAG + sl * 4));
            if ((tg >> 16) != n + 1) {
                COMPLETE_NOW();
                do { __builtin_amdgcn_s_sleep(1); tg = rfl(lds_ld(lbase + L_PTAG + sl * 4)); } while ((tg >> 16) != n + 1);
            }
            const unsigned t = tg & 0xffffu;
            if (t == 0xffffu) { ++n; lds_st(lbase + L_PTAG + 12, n); sl = sl + 1 == NPS ? 0 : sl + 1; break; }
            SLOT_GET();
            const unsigned off = t * 4096;
            const char* const cp = ws + CNT1 + (g * 17 + pkb) * 128;
            unsigned cv;
            if (WIN1 == 3) {
                VMCNT(2); ST1(0, off); VMCNT(2); ST1(1, off); VMCNT(2); cv = atomic_inc_masked_async(cp, pmask, lane * 4);
                ST1(2, off); VMCNT(3); ST1(3, off);
                ATOMIC_WAIT(cv, 2);
            } else if (wide) {
                VMCNT(1); ST1(0, off); VMCNT(1); cv = atomic_inc_masked_async(cp, pmask, lane * 4);
                ST1(1, off); VMCNT(2); ST1(2, off); VMCNT(1); ST1(3, off);
                ATOMIC_WAIT(cv, 2);
            } else {
                VMCNT(0); cv = atomic_inc_masked_async(cp, pmask, lane * 4);
                ST1(0, off); VMCNT(0); ST1(1, off); VMCNT(0); ST1(2, off); VMCNT(0); ST1(3, off);
                ATOMIC_WAIT(cv, 1);
            }
            if (pkb < 16u) SIGNAL(cv, pkb);
            pkb = t / 96;
            const unsigned z = t - pkb * 96;
            pmask = z < 64u ? 1u << (z >> 3) : 0xffu;
        }
        COMPLETE_NOW();
        STAMP(2);
        BAR();
        BAR();
        BAR();
        STAMP(6);
        if (p != (unsigned)rank) {
            const __amdgpu_buffer_rsrc_t rd = mk_rsrc(LANE_PTR(pol, p), OUT_BYTES);
            for (int it = 0; it < DPUSH; ++it) {
#pragma unroll
                for (int u = 0; u < 4; ++u) {
                    const int row = it * 16 + u * 4 + q;
                    o[u] = *(const LAS v4u*)(smem + row * 256 + ((r16 ^ (row & 15)) * 16));
                }
                PIN_O();
#pragma unroll
                for (int u = 0; u < 4; ++u) {
                    WAITW2();
                    __builtin_amdgcn_raw_buffer_store_b128(o[u], rd, (u * 4 + q) * NN + r16 * 16 + (tm * 128 + it * 16) * NN + rank * NS + tn * 256, 0, AUX_SYS);
                }
            }
            lds_st(lbase + L_PCALL + 8, 1u);
            VMCNT(0);
            if (lane == 0) __hip_atomic_store((gu32*)(pws + FIN + rank * 128) + gi, call, RLX_SYS);
            const __amdgpu_buffer_rsrc_t rl = mk_rsrc(LANE_PTR(pol, rank), OUT_BYTES);
            const unsigned vo = lane * 16 + p * NS;
            for (;;) {
                unsigned tg = rfl(lds_ld(lbase + L_PTAG + sl * 4));
                while ((tg >> 16) != n + 1) { __builtin_amdgcn_s_sleep(1); tg = rfl(lds_ld(lbase + L_PTAG + sl * 4)); }
                const unsigned t = tg & 0xffffu;
                if (t == 0xffffu) break;
                SLOT_GET();
#pragma unroll
                for (int u = 0; u < 4; ++u)
                    __builtin_amdgcn_raw_buffer_store_b128(o[u], rl, vo + (u >> 1) * NN + (u & 1) * 1024 + t * 2 * NN, 0, AUX_LOC);
            }
            STAMP(8);
        } else if (gi == 0) {
            const unsigned long long t0 = rt();
            unsigned spins = 0;
            const __amdgpu_buffer_rsrc_t rw = mk_rsrc(ws + FIN, 1024);
            for (;;) {
                const v4u f = __builtin_amdgcn_raw_buffer_load_b128(rw, lane * 16, 0, AUX_SYS);
                if (__all((lane >> 3) == rank || (f[0] >= call && f[1] >= call && f[2] >= call && f[3] >= call))) break;
                __builtin_amdgcn_s_sleep(2);
                if ((++spins & 63u) == 0u && spin_expired(tmo, call, t0)) {
                    if (lane == 0) ((unsigned char*)d_out_all[rank])[(size_t)tm * 128 * NN + rank * NS + tn * 256] = 0x7f;
                    break;
                }
                asm volatile("" ::: "memory");
            }
            __builtin_amdgcn_fence(__ATOMIC_ACQUIRE, "");
            STAMP(9);
        }
    }
}

extern "C" void kernel_launch(int rank, int nranks, ncclComm_t comm, hipStream_t stream,
                              void* const* d_in, const int* in_sizes, int n_in,
                              void* d_out, int out_size,
                              const int* in_dtypes, int out_dtype,
                              void* const* d_out_all,
                              void* const* d_ws_all, size_t ws_bytes) {
    (void)comm; (void)d_out; (void)in_dtypes; (void)out_dtype;
    if (nranks != NR || (unsigned)rank >= (unsigned)NR || n_in < 2 || ws_bytes < WS_NEED ||
        in_sizes[0] != MM * KS || in_sizes[1] != KS * NN || out_size != MM * NN) {
        fprintf(stderr, "kernel_launch: unexpected problem shape\n");
        return;
    }
    hipLaunchKernelGGL(gemm_ar_q8, dim3(NWG), dim3(NTHREADS), 0, stream,
                       (const float*)d_in[0], (const float*)d_in[1], d_out_all, d_ws_all, rank);
    HIP_CHECK(hipGetLastError());
}
```

```cpp
#include <hip/hip_runtime.h>
#include <rccl/rccl.h>
#include <cstdio>
#include <cstdint>

#define HIP_CHECK(expr) do { hipError_t _e = (expr); if (_e != hipSuccess) { \
    fprintf(stderr, "HIP error %s at %s:%d\n", hipGetErrorString(_e), __FILE__, __LINE__); \
    return; } } while (0)

#define GAS __attribute__((address_space(1)))
#define LAS __attribute__((address_space(3)))
typedef unsigned v4u __attribute__((ext_vector_type(4)));
typedef unsigned v2u __attribute__((ext_vector_type(2)));
typedef float f32x4 __attribute__((ext_vector_type(4)));
typedef short bf16x8 __attribute__((ext_vector_type(8)));
typedef __bf16 bf2 __attribute__((ext_vector_type(2)));
typedef GAS unsigned gu32;
typedef GAS unsigned long long gu64;

namespace {

constexpr int NR = 8;
constexpr int MM = 4096, NN = 16384;
constexpr int KS = 512;
constexpr int NS = 2048;
constexpr int KBS = 16;
constexpr unsigned A_BYTES = 256u * 1024u;
constexpr unsigned B_BYTES = 128u * 1024u;
constexpr unsigned BLK_BYTES = A_BYTES + B_BYTES;
constexpr unsigned SRC_BYTES = KBS * BLK_BYTES;
constexpr size_t STAGE_OFF = 1u << 20;
constexpr unsigned EXPW = NR * SRC_BYTES;
constexpr unsigned DATA_BYTES = EXPW + NR * KBS * B_BYTES;
constexpr size_t WS_NEED = STAGE_OFF + DATA_BYTES;
constexpr unsigned OUT_BYTES = (unsigned)MM * NN;

constexpr unsigned DRAW1 = 0x0000;
constexpr unsigned DRAW2 = 0x0400;
constexpr unsigned TMO   = 0x0800;
constexpr unsigned EXPF  = 0x1000;
constexpr unsigned AFLAG = 0x2000;
constexpr unsigned GMAX  = 0x5000;
constexpr unsigned RMAX  = 0x6000;
constexpr unsigned RCNT  = 0x8000;
constexpr unsigned RDY   = 0x8400;
constexpr unsigned FIN   = 0x9000;
constexpr unsigned CNT1  = 0x28000;
constexpr unsigned ECNT  = 0x30000;
constexpr unsigned PROF  = 0x10000;
constexpr unsigned PRIV  = 0x20000;
constexpr unsigned CLAIM = 0x24000;

constexpr int NWG = 256, NTHREADS = 896;
constexpr int NSLOT = 6;
constexpr unsigned SLOT_BYTES = 24576;
constexpr unsigned L_FULL = NSLOT * SLOT_BYTES;
constexpr unsigned L_FREE = L_FULL + 16;
constexpr unsigned L_WMAX = L_FREE + 32;
constexpr unsigned L_MISC = L_WMAX + 32;
constexpr unsigned L_PTAG = L_MISC + 16;
constexpr unsigned L_PCALL = L_PTAG + 16;
constexpr unsigned L_SCHED = L_PCALL + 16;
constexpr unsigned L_PSLOT = L_SCHED + 512;
constexpr unsigned NPS = 3;
constexpr unsigned LDS_BYTES = L_PSLOT + NPS * 4096;
static_assert(LDS_BYTES <= 163840 && L_PSLOT % 16 == 0 && L_MISC % 16 == 0, "LDS layout");
constexpr int NSTEP = NR * KBS;
constexpr unsigned long long DEADLINE = 100000000ull;

#define DPUSH 2
#define FDEPTH 2
#define NPULL1 22
#define NPULL2 28
#define ROT (p + rank)
#define GATE1 0
#define FENCE0 0
#define PW1 2
#define PW2 2
#define CONV_T0 400u
#define CONV_TAU 384u
#define POLL_SLOW 127
#define POLL_MISS 16
#define POLL_MID 32
#define FASTCNT 112u
#define FASTCNT2 124u
constexpr int SELFG = 3;
constexpr unsigned NITEM1 = KBS * 96;
constexpr unsigned NCONV = 14 * 40;
constexpr unsigned BLK_ITEMS = 64 - 8 * DPUSH;
constexpr unsigned NITEM2 = 7 * 256 + 4 * BLK_ITEMS;

__device__ __forceinline__ unsigned dest_of(int rank, int g) {
    const unsigned set = (0xB2u >> rank) & 1u;
    const unsigned pos = (0x33212100u >> (4 * rank)) & 3u;
    const unsigned members = ((g < 4) == (set == 0)) ? 0x6320u : 0x7541u;
    return (members >> (4 * ((pos + 1 + g) & 3))) & 7u;
}

#define VMCNT_(n) asm volatile("s_waitcnt vmcnt(" #n ")" ::: "memory")
#define VMCNT(n) VMCNT_(n)
#define LGKM0() asm volatile("s_waitcnt lgkmcnt(0)" ::: "memory")
#define BAR() do { asm volatile("s_waitcnt lgkmcnt(0)" ::: "memory"); __builtin_amdgcn_s_barrier(); asm volatile("" ::: "memory"); } while (0)
#define RLX_AGENT __ATOMIC_RELAXED, __HIP_MEMORY_SCOPE_AGENT
#define RLX_SYS __ATOMIC_RELAXED, __HIP_MEMORY_SCOPE_SYSTEM

__device__ __forceinline__ unsigned long long rt() { return __builtin_amdgcn_s_memrealtime(); }
__device__ __forceinline__ unsigned rfl(unsigned v) { return (unsigned)__builtin_amdgcn_readfirstlane((int)v); }
__device__ __forceinline__ unsigned pk_bf16(float lo, float hi) {
    bf2 v; v.x = (__bf16)lo; v.y = (__bf16)hi; return __builtin_bit_cast(unsigned, v);
}
__device__ __forceinline__ unsigned wave_umax(unsigned v) {
    v = max(v, (unsigned)__builtin_amdgcn_update_dpp(0, (int)v, 0x128, 0xf, 0xf, false));
    v = max(v, (unsigned)__builtin_amdgcn_update_dpp(0, (int)v, 0x124, 0xf, 0xf, false));
    v = max(v, (unsigned)__builtin_amdgcn_update_dpp(0, (int)v, 0x122, 0xf, 0xf, false));
    v = max(v, (unsigned)__builtin_amdgcn_update_dpp(0, (int)v, 0x121, 0xf, 0xf, false));
    const unsigned a = (unsigned)__builtin_amdgcn_readlane((int)v, 0), b = (unsigned)__builtin_amdgcn_readlane((int)v, 16);
    const unsigned c = (unsigned)__builtin_amdgcn_readlane((int)v, 32), d = (unsigned)__builtin_amdgcn_readlane((int)v, 48);
    return max(max(a, b), max(c, d));
}
__device__ __forceinline__ char* lane_ptr(const void* v, unsigned l) {
    const unsigned long long u = (unsigned long long)v;
    return (char*)(((unsigned long long)(unsigned)__builtin_amdgcn_readlane((int)(u >> 32), (int)l) << 32) |
                   (unsigned)__builtin_amdgcn_readlane((int)u, (int)l));
}
__device__ __forceinline__ void lds_st(unsigned addr, unsigned v) { asm volatile("ds_write_b32 %0, %1" :: "v"(addr), "v"(v) : "memory"); }
__device__ __forceinline__ unsigned lds_ld(unsigned addr) {
    unsigned v; asm volatile("ds_read_b32 %0, %1\n\ts_waitcnt lgkmcnt(0)" : "=&v"(v) : "v"(addr) : "memory"); return v;
}
__device__ __forceinline__ void glds4(unsigned voff, const char* sbase, unsigned m0v) {
    unsigned keep;
    asm volatile("s_mov_b32 %0, m0\n\ts_mov_b32 m0, %3\n\ts_nop 4\n\t"
                 "global_load_lds_dwordx4 %1, %2 sc0 sc1\n\t"
                 "global_load_lds_dwordx4 %1, %2 offset:1024 sc0 sc1\n\t"
                 "global_load_lds_dwordx4 %1, %2 offset:2048 sc0 sc1\n\t"
                 "global_load_lds_dwordx4 %1, %2 offset:3072 sc0 sc1\n\t"
                 "s_mov_b32 m0, %0"
                 : "=&s"(keep) : "v"(voff), "s"(sbase), "s"(m0v) : "memory");
}
template <int IMM> __device__ __forceinline__ void glds1(unsigned voff, const char* sbase, unsigned m0v) {
    unsigned keep;
    asm volatile("s_mov_b32 %0, m0\n\ts_mov_b32 m0, %3\n\ts_nop 4\n\t"
                 "global_load_lds_dwordx4 %1, %2 offset:%4 sc0 sc1\n\t"
                 "s_mov_b32 m0, %0"
                 : "=&s"(keep) : "v"(voff), "s"(sbase), "s"(m0v), "i"(IMM) : "memory");
}
__device__ __forceinline__ __amdgpu_buffer_rsrc_t mk_rsrc(void* base, unsigned bytes) {
    return __builtin_amdgcn_make_buffer_rsrc(base, 0, (int)bytes, 0x00020000);
}
__device__ __forceinline__ unsigned atomic_inc_async(const void* p) {
    unsigned old; unsigned long long sv;
    asm volatile("s_mov_b64 %1, exec\n\ts_mov_b64 exec, 1\n\ts_nop 3\n\t"
                 "global_atomic_add %0, %2, %3, %4 sc0\n\t"
                 "s_mov_b64 exec, %1"
                 : "=&v"(old), "=&s"(sv) : "v"(0u), "v"(1u), "s"(p) : "memory");
    return old;
}
#define ATOMIC_RESULT(dst, old, n) ATOMIC_RESULT_(dst, old, n)
#define ATOMIC_RESULT_(dst, old, n) asm volatile("s_waitcnt vmcnt(" #n ")\n\tv_readfirstlane_b32 %0, %1\n\ts_nop 4" : "=s"(dst) : "v"(old) : "memory")
__device__ __forceinline__ unsigned atomic_inc_masked_async(const void* p, unsigned mask, unsigned lane4) {
    unsigned old = 0; unsigned long long sv;
    asm volatile("s_mov_b64 %1, exec\n\ts_mov_b32 exec_lo, %5\n\ts_mov_b32 exec_hi, 0\n\ts_nop 2\n\t"
                 "global_atomic_add %0, %2, %3, %4 sc0\n\t"
                 "s_mov_b64 exec, %1"
                 : "+v"(old), "=&s"(sv) : "v"(lane4), "v"(1u), "s"(p), "s"(rfl(mask)) : "memory");
    return old;
}
#define ATOMIC_WAIT(old, n) asm volatile("s_waitcnt vmcnt(" #n ")" : "+v"(old) :: "memory")
constexpr int AUX_SYS = 17;
constexpr int AUX_DEV = 16;

__device__ __forceinline__ bool spin_expired(gu32* tmo, unsigned call, unsigned long long t0) {
    if (rfl(__hip_atomic_load(tmo, RLX_AGENT)) == call || rt() - t0 > DEADLINE) {
        __hip_atomic_store(tmo, call, RLX_AGENT);
        return true;
    }
    return false;
}

__device__ __forceinline__ void load_item(f32x4 (&buf)[8], const float* x, const float* w, unsigned kb, unsigned z, int lane) {
    const int r16 = lane & 15, q = lane >> 4;
    if (z < 64u) {
        const float* a = x + (size_t)(z * 64 + r16) * KS + kb * 32 + 4 * q;
#pragma unroll
        for (int e = 0; e < 8; ++e) buf[e] = *(const f32x4*)(a + (e >> 1) * 16 * KS + (e & 1) * 16);
    } else {
        const float* b = w + (size_t)(kb * 32 + 4 * q) * NN + (z - 64u) * 64 + 4 * r16;
#pragma unroll
        for (int e = 0; e < 8; ++e) buf[e] = *(const f32x4*)(b + (size_t)((e >> 2) * 16 + (e & 3)) * NN);
    }
}
__device__ __forceinline__ void cvt_item(v4u (&o)[4], const f32x4 (&buf)[8], bool is_x) {
    if (is_x) {
#pragma unroll
        for (int f = 0; f < 4; ++f) {
            o[f][0] = pk_bf16(buf[2*f][0], buf[2*f][1]); o[f][1] = pk_bf16(buf[2*f][2], buf[2*f][3]);
            o[f][2] = pk_bf16(buf[2*f+1][0], buf[2*f+1][1]); o[f][3] = pk_bf16(buf[2*f+1][2], buf[2*f+1][3]);
        }
    } else {
#pragma unroll
        for (int j = 0; j < 4; ++j) {
            o[j][0] = pk_bf16(buf[0][j], buf[1][j]); o[j][1] = pk_bf16(buf[2][j], buf[3][j]);
            o[j][2] = pk_bf16(buf[4][j], buf[5][j]); o[j][3] = pk_bf16(buf[6][j], buf[7][j]);
        }
    }
}
__device__ __forceinline__ unsigned item_of(unsigned t, unsigned src) {
    if (t < 128u) return t < 64u ? (t + src * 8) & 63u : (t >= 96u ? 96u : 0u) + 64u + (t & 31u);
    const unsigned k = (t - 128u) / 96u, u = t - 128u - k * 96u, j = u / 12u, v = u - j * 12u;
    if (k == 14u) return 15u * 96u + ((u + src * 8) & 63u);
    if (v < 8u) return (k + 1u) * 96u + ((j + src) & 7u) * 8u + v;
    return (k + 2u) * 96u + 64u + j * 4u + v - 8u;
}

#define STAMP(idx) do {} while (0)

}

__global__ void __launch_bounds__(NTHREADS, 4)
gemm_ar_q8(const float* __restrict__ x, const float* __restrict__ w, void* const* __restrict__ d_out_all,
           void* const* __restrict__ d_ws_all, int rank) {
    __shared__ __attribute__((aligned(1024))) unsigned char smem[LDS_BYTES];
    const unsigned lbase = (unsigned)(size_t)(LAS unsigned char*)smem;
    const int lane = threadIdx.x & 63;
    const int wave = (int)rfl(threadIdx.x >> 6);
    const int gi = blockIdx.x >> 3;
    char* const ws = (char*)d_ws_all[rank];
    gu32* const tmo = (gu32*)(ws + TMO);
    const int tn = gi & 7;
#define GET_GROUP() const v2u gc_ = *(volatile LAS v2u*)(smem + L_PCALL); \
    const unsigned call = rfl(gc_[0]); const int g = (int)(rfl(gc_[1]) & 7u); const int tm = g * 4 + (gi >> 3); (void)call; (void)tm
#define PIN_O() asm volatile("" : "+v"(o[0]), "+v"(o[1]), "+v"(o[2]), "+v"(o[3]) :: "memory")

#define CONVERT(C) do { \
        f32x4 buf[8]; v4u o[4]; \
        const unsigned d = dest_of(rank, g), kb = 2u + (C) / 40u, u = (C) + 80u - kb * 40u; \
        load_item(buf, x, w, kb, u < 32u ? 64u + d * 32 + u : g * 8 + u - 32u, lane); \
        cvt_item(o, buf, u >= 32u); \
        const unsigned off = u >= 32u ? (rank * KBS + kb) * BLK_BYTES + (g * 8 + u - 32u) * 4096 \
                           : d == (unsigned)rank ? (rank * KBS + kb) * BLK_BYTES + A_BYTES + u * 4096 \
                           : EXPW + ((d * KBS + kb) * 32 + u) * 4096; \
        const __amdgpu_buffer_rsrc_t rs = mk_rsrc(ws + STAGE_OFF, DATA_BYTES); \
        _Pragma("unroll") for (int f = 0; f < 4; ++f) __builtin_amdgcn_raw_buffer_store_b128(o[f], rs, off + lane * 16 + f * 1024, 0, AUX_SYS); \
        VMCNT(0); \
        const unsigned cv = atomic_inc_async(ws + ECNT + (g * 16 + kb) * 128); \
        unsigned old; \
        ATOMIC_RESULT(old, cv, 0); \
        if (old + 1 == 40u * call && lane < 8) __hip_atomic_store((gu32*)(pw + EXPF + rank * 512) + (kb * 8 + g), call, RLX_SYS); \
    } while (0)

    if (wave == 13) {
        char* pw = (char*)d_ws_all[lane & 7];
        BAR();
        GET_GROUP();
        const unsigned long long t0 = rt();
        for (unsigned j = 0; j * 32 + gi < NCONV; j += j == 0 ? 2 : 1) {
            if (j > 0) while (rt() - t0 < CONV_T0 + (j - 2) * CONV_TAU + gi * (CONV_TAU / 32)) __builtin_amdgcn_s_sleep(4);
            CONVERT(j * 32 + gi);
        }
        STAMP(1);
        BAR();
        BAR();
        BAR();
    } else if (wave < 8) {
        char* pw = (char*)d_ws_all[lane & 7];
        {
            f32x4 buf[8];
            v4u o[4];
            unsigned xcc;
            asm volatile("s_getreg_b32 %0, hwreg(HW_REG_XCC_ID)" : "=s"(xcc));
            const unsigned kb = (wave >> 1) & 1, z = (wave & 1) * 32 + gi;
            const bool ahead = wave < 4 && (xcc & 7u) == (unsigned)SELFG;
            if (ahead) load_item(buf, x, w, kb, z, lane);
            BAR();
            const v2u gc = *(volatile LAS v2u*)(smem + L_PCALL);
            if (wave < 4 && (rfl(gc[1]) & 7u) == (unsigned)SELFG) {
                const unsigned call = rfl(gc[0]);
                if (!ahead) load_item(buf, x, w, kb, z, lane);
                cvt_item(o, buf, true);
                const __amdgpu_buffer_rsrc_t rs = mk_rsrc(ws + STAGE_OFF, DATA_BYTES);
#pragma unroll
                for (int f = 0; f < 4; ++f)
                    __builtin_amdgcn_raw_buffer_store_b128(o[f], rs, (rank * KBS + kb) * BLK_BYTES + z * 4096 + lane * 16 + f * 1024, 0, AUX_SYS);
                VMCNT(0);
                const unsigned cv = atomic_inc_async(ws + ECNT + (SELFG * 16 + kb) * 128);
                unsigned old;
                ATOMIC_RESULT(old, cv, 0);
                if (old + 1 == 64u * call) __hip_atomic_store((gu32*)(pw + EXPF + rank * 512) + (kb * 8 + (lane >> 3)), call, RLX_SYS);
            }
        }
        GET_GROUP();
        if (wave == 4) CONVERT(32u + gi);
        const int wm = wave >> 2, wn = wave & 3;
        f32x4 acc[4][4];
#pragma unroll
        for (int i = 0; i < 4; ++i)
#pragma unroll
            for (int j = 0; j < 4; ++j) acc[i][j] = (f32x4){0.f, 0.f, 0.f, 0.f};
        const LAS unsigned char* abase = (const LAS unsigned char*)smem + wm * 4096 + lane * 16;
        const LAS unsigned char* bbase = (const LAS unsigned char*)smem + 8192 + wn * 4096 + lane * 16;
        volatile LAS v2u* full = (volatile LAS v2u*)(smem + L_FULL);
        unsigned slot_off = 0;
        for (unsigned s = 0; s < (unsigned)NSTEP; ++s) {
            for (;;) {
                const v2u f = *full;
                if (rfl(f.x) > s && rfl(f.y) > s) break;
                __builtin_amdgcn_s_sleep(1);
            }
            asm volatile("" ::: "memory");
            v4u a[4], b[4];
#pragma unroll
            for (int i = 0; i < 4; ++i) b[i] = *(const LAS v4u*)(bbase + slot_off + i * 1024);
#pragma unroll
            for (int i = 0; i < 4; ++i) a[i] = *(const LAS v4u*)(abase + slot_off + i * 1024);
            LGKM0();
            lds_st(lbase + L_FREE + wave * 4, s + 1);
            asm volatile("" : "+v"(a[0]), "+v"(a[1]), "+v"(a[2]), "+v"(a[3]), "+v"(b[0]), "+v"(b[1]), "+v"(b[2]), "+v"(b[3]) :: "memory");
#pragma unroll
            for (int i = 0; i < 4; ++i)
#pragma unroll
                for (int j = 0; j < 4; ++j)
                    acc[i][j] = __builtin_amdgcn_mfma_f32_16x16x32_bf16(__builtin_bit_cast(bf16x8, b[j]), __builtin_bit_cast(bf16x8, a[i]), acc[i][j], 0, 0, 0);
            slot_off += SLOT_BYTES;
            if (slot_off == NSLOT * SLOT_BYTES) slot_off = 0;
        }
        if (wave == 0) STAMP(4);
        float mx = 0.f;
#pragma unroll
        for (int i = 0; i < 4; ++i)
#pragma unroll
            for (int j = 0; j < 4; ++j)
#pragma unroll
                for (int r = 0; r < 4; ++r) mx = fmaxf(mx, fabsf(acc[i][j][r]));
        lds_st(lbase + L_WMAX + wave * 4, wave_umax(__builtin_bit_cast(unsigned, mx)));
        BAR();
        BAR();
        const v4u misc = *(volatile LAS v4u*)(smem + L_MISC);
        const float inv = __builtin_bit_cast(float, misc[0]);
        const unsigned bad = misc[2];
        {
            const int r16 = lane & 15, q = lane >> 4;
#pragma unroll
            for (int i = 0; i < 4; ++i) {
                v4u o;
#pragma unroll
                for (int r = 0; r < 4; ++r) {
                    int d = __builtin_amdgcn_cvt_pk_fp8_f32(acc[i][0][r] * inv, acc[i][1][r] * inv, 0, false);
                    d = __builtin_amdgcn_cvt_pk_fp8_f32(acc[i][2][r] * inv, acc[i][3][r] * inv, d, true);
                    o[r] = bad ? 0x7f7f7f7fu : (unsigned)d;
                }
                const int row = wm * 64 + i * 16 + r16;
                const int ch = (wn * 4 + q) ^ r16;
                *(LAS v4u*)(smem + row * 256 + ch * 16) = o;
            }
        }
        BAR();
        if (g != SELFG) while (rfl(lds_ld(lbase + L_PCALL + 8)) == 0u) __builtin_amdgcn_s_sleep(4);
        {
            const __amdgpu_buffer_rsrc_t ro = mk_rsrc(d_out_all[rank], OUT_BYTES);
#pragma unroll
            for (int u = 0; u < 4; ++u) {
                const int row = wave * 16 + u * 4 + (lane >> 4), c = lane & 15;
                const v4u v = *(const LAS v4u*)(smem + row * 256 + ((c ^ (row & 15)) * 16));
                __builtin_amdgcn_raw_buffer_store_b128(v, ro, (tm * 128 + row) * NN + rank * NS + tn * 256 + c * 16, 0, AUX_SYS);
            }
            VMCNT(0);
            unsigned old = 0;
            if (lane == 0) old = __hip_atomic_fetch_add((LAS unsigned*)(smem + L_MISC + 12), 1u, __ATOMIC_RELAXED, __HIP_MEMORY_SCOPE_WORKGROUP);
            if (rfl(old) == 7) {
                const unsigned cv = atomic_inc_async(ws + RCNT + g * 128);
                unsigned oc;
                ATOMIC_RESULT(oc, cv, 0);
                if (oc + 1 == 32u * call && lane < 8) __hip_atomic_store((gu32*)(pw + RDY + rank * 128) + g, call, RLX_SYS);
                STAMP(7);
            }
        }
    } else if (wave < 10) {
        const int lw = wave - 8;
        BAR();
        GET_GROUP();
        const char* stage = ws + STAGE_OFF;
        const unsigned voff = lane * 16;
        unsigned slot_off = 0, pub = 0, s = 0;
        const unsigned a_full = lbase + L_FULL + lw * 4;
        while (pub < (unsigned)NSTEP) {
            bool go = false;
            unsigned e = 0;
            if (s < (unsigned)NSTEP && s - pub < 5u) {
                const unsigned v = lds_ld(lane < 8 ? lbase + L_FREE + lane * 4 : lbase + L_SCHED + s * 4);
                e = (unsigned)__builtin_amdgcn_readlane((int)v, 8);
                go = e != 0xffffffffu && __all(lane >= 8 || v + NSLOT > s);
            }
            if (go) {
                const char* blk = stage + (size_t)(e & 127u) * BLK_BYTES;
                const unsigned dst = lbase + slot_off;
                if (lw == 0) {
                    glds4(voff, blk + tm * 8192, dst);
                    glds4(voff, blk + tm * 8192 + 4096, dst + 4096);
                    glds4(voff, blk + A_BYTES + tn * 16384, dst + 8192);
                } else {
                    glds4(voff, blk + A_BYTES + tn * 16384 + 4096, dst + 12288);
                    glds4(voff, blk + A_BYTES + tn * 16384 + 8192, dst + 16384);
                    glds4(voff, blk + A_BYTES + tn * 16384 + 12288, dst + 20480);
                }
                ++s;
                slot_off += SLOT_BYTES;
                if (slot_off == NSLOT * SLOT_BYTES) slot_off = 0;
            } else if (pub < s) {
                const unsigned infl = s - pub;
                if (infl == 1) VMCNT(0); else if (infl == 2) VMCNT(12); else if (infl == 3) VMCNT(24);
                else if (infl == 4) VMCNT(36); else VMCNT(48);
                ++pub;
                lds_st(a_full, pub);
            } else {
                __builtin_amdgcn_s_sleep(1);
            }
        }
        BAR();
        BAR();
        BAR();
    } else if (wave == 10) {
        unsigned xcc;
        asm volatile("s_getreg_b32 %0, hwreg(HW_REG_XCC_ID)" : "=s"(xcc));
        int g = (int)(xcc & 7u);
        unsigned tk = 0;
        if (lane < 2) tk = __hip_atomic_fetch_add((gu32*)(ws + (lane == 0 ? PRIV + blockIdx.x * 64 : CLAIM + (gi * 8 + g) * 64)), 1u, RLX_AGENT);
        lds_st(lbase + L_SCHED + lane * 4, 0xffffffffu);
        lds_st(lbase + L_SCHED + 256 + lane * 4, 0xffffffffu);
        if (lane < 28) lds_st(lbase + L_FULL + lane * 4, 0u);
        const unsigned call = rfl(tk) + 1;
        unsigned got = (unsigned)__builtin_amdgcn_readlane((int)tk, 1) + 1;
        for (int tries = 0; got != call && tries < 7; ++tries) {
            unsigned o2 = 0;
            if (lane == 0) {
                __hip_atomic_fetch_add((gu32*)(ws + CLAIM + (gi * 8 + g) * 64), 0xffffffffu, RLX_AGENT);
                o2 = __hip_atomic_fetch_add((gu32*)(ws + CLAIM + (gi * 8 + ((g + 1) & 7)) * 64), 1u, RLX_AGENT);
            }
            g = (g + 1) & 7;
            got = rfl(o2) + 1;
        }
        if (lane < 4) lds_st(lbase + L_PCALL + lane * 4, lane == 0 ? call : lane == 1 ? (unsigned)g : 0u);
        BAR();
        bool bad = false;
        {
            const bool own = (lane & 7) == rank;
            gu32* fa = own ? (gu32*)(ws + EXPF + rank * 512) + ((lane & 56) + g) : (gu32*)(ws + AFLAG + g * 1024) + lane;
            gu32* fb = own ? (gu32*)(ws + EXPF + rank * 512) + ((lane & 56) + SELFG) : fa;
            gu32* fc = lane < 16 ? (gu32*)(ws + AFLAG + g * 1024) + (128 + (lane & 7)) : fb;
            unsigned long long done0 = 0, done1 = 0;
            unsigned cnt = 0, spins = 0;
            const unsigned ent = (lane & 7) * KBS + (lane >> 3);
            const unsigned long long t0 = rt();
            while ((done0 & done1) != ~0ull) {
                const bool h = done0 == ~0ull;
                const unsigned a0 = __hip_atomic_load(h ? fa + 64 : fa, RLX_SYS), a1 = __hip_atomic_load(fa + 64, RLX_SYS);
                const unsigned b0 = __hip_atomic_load(h ? fb + 64 : fc, RLX_SYS), b1 = __hip_atomic_load(fb + 64, RLX_SYS);
                const unsigned long long n0 = __ballot(bad || (a0 >= call && b0 >= call)) & ~done0;
                const unsigned long long n1 = __ballot(bad || (a1 >= call && b1 >= call)) & ~done1;
                if (n0 | n1) {
                    const unsigned long long lt = (1ull << lane) - 1;
                    const unsigned c0 = __popcll(n0), c1 = __popcll(n1);
                    if ((n0 >> lane) & 1) lds_st(lbase + L_SCHED + (cnt + __popcll(n0 & lt)) * 4, ent);
                    if ((n1 >> lane) & 1) lds_st(lbase + L_SCHED + (cnt + c0 + __popcll(n1 & lt)) * 4, ent + 8);
                    cnt += c0 + c1;
                    done0 |= n0; done1 |= n1;
                    if (cnt < FASTCNT) __builtin_amdgcn_s_sleep(POLL_SLOW);
                    else if (cnt < FASTCNT2) __builtin_amdgcn_s_sleep(POLL_MID);
                } else {
                    if (cnt < FASTCNT) __builtin_amdgcn_s_sleep(POLL_MISS);
                    else if (cnt < FASTCNT2) __builtin_amdgcn_s_sleep(POLL_MID);
                    else __builtin_amdgcn_s_sleep(1);
                    if ((++spins & 63u) == 0u && spin_expired(tmo, call, t0)) bad = true;
                }
            }
        }
        STAMP(3);
        char* pw = (char*)d_ws_all[lane >> 3];
        asm volatile("" : "+v"(pw));
        BAR();
        {
            unsigned m = wave_umax(lds_ld(lbase + L_WMAX + (lane & 7) * 4));
            if (lane == 0) __hip_atomic_store((gu64*)(ws + GMAX + g * 256) + gi, ((unsigned long long)call << 32) | m, RLX_AGENT);
            const unsigned long long t0 = rt();
            unsigned spins = 0;
            if (gi == 0) {
                unsigned long long v;
                for (;;) {
                    v = __hip_atomic_load((gu64*)(ws + GMAX + g * 256) + (lane & 31), RLX_AGENT);
                    if (__all((unsigned)(v >> 32) == call)) break;
                    if ((++spins & 63u) == 0u && spin_expired(tmo, call, t0)) { bad = true; break; }
                }
                m = wave_umax((unsigned)v);
                __hip_atomic_store((gu64*)(pw + RMAX + (lane & 7) * 512) + (rank * 8 + g), ((unsigned long long)call << 32) | m, RLX_SYS);
            }
            unsigned long long v;
            for (;;) {
                v = __hip_atomic_load((gu64*)(ws + RMAX + g * 512) + lane, RLX_SYS);
                if (rfl(__hip_atomic_load(tmo, RLX_AGENT)) == call) bad = true;
                if (bad || __all((unsigned)(v >> 32) == call)) break;
                if ((++spins & 63u) == 0u && spin_expired(tmo, call, t0)) { bad = true; break; }
            }
            m = wave_umax((unsigned)v);
            const float inv = 448.0f / __builtin_bit_cast(float, m);
            if (lane < 3) lds_st(lbase + L_MISC + lane * 4, lane == 0 ? __builtin_bit_cast(unsigned, inv) : (bad ? 1u : 0u));
        }
        STAMP(5);
        BAR();
        BAR();
    } else if (wave == 11) {
        char* pwl = (char*)d_ws_all[lane & 7];
        char* pol = (char*)d_out_all[lane & 7];
        f32x4 buf[8];
        v4u o[4];
#define SLOT_TAG(T) do { lds_st(lbase + L_PTAG + sl * 4, ((n + 1) << 16) | (T)); ++n; sl = sl + 1 == NPS ? 0 : sl + 1; } while (0)
#define PUT_H(H) _Pragma("unroll") for (int u = 0; u < 2; ++u) \
            *(LAS v4u*)(smem + L_PSLOT + sl * 4096 + (2 * (H) + u) * 1024 + lane * 16) = o[2 * (H) + u]
        unsigned n = 0, sl = 0;
        bool bad = false;
        f32x4 buf2[8];
        unsigned xcc;
        asm volatile("s_getreg_b32 %0, hwreg(HW_REG_XCC_ID)" : "=s"(xcc));
        load_item(buf, x, w, 0u, 64u + dest_of(rank, xcc & 7u) * 32 + gi, lane);
        load_item(buf2, x, w, 1u, 64u + dest_of(rank, xcc & 7u) * 32 + gi, lane);
        BAR();
        GET_GROUP();
        const unsigned p = dest_of(rank, g);
        if ((unsigned)g != (xcc & 7u)) {
            load_item(buf, x, w, 0u, 64u + p * 32 + gi, lane);
            load_item(buf2, x, w, 1u, 64u + p * 32 + gi, lane);
        }
        cvt_item(o, buf, false); PUT_H(0); PUT_H(1);
        SLOT_TAG(64u + gi);
        cvt_item(o, buf2, false); PUT_H(0); PUT_H(1);
        SLOT_TAG(96u + 64u + gi);
        unsigned gp = 0;
        for (int j = 0; j < 8; ++j) if (dest_of((int)p, j) == (unsigned)rank) gp = j;
        unsigned long long rdy0 = 0, rdy1 = 0;
#define FLAG_SET(B) ((((B) < 64u ? rdy0 : rdy1) >> ((B) & 63u)) & 1u)
#define NEED_FLAG(FP, MASK, B) do { if (!FLAG_SET(B)) { \
                const unsigned long long t0 = rt(); unsigned spins = 0; \
                for (;;) { \
                    const unsigned f0 = __hip_atomic_load((gu32*)(FP) + (lane & (MASK)), RLX_SYS); \
                    const unsigned f1 = __hip_atomic_load((gu32*)(FP) + ((lane + 64) & (MASK)), RLX_SYS); \
                    rdy0 = __ballot(f0 >= call); rdy1 = __ballot(f1 >= call); \
                    if (FLAG_SET(B)) break; \
                    __builtin_amdgcn_s_sleep(2); \
                    if ((++spins & 63u) == 0u && spin_expired(tmo, call, t0)) { bad = true; rdy0 = rdy1 = ~0ull; break; } \
                } \
                asm volatile("" ::: "memory"); } } while (0)
#define PROG(C, H) do {} while (0)
#define STARTST() do {} while (0)
#define PEND_TAG() do { if (pend != 0xffffffffu) { lds_st(lbase + L_PTAG + psl * 4, (n << 16) | pend | (bad ? 0x8000u : 0u)); pend = 0xffffffffu; } } while (0)
#define PULL_LOOP(DRAWP, BASE, NITEM, SRC, HSTEP, PREP, FP, FMASK, W, ON) do { \
            unsigned d = 0, so, fbit, pend = 0xffffffffu, psl = 0, dv; \
            if (ON) { dv = atomic_inc_async(DRAWP); ATOMIC_RESULT(d, dv, 0); } \
            while (ON) { \
                unsigned cur = d - (BASE); \
                if (cur >= (NITEM)) break; \
                PROG(cur, HSTEP); \
                PREP(cur); \
                if (!FLAG_SET(fbit) || rfl(lds_ld(lbase + L_PTAG + 12)) + NPS <= n) { \
                    VMCNT(0); PEND_TAG(); \
                    NEED_FLAG(FP, FMASK, fbit); \
                    while (rfl(lds_ld(lbase + L_PTAG + 12)) + NPS <= n) __builtin_amdgcn_s_sleep(1); \
                } \
                STARTST(); \
                const char* sb = (SRC) + rfl(so); \
                const unsigned dst = lbase + L_PSLOT + sl * 4096; \
                if (W == 2) VMCNT(1); else if (W == 3) VMCNT(2); else VMCNT(0); \
                if (W == 1) PEND_TAG(); \
                glds1<0>(lane * 16, sb, dst); \
                dv = atomic_inc_async(DRAWP); \
                if (W == 2) { VMCNT(2); PEND_TAG(); } else if (W == 3) VMCNT(3); else VMCNT(1); \
                glds1<1024>(lane * 16, sb, dst); \
                if (W == 2) VMCNT(2); else if (W == 3) { VMCNT(3); PEND_TAG(); } else VMCNT(0); \
                glds1<2048>(lane * 16, sb + ((HSTEP) - 2048), dst); \
                if (W == 2) VMCNT(1); else if (W == 3) VMCNT(3); else VMCNT(0); \
                glds1<3072>(lane * 16, sb + ((HSTEP) - 2048), dst); \
                ATOMIC_RESULT(d, dv, 3); \
                pend = cur; psl = sl; ++n; sl = sl + 1 == NPS ? 0 : sl + 1; \
            } \
            VMCNT(0); PEND_TAG(); \
            while (rfl(lds_ld(lbase + L_PTAG + 12)) + NPS <= n) __builtin_amdgcn_s_sleep(1); \
            SLOT_TAG(0xffffu); } while (0)
        if (p != (unsigned)rank) {
            if (GATE1) while (rfl(lds_ld(lbase + L_PCALL + 12)) == 0u) __builtin_amdgcn_s_sleep(2);
            const char* const src1 = lane_ptr(pwl, p) + STAGE_OFF;
#define PREP1(T) do { T = item_of(T < 64u ? T : T + 64u, ROT); const unsigned kb_ = T / 96u, z_ = T - kb_ * 96u; \
                fbit = kb_ * 8 + (z_ < 64u ? z_ >> 3 : gp); \
                so = z_ < 64u ? p * SRC_BYTES + T * 4096 : EXPW + ((rank * KBS + kb_) * 32 + z_ - 64u) * 4096; } while (0)
            PULL_LOOP(ws + DRAW1 + g * 128, (call - 1) * (NITEM1 - 64 + NPULL1), NITEM1 - 64, src1, 2048, PREP1,
                      ws + EXPF + p * 512, 127, PW1, ((gi * 5) & 31) < NPULL1);
        }
        BAR();
        BAR();
        BAR();
        if (p != (unsigned)rank) {
            const char* const src2 = lane_ptr(pol, p);
            const unsigned skip = ((gp - SELFG) & 7u) * 256;
            rdy0 = rdy1 = bad ? ~0ull : 0ull;
#define PREP2(T) do { T = T < 1792u ? (SELFG * 256 + T + (T >= skip ? 256u : 0u)) & 2047u \
                                    : gp * 256 + ((T - 1792u) / BLK_ITEMS) * 64 + 8 * DPUSH + (T - 1792u) % BLK_ITEMS; \
                fbit = T >> 8; \
                so = p * NS + T * 2 * NN; } while (0)
            PULL_LOOP(ws + DRAW2 + g * 128, (call - 1) * (NITEM2 + NPULL2), NITEM2, src2, NN, PREP2, ws + RDY + p * 128, 7, PW2, ((gi * 5) & 31) < NPULL2);
        }
    } else {
        char* pwl = (char*)d_ws_all[lane & 7];
        char* pol = (char*)d_out_all[lane & 7];
        asm volatile("" : "+v"(pwl), "+v"(pol));
        BAR();
        GET_GROUP();
        const unsigned p = dest_of(rank, g);
        char* const pws = lane_ptr(pwl, p);
        const int r16 = lane & 15, q = lane >> 4;
        v4u o[4];
        unsigned n = 0, sl = 0, t;
#define TAG_WAIT() do { unsigned tg = rfl(lds_ld(lbase + L_PTAG + sl * 4)); \
            while ((tg >> 16) != n + 1) { __builtin_amdgcn_s_sleep(1); tg = rfl(lds_ld(lbase + L_PTAG + sl * 4)); } \
            t = tg & 0xffffu; } while (0)
#define SLOT_NEXT() do { ++n; lds_st(lbase + L_PTAG + 12, n); sl = sl + 1 == NPS ? 0 : sl + 1; } while (0)
#define SLOT_GET() do { _Pragma("unroll") for (int f = 0; f < 4; ++f) o[f] = *(const LAS v4u*)(smem + L_PSLOT + sl * 4096 + f * 1024 + lane * 16); \
            LGKM0(); PIN_O(); SLOT_NEXT(); } while (0)
        {
            const __amdgpu_buffer_rsrc_t rs = mk_rsrc(pws + STAGE_OFF + (size_t)rank * SRC_BYTES, 2 * BLK_BYTES);
#pragma unroll
            for (int j = 0; j < 2; ++j) {
                TAG_WAIT(); SLOT_GET();
#pragma unroll
                for (int f = 0; f < 4; ++f) {
                    VMCNT(1);
                    __builtin_amdgcn_raw_buffer_store_b128(o[f], rs, t * 4096 + lane * 16 + f * 1024, 0, AUX_SYS);
                    if (j * 4 + f + 1 == GATE1) lds_st(lbase + L_PCALL + 12, 1u);
                }
            }
            VMCNT(0);
            const unsigned cv = atomic_inc_async(ws + CNT1 + (g * 17 + 16) * 128);
            unsigned old;
            ATOMIC_RESULT(old, cv, 0);
            if (old + 1 == 32u * call && lane < 8) __hip_atomic_store((gu32*)(pws + AFLAG + lane * 1024) + (128 + rank), call, RLX_SYS);
        }
        if (p != (unsigned)rank) {
            const __amdgpu_buffer_rsrc_t rl = mk_rsrc(ws + STAGE_OFF + (size_t)p * SRC_BYTES, SRC_BYTES);
            for (;;) {
                TAG_WAIT();
                if (t == 0xffffu) { SLOT_NEXT(); break; }
                SLOT_GET();
                t &= 0x7fffu;
#pragma unroll
                for (int f = 0; f < 4; ++f) __builtin_amdgcn_raw_buffer_store_b128(o[f], rl, t * 4096 + lane * 16 + f * 1024, 0, AUX_DEV);
                const unsigned kb = t / 96u, z = t - kb * 96u;
                VMCNT(0);
                unsigned cv = atomic_inc_masked_async(ws + CNT1 + (g * 17 + kb) * 128, z < 64u ? 1u << (z >> 3) : 0xffu, lane * 4);
                ATOMIC_WAIT(cv, 0);
                if (cv + 1 == (kb < 2u ? 8u : 40u) * call)
                    __hip_atomic_store((gu32*)(ws + AFLAG + lane * 1024) + (kb * 8 + p), call, RLX_AGENT);
            }
        }
        STAMP(2);
        BAR();
        BAR();
        BAR();
        STAMP(6);
        if (p != (unsigned)rank) {
            const __amdgpu_buffer_rsrc_t rd = mk_rsrc(lane_ptr(pol, p), OUT_BYTES);
            for (int it = 0; it < DPUSH; ++it) {
#pragma unroll
                for (int u = 0; u < 4; ++u) {
                    const int row = it * 16 + u * 4 + q;
                    o[u] = *(const LAS v4u*)(smem + row * 256 + ((r16 ^ (row & 15)) * 16));
                }
                PIN_O();
#pragma unroll
                for (int u = 0; u < 4; ++u) {
                    VMCNT(1);
                    __builtin_amdgcn_raw_buffer_store_b128(o[u], rd, (u * 4 + q) * NN + r16 * 16 + (tm * 128 + it * 16) * NN + rank * NS + tn * 256, 0, AUX_SYS);
                }
            }
            lds_st(lbase + L_PCALL + 8, 1u);
            VMCNT(0);
            if (lane == 0) __hip_atomic_store((gu32*)(pws + FIN + rank * 128) + gi, call, RLX_SYS);
            const __amdgpu_buffer_rsrc_t rl = mk_rsrc(lane_ptr(pol, rank), OUT_BYTES);
            const unsigned vo = lane * 16 + p * NS;
            for (;;) {
                TAG_WAIT();
                if (t == 0xffffu) break;
                SLOT_GET();
                if (t & 0x8000u) { t &= 0x7fffu; o[0] = o[1] = o[2] = o[3] = (v4u){0x7f7f7f7fu, 0x7f7f7f7fu, 0x7f7f7f7fu, 0x7f7f7f7fu}; }
#pragma unroll
                for (int u = 0; u < 4; ++u)
                    __builtin_amdgcn_raw_buffer_store_b128(o[u], rl, vo + (u >> 1) * NN + (u & 1) * 1024 + t * 2 * NN, 0, AUX_DEV);
            }
            STAMP(8);
        } else if (gi == 0) {
            const unsigned long long t0 = rt();
            unsigned spins = 0;
            const __amdgpu_buffer_rsrc_t rw = mk_rsrc(ws + FIN, 1024);
            for (;;) {
                const v4u f = __builtin_amdgcn_raw_buffer_load_b128(rw, lane * 16, 0, AUX_SYS);
                if (__all((lane >> 3) == rank || (f[0] >= call && f[1] >= call && f[2] >= call && f[3] >= call))) break;
                __builtin_amdgcn_s_sleep(8);
                if ((++spins & 63u) == 0u && spin_expired(tmo, call, t0)) {
                    if (lane == 0) ((unsigned char*)d_out_all[rank])[(size_t)tm * 128 * NN + rank * NS + tn * 256] = 0x7f;
                    break;
                }
                asm volatile("" ::: "memory");
            }
            __builtin_amdgcn_fence(__ATOMIC_ACQUIRE, "");
            STAMP(9);
        }
    }
}

extern "C" void kernel_launch(int rank, int nranks, ncclComm_t comm, hipStream_t stream,
                              void* const* d_in, const int* in_sizes, int n_in,
                              void* d_out, int out_size,
                              const int* in_dtypes, int out_dtype,
                              void* const* d_out_all,
                              void* const* d_ws_all, size_t ws_bytes) {
    (void)comm; (void)d_out; (void)in_dtypes; (void)out_dtype;
    if (nranks != NR || (unsigned)rank >= (unsigned)NR || n_in < 2 || ws_bytes < WS_NEED ||
        in_sizes[0] != MM * KS || in_sizes[1] != KS * NN || out_size != MM * NN) {
        fprintf(stderr, "kernel_launch: unexpected problem shape\n");
        return;
    }
    hipLaunchKernelGGL(gemm_ar_q8, dim3(NWG), dim3(NTHREADS), 0, stream,
                       (const float*)d_in[0], (const float*)d_in[1], d_out_all, d_ws_all, rank);
    HIP_CHECK(hipGetLastError());
}
```

```cpp
#include <hip/hip_runtime.h>
#include <rccl/rccl.h>
#include <cstdio>
#include <cstdint>

#define HIP_CHECK(expr) do { hipError_t _e = (expr); if (_e != hipSuccess) { \
    fprintf(stderr, "HIP error %s at %s:%d\n", hipGetErrorString(_e), __FILE__, __LINE__); \
    return; } } while (0)

#define GAS __attribute__((address_space(1)))
#define LAS __attribute__((address_space(3)))
typedef unsigned v4u __attribute__((ext_vector_type(4)));
typedef unsigned v2u __attribute__((ext_vector_type(2)));
typedef float f32x4 __attribute__((ext_vector_type(4)));
typedef short bf16x8 __attribute__((ext_vector_type(8)));
typedef __bf16 bf2 __attribute__((ext_vector_type(2)));
typedef GAS unsigned gu32;
typedef GAS unsigned long long gu64;

namespace {

constexpr int NR = 8;
constexpr int MM = 4096, NN = 16384;
constexpr int KS = 512;
constexpr int NS = 2048;
constexpr int KBS = 16;
constexpr unsigned A_BYTES = 256u * 1024u;
constexpr unsigned B_BYTES = 128u * 1024u;
constexpr unsigned BLK_BYTES = A_BYTES + B_BYTES;
constexpr unsigned SRC_BYTES = KBS * BLK_BYTES;
constexpr size_t STAGE_OFF = 1u << 20;
constexpr unsigned EXPW = NR * SRC_BYTES;
constexpr unsigned DATA_BYTES = EXPW + NR * KBS * B_BYTES;
constexpr size_t WS_NEED = STAGE_OFF + DATA_BYTES;
constexpr unsigned OUT_BYTES = (unsigned)MM * NN;

constexpr unsigned DRAW1 = 0x0000;
constexpr unsigned DRAW2 = 0x0400;
constexpr unsigned TMO   = 0x0800;
constexpr unsigned EXPF  = 0x1000;
constexpr unsigned AFLAG = 0x2000;
constexpr unsigned GMAX  = 0x5000;
constexpr unsigned RMAX  = 0x6000;
constexpr unsigned RCNT  = 0x8000;
constexpr unsigned RDY   = 0x8400;
constexpr unsigned FIN   = 0x9000;
constexpr unsigned CNT1  = 0x28000;
constexpr unsigned ECNT  = 0x30000;
constexpr unsigned PROF  = 0x10000;
constexpr unsigned PRIV  = 0x20000;
constexpr unsigned CLAIM = 0x24000;

constexpr int NWG = 256, NTHREADS = 896;
constexpr int NSLOT = 6;
constexpr unsigned SLOT_BYTES = 24576;
constexpr unsigned L_FULL = NSLOT * SLOT_BYTES;
constexpr unsigned L_FREE = L_FULL + 16;
constexpr unsigned L_WMAX = L_FREE + 32;
constexpr unsigned L_MISC = L_WMAX + 32;
constexpr unsigned L_PTAG = L_MISC + 16;
constexpr unsigned L_PCALL = L_PTAG + 16;
constexpr unsigned L_SCHED = L_PCALL + 16;
constexpr unsigned L_PSLOT = L_SCHED + 512;
constexpr unsigned NPS = 3;
constexpr unsigned LDS_BYTES = L_PSLOT + NPS * 4096;
static_assert(LDS_BYTES <= 163840 && L_PSLOT % 16 == 0 && L_MISC % 16 == 0, "LDS layout");
constexpr int NSTEP = NR * KBS;
constexpr unsigned long long DEADLINE = 100000000ull;

#define DPUSH 2
#define FDEPTH 2
#define NPULL1 22
#define NPULL2 28
#define ROT (p + rank)
#define GATE1 0
#define FENCE0 0
#define CUT2 0
#define SLOWCNT 92u
#define POLL_REP 3
#define CONV_FIRST 0u
#define LPACE 0u
#define LPACE0 0u
#define GATE2 1
#define PUSHW 1
#define PW1 2
#define PW2 2
#define CONV_T0 400u
#define CONV_TAU 384u
#define POLL_SLOW 127
#define POLL_MISS 16
#define POLL_MID 32
#define FASTCNT 112u
#define FASTCNT2 124u
constexpr int SELFG = 3;
constexpr unsigned NITEM1 = KBS * 96;
constexpr unsigned NCONV = 14 * 40;
constexpr unsigned BLK_ITEMS = 64 - 8 * DPUSH;
constexpr unsigned NITEM2 = 7 * 256 + 4 * BLK_ITEMS;

__device__ __forceinline__ unsigned dest_of(int rank, int g) {
    const unsigned set = (0xB2u >> rank) & 1u;
    const unsigned pos = (0x33212100u >> (4 * rank)) & 3u;
    const unsigned members = ((g < 4) == (set == 0)) ? 0x6320u : 0x7541u;
    return (members >> (4 * ((pos + 1 + g) & 3))) & 7u;
}

#define VMCNT_(n) asm volatile("s_waitcnt vmcnt(" #n ")" ::: "memory")
#define VMCNT(n) VMCNT_(n)
#define LGKM0() asm volatile("s_waitcnt lgkmcnt(0)" ::: "memory")
#define BAR() do { asm volatile("s_waitcnt lgkmcnt(0)" ::: "memory"); __builtin_amdgcn_s_barrier(); asm volatile("" ::: "memory"); } while (0)
#define RLX_AGENT __ATOMIC_RELAXED, __HIP_MEMORY_SCOPE_AGENT
#define RLX_SYS __ATOMIC_RELAXED, __HIP_MEMORY_SCOPE_SYSTEM

__device__ __forceinline__ unsigned long long rt() { return __builtin_amdgcn_s_memrealtime(); }
__device__ __forceinline__ unsigned rfl(unsigned v) { return (unsigned)__builtin_amdgcn_readfirstlane((int)v); }
__device__ __forceinline__ unsigned xcc_id() {
    unsigned v;
    asm volatile("s_getreg_b32 %0, hwreg(HW_REG_XCC_ID)" : "=s"(v));
    return v & 7u;
}
__device__ __forceinline__ unsigned pk_bf16(float lo, float hi) {
    bf2 v; v.x = (__bf16)lo; v.y = (__bf16)hi; return __builtin_bit_cast(unsigned, v);
}
__device__ __forceinline__ unsigned wave_umax(unsigned v) {
    v = max(v, (unsigned)__builtin_amdgcn_update_dpp(0, (int)v, 0x128, 0xf, 0xf, false));
    v = max(v, (unsigned)__builtin_amdgcn_update_dpp(0, (int)v, 0x124, 0xf, 0xf, false));
    v = max(v, (unsigned)__builtin_amdgcn_update_dpp(0, (int)v, 0x122, 0xf, 0xf, false));
    v = max(v, (unsigned)__builtin_amdgcn_update_dpp(0, (int)v, 0x121, 0xf, 0xf, false));
    const unsigned a = (unsigned)__builtin_amdgcn_readlane((int)v, 0), b = (unsigned)__builtin_amdgcn_readlane((int)v, 16);
    const unsigned c = (unsigned)__builtin_amdgcn_readlane((int)v, 32), d = (unsigned)__builtin_amdgcn_readlane((int)v, 48);
    return max(max(a, b), max(c, d));
}
__device__ __forceinline__ char* lane_ptr(const void* v, unsigned l) {
    const unsigned long long u = (unsigned long long)v;
    return (char*)(((unsigned long long)(unsigned)__builtin_amdgcn_readlane((int)(u >> 32), (int)l) << 32) |
                   (unsigned)__builtin_amdgcn_readlane((int)u, (int)l));
}
__device__ __forceinline__ void lds_st(unsigned addr, unsigned v) { asm volatile("ds_write_b32 %0, %1" :: "v"(addr), "v"(v) : "memory"); }
__device__ __forceinline__ unsigned lds_ld(unsigned addr) {
    unsigned v; asm volatile("ds_read_b32 %0, %1\n\ts_waitcnt lgkmcnt(0)" : "=&v"(v) : "v"(addr) : "memory"); return v;
}
__device__ __forceinline__ void glds4(unsigned voff, const char* sbase, unsigned m0v) {
    unsigned keep;
    asm volatile("s_mov_b32 %0, m0\n\ts_mov_b32 m0, %3\n\ts_nop 4\n\t"
                 "global_load_lds_dwordx4 %1, %2 sc0 sc1\n\t"
                 "global_load_lds_dwordx4 %1, %2 offset:1024 sc0 sc1\n\t"
                 "global_load_lds_dwordx4 %1, %2 offset:2048 sc0 sc1\n\t"
                 "global_load_lds_dwordx4 %1, %2 offset:3072 sc0 sc1\n\t"
                 "s_mov_b32 m0, %0"
                 : "=&s"(keep) : "v"(voff), "s"(sbase), "s"(m0v) : "memory");
}
template <int IMM, bool CUT = false> __device__ __forceinline__ void glds1(unsigned voff, const char* sbase, unsigned m0v) {
    unsigned keep;
    if (CUT) {
        asm volatile("s_mov_b32 %0, m0\n\ts_mov_b32 m0, %3\n\ts_mov_b32 exec_hi, 0x0fffffff\n\ts_nop 3\n\t"
                     "global_load_lds_dwordx4 %1, %2 offset:%4 sc0 sc1\n\t"
                     "s_mov_b32 exec_hi, -1\n\ts_mov_b32 m0, %0"
                     : "=&s"(keep) : "v"(voff), "s"(sbase), "s"(m0v), "i"(IMM) : "memory");
        return;
    }
    asm volatile("s_mov_b32 %0, m0\n\ts_mov_b32 m0, %3\n\ts_nop 4\n\t"
                 "global_load_lds_dwordx4 %1, %2 offset:%4 sc0 sc1\n\t"
                 "s_mov_b32 m0, %0"
                 : "=&s"(keep) : "v"(voff), "s"(sbase), "s"(m0v), "i"(IMM) : "memory");
}
__device__ __forceinline__ __amdgpu_buffer_rsrc_t mk_rsrc(void* base, unsigned bytes) {
    return __builtin_amdgcn_make_buffer_rsrc(base, 0, (int)bytes, 0x00020000);
}
__device__ __forceinline__ unsigned atomic_inc_async(const void* p) {
    unsigned old; unsigned long long sv;
    asm volatile("s_mov_b64 %1, exec\n\ts_mov_b64 exec, 1\n\ts_nop 3\n\t"
                 "global_atomic_add %0, %2, %3, %4 sc0\n\t"
                 "s_mov_b64 exec, %1"
                 : "=&v"(old), "=&s"(sv) : "v"(0u), "v"(1u), "s"(p) : "memory");
    return old;
}
#define ATOMIC_RESULT(dst, old, n) ATOMIC_RESULT_(dst, old, n)
#define ATOMIC_RESULT_(dst, old, n) asm volatile("s_waitcnt vmcnt(" #n ")\n\tv_readfirstlane_b32 %0, %1\n\ts_nop 4" : "=s"(dst) : "v"(old) : "memory")
__device__ __forceinline__ unsigned atomic_inc_masked_async(const void* p, unsigned mask, unsigned lane4) {
    unsigned old = 0; unsigned long long sv;
    asm volatile("s_mov_b64 %1, exec\n\ts_mov_b32 exec_lo, %5\n\ts_mov_b32 exec_hi, 0\n\ts_nop 2\n\t"
                 "global_atomic_add %0, %2, %3, %4 sc0\n\t"
                 "s_mov_b64 exec, %1"
                 : "+v"(old), "=&s"(sv) : "v"(lane4), "v"(1u), "s"(p), "s"(rfl(mask)) : "memory");
    return old;
}
#define ATOMIC_WAIT(old, n) asm volatile("s_waitcnt vmcnt(" #n ")" : "+v"(old) :: "memory")
constexpr int AUX_SYS = 17;
constexpr int AUX_DEV = 16;

__device__ __forceinline__ bool spin_expired(gu32* tmo, unsigned call, unsigned long long t0) {
    if (rfl(__hip_atomic_load(tmo, RLX_AGENT)) == call || rt() - t0 > DEADLINE) {
        __hip_atomic_store(tmo, call, RLX_AGENT);
        return true;
    }
    return false;
}

__device__ __forceinline__ void load_item(f32x4 (&buf)[8], const float* x, const float* w, unsigned kb, unsigned z, int lane) {
    const int r16 = lane & 15, q = lane >> 4;
    if (z < 64u) {
        const float* a = x + (size_t)(z * 64 + r16) * KS + kb * 32 + 4 * q;
#pragma unroll
        for (int e = 0; e < 8; ++e) buf[e] = *(const f32x4*)(a + (e >> 1) * 16 * KS + (e & 1) * 16);
    } else {
        const float* b = w + (size_t)(kb * 32 + 4 * q) * NN + (z - 64u) * 64 + 4 * r16;
#pragma unroll
        for (int e = 0; e < 8; ++e) buf[e] = *(const f32x4*)(b + (size_t)((e >> 2) * 16 + (e & 3)) * NN);
    }
}
__device__ __forceinline__ void cvt_item(v4u (&o)[4], const f32x4 (&buf)[8], bool is_x) {
    if (is_x) {
#pragma unroll
        for (int f = 0; f < 4; ++f) {
            o[f][0] = pk_bf16(buf[2*f][0], buf[2*f][1]); o[f][1] = pk_bf16(buf[2*f][2], buf[2*f][3]);
            o[f][2] = pk_bf16(buf[2*f+1][0], buf[2*f+1][1]); o[f][3] = pk_bf16(buf[2*f+1][2], buf[2*f+1][3]);
        }
    } else {
#pragma unroll
        for (int j = 0; j < 4; ++j) {
            o[j][0] = pk_bf16(buf[0][j], buf[1][j]); o[j][1] = pk_bf16(buf[2][j], buf[3][j]);
            o[j][2] = pk_bf16(buf[4][j], buf[5][j]); o[j][3] = pk_bf16(buf[6][j], buf[7][j]);
        }
    }
}
__device__ __forceinline__ unsigned item_of(unsigned t, unsigned src) {
    if (t < 128u) return t < 64u ? (t + src * 8) & 63u : (t >= 96u ? 96u : 0u) + 64u + (t & 31u);
    const unsigned k = (t - 128u) / 96u, u = t - 128u - k * 96u, j = u / 12u, v = u - j * 12u;
    if (k == 14u) return 15u * 96u + ((u + src * 8) & 63u);
    if (v < 8u) return (k + 1u) * 96u + ((j + src) & 7u) * 8u + v;
    return (k + 2u) * 96u + 64u + j * 4u + v - 8u;
}

#define STAMP(idx) do {} while (0)

}

__global__ void __launch_bounds__(NTHREADS, 4)
gemm_ar_q8(const float* __restrict__ x, const float* __restrict__ w, void* const* __restrict__ d_out_all,
           void* const* __restrict__ d_ws_all, size_t ws_off, int rank) {
    __shared__ __attribute__((aligned(1024))) unsigned char smem[LDS_BYTES];
    const unsigned lbase = (unsigned)(size_t)(LAS unsigned char*)smem;
    const int lane = threadIdx.x & 63;
    const int wave = (int)rfl(threadIdx.x >> 6);
    const int gi = blockIdx.x >> 3;
    char* const ws = (char*)d_ws_all[rank] + ws_off;
    gu32* const tmo = (gu32*)(ws + TMO);
    const int tn = gi & 7;
#define GET_GROUP() const v2u gc_ = *(volatile LAS v2u*)(smem + L_PCALL); \
    const unsigned call = rfl(gc_[0]); const int g = (int)(rfl(gc_[1]) & 7u); const int tm = g * 4 + (gi >> 3); (void)call; (void)tm
#define PIN_O() asm volatile("" : "+v"(o[0]), "+v"(o[1]), "+v"(o[2]), "+v"(o[3]) :: "memory")

#define CONVERT(C) do { \
        f32x4 buf[8]; v4u o[4]; \
        const unsigned d = dest_of(rank, g), kb = 2u + (C) / 40u, u = (C) + 80u - kb * 40u; \
        load_item(buf, x, w, kb, u < 32u ? 64u + d * 32 + u : g * 8 + u - 32u, lane); \
        cvt_item(o, buf, u >= 32u); \
        const unsigned off = u >= 32u ? (rank * KBS + kb) * BLK_BYTES + (g * 8 + u - 32u) * 4096 \
                           : d == (unsigned)rank ? (rank * KBS + kb) * BLK_BYTES + A_BYTES + u * 4096 \
                           : EXPW + ((d * KBS + kb) * 32 + u) * 4096; \
        const __amdgpu_buffer_rsrc_t rs = mk_rsrc(ws + STAGE_OFF, DATA_BYTES); \
        _Pragma("unroll") for (int f = 0; f < 4; ++f) __builtin_amdgcn_raw_buffer_store_b128(o[f], rs, off + lane * 16 + f * 1024, 0, AUX_SYS); \
        VMCNT(0); \
        const unsigned cv = atomic_inc_async(ws + ECNT + (g * 16 + kb) * 128); \
        unsigned old; \
        ATOMIC_RESULT(old, cv, 0); \
        if (old + 1 == 40u * call && lane < 8) __hip_atomic_store((gu32*)(pw + EXPF + rank * 512) + (kb * 8 + g), call, RLX_SYS); \
    } while (0)

    if (wave == 13) {
        char* pw = (char*)d_ws_all[lane & 7] + ws_off;
        BAR();
        GET_GROUP();
        const unsigned long long t0 = rt();
        for (unsigned j = 0; j * 32 + gi < NCONV; j += j == 0 ? 2 : 1) {
            while (rt() - t0 < (j == 0 ? CONV_FIRST : CONV_T0 + (j - 2) * CONV_TAU + gi * (CONV_TAU / 32))) __builtin_amdgcn_s_sleep(4);
            CONVERT(j * 32 + gi);
        }
        STAMP(1);
        BAR();
        BAR();
        BAR();
    } else if (wave < 8) {
        char* pw = (char*)d_ws_all[lane & 7] + ws_off;
        {
            f32x4 buf[8];
            v4u o[4];
            const unsigned xcc = xcc_id();
            const unsigned kb = (wave >> 1) & 1, z = (wave & 1) * 32 + gi;
            const bool ahead = wave < 4 && (xcc & 7u) == (unsigned)SELFG;
            if (ahead) load_item(buf, x, w, kb, z, lane);
            BAR();
            const v2u gc = *(volatile LAS v2u*)(smem + L_PCALL);
            if (wave < 4 && (rfl(gc[1]) & 7u) == (unsigned)SELFG) {
                const unsigned call = rfl(gc[0]);
                if (!ahead) load_item(buf, x, w, kb, z, lane);
                cvt_item(o, buf, true);
                const __amdgpu_buffer_rsrc_t rs = mk_rsrc(ws + STAGE_OFF, DATA_BYTES);
#pragma unroll
                for (int f = 0; f < 4; ++f)
                    __builtin_amdgcn_raw_buffer_store_b128(o[f], rs, (rank * KBS + kb) * BLK_BYTES + z * 4096 + lane * 16 + f * 1024, 0, AUX_SYS);
                VMCNT(0);
                const unsigned cv = atomic_inc_async(ws + ECNT + (SELFG * 16 + kb) * 128);
                unsigned old;
                ATOMIC_RESULT(old, cv, 0);
                if (old + 1 == 64u * call) __hip_atomic_store((gu32*)(pw + EXPF + rank * 512) + (kb * 8 + (lane >> 3)), call, RLX_SYS);
            }
        }
        GET_GROUP();
        if (wave == 4) {
            const unsigned long long t0 = rt();
            while (rt() - t0 < CONV_FIRST) __builtin_amdgcn_s_sleep(4);
            CONVERT(32u + gi);
        }
        const int wm = wave >> 2, wn = wave & 3;
        f32x4 acc[4][4];
#pragma unroll
        for (int i = 0; i < 4; ++i)
#pragma unroll
            for (int j = 0; j < 4; ++j) acc[i][j] = (f32x4){0.f, 0.f, 0.f, 0.f};
        const LAS unsigned char* abase = (const LAS unsigned char*)smem + wm * 4096 + lane * 16;
        const LAS unsigned char* bbase = (const LAS unsigned char*)smem + 8192 + wn * 4096 + lane * 16;
        volatile LAS v2u* full = (volatile LAS v2u*)(smem + L_FULL);
        unsigned slot_off = 0;
        for (unsigned s = 0; s < (unsigned)NSTEP; ++s) {
            for (;;) {
                const v2u f = *full;
                if (rfl(f.x) > s && rfl(f.y) > s) break;
                __builtin_amdgcn_s_sleep(1);
            }
            asm volatile("" ::: "memory");
            v4u a[4], b[4];
#pragma unroll
            for (int i = 0; i < 4; ++i) b[i] = *(const LAS v4u*)(bbase + slot_off + i * 1024);
#pragma unroll
            for (int i = 0; i < 4; ++i) a[i] = *(const LAS v4u*)(abase + slot_off + i * 1024);
            LGKM0();
            lds_st(lbase + L_FREE + wave * 4, s + 1);
            asm volatile("" : "+v"(a[0]), "+v"(a[1]), "+v"(a[2]), "+v"(a[3]), "+v"(b[0]), "+v"(b[1]), "+v"(b[2]), "+v"(b[3]) :: "memory");
#pragma unroll
            for (int i = 0; i < 4; ++i)
#pragma unroll
                for (int j = 0; j < 4; ++j)
                    acc[i][j] = __builtin_amdgcn_mfma_f32_16x16x32_bf16(__builtin_bit_cast(bf16x8, b[j]), __builtin_bit_cast(bf16x8, a[i]), acc[i][j], 0, 0, 0);
            slot_off += SLOT_BYTES;
            if (slot_off == NSLOT * SLOT_BYTES) slot_off = 0;
        }
        if (wave == 0) STAMP(4);
        float mx = 0.f;
#pragma unroll
        for (int i = 0; i < 4; ++i)
#pragma unroll
            for (int j = 0; j < 4; ++j)
#pragma unroll
                for (int r = 0; r < 4; ++r) mx = fmaxf(mx, fabsf(acc[i][j][r]));
        lds_st(lbase + L_WMAX + wave * 4, wave_umax(__builtin_bit_cast(unsigned, mx)));
        BAR();
        BAR();
        const v4u misc = *(volatile LAS v4u*)(smem + L_MISC);
        const float inv = __builtin_bit_cast(float, misc[0]);
        const unsigned bad = misc[2];
        {
            const int r16 = lane & 15, q = lane >> 4;
#pragma unroll
            for (int i = 0; i < 4; ++i) {
                v4u o;
#pragma unroll
                for (int r = 0; r < 4; ++r) {
                    int d = __builtin_amdgcn_cvt_pk_fp8_f32(acc[i][0][r] * inv, acc[i][1][r] * inv, 0, false);
                    d = __builtin_amdgcn_cvt_pk_fp8_f32(acc[i][2][r] * inv, acc[i][3][r] * inv, d, true);
                    o[r] = bad ? 0x7f7f7f7fu : (unsigned)d;
                }
                const int row = wm * 64 + i * 16 + r16;
                const int ch = (wn * 4 + q) ^ r16;
                *(LAS v4u*)(smem + row * 256 + ch * 16) = o;
            }
        }
        BAR();
        if (g != SELFG) {
            const unsigned long long t0 = rt();
            while (rfl(lds_ld(lbase + L_PCALL + 8)) == 0u) __builtin_amdgcn_s_sleep(4);
            const unsigned hold = LPACE0 + (((g - SELFG) & 7) - 1) * LPACE + gi * (LPACE / 32);
            while (rt() - t0 < hold) __builtin_amdgcn_s_sleep(8);
        }
        {
            const __amdgpu_buffer_rsrc_t ro = mk_rsrc(d_out_all[rank], OUT_BYTES);
#pragma unroll
            for (int u = 0; u < 4; ++u) {
                const int row = wave * 16 + u * 4 + (lane >> 4), c = lane & 15;
                const v4u v = *(const LAS v4u*)(smem + row * 256 + ((c ^ (row & 15)) * 16));
                __builtin_amdgcn_raw_buffer_store_b128(v, ro, (tm * 128 + row) * NN + rank * NS + tn * 256 + c * 16, 0, AUX_SYS);
            }
            VMCNT(0);
            unsigned old = 0;
            if (lane == 0) old = __hip_atomic_fetch_add((LAS unsigned*)(smem + L_MISC + 12), 1u, __ATOMIC_RELAXED, __HIP_MEMORY_SCOPE_WORKGROUP);
            if (rfl(old) == 7) {
                const unsigned cv = atomic_inc_async(ws + RCNT + g * 128);
                unsigned oc;
                ATOMIC_RESULT(oc, cv, 0);
                if (oc + 1 == 32u * call && lane < 8) __hip_atomic_store((gu32*)(pw + RDY + rank * 128) + g, call, RLX_SYS);
                STAMP(7);
            }
        }
    } else if (wave < 10) {
        const int lw = wave - 8;
        BAR();
        GET_GROUP();
        const char* stage = ws + STAGE_OFF;
        const unsigned voff = lane * 16;
        unsigned slot_off = 0, pub = 0, s = 0;
        const unsigned a_full = lbase + L_FULL + lw * 4;
        while (pub < (unsigned)NSTEP) {
            bool go = false;
            unsigned e = 0;
            if (s < (unsigned)NSTEP && s - pub < 5u) {
                const unsigned v = lds_ld(lane < 8 ? lbase + L_FREE + lane * 4 : lbase + L_SCHED + s * 4);
                e = (unsigned)__builtin_amdgcn_readlane((int)v, 8);
                go = e != 0xffffffffu && __all(lane >= 8 || v + NSLOT > s);
            }
            if (go) {
                const char* blk = stage + (size_t)(e & 127u) * BLK_BYTES;
                const unsigned dst = lbase + slot_off;
                if (lw == 0) {
                    glds4(voff, blk + tm * 8192, dst);
                    glds4(voff, blk + tm * 8192 + 4096, dst + 4096);
                    glds4(voff, blk + A_BYTES + tn * 16384, dst + 8192);
                } else {
                    glds4(voff, blk + A_BYTES + tn * 16384 + 4096, dst + 12288);
                    glds4(voff, blk + A_BYTES + tn * 16384 + 8192, dst + 16384);
                    glds4(voff, blk + A_BYTES + tn * 16384 + 12288, dst + 20480);
                }
                ++s;
                slot_off += SLOT_BYTES;
                if (slot_off == NSLOT * SLOT_BYTES) slot_off = 0;
            } else if (pub < s) {
                const unsigned infl = s - pub;
                if (infl == 1) VMCNT(0); else if (infl == 2) VMCNT(12); else if (infl == 3) VMCNT(24);
                else if (infl == 4) VMCNT(36); else VMCNT(48);
                ++pub;
                lds_st(a_full, pub);
            } else {
                __builtin_amdgcn_s_sleep(1);
            }
        }
        BAR();
        BAR();
        BAR();
    } else if (wave == 10) {
        const unsigned xcc = xcc_id();
        int g = (int)(xcc & 7u);
        unsigned tk = 0;
        if (lane < 2) tk = __hip_atomic_fetch_add((gu32*)(ws + (lane == 0 ? PRIV + blockIdx.x * 64 : CLAIM + (gi * 8 + g) * 64)), 1u, RLX_AGENT);
        lds_st(lbase + L_SCHED + lane * 4, 0xffffffffu);
        lds_st(lbase + L_SCHED + 256 + lane * 4, 0xffffffffu);
        if (lane < 28) lds_st(lbase + L_FULL + lane * 4, 0u);
        const unsigned call = rfl(tk) + 1;
        unsigned got = (unsigned)__builtin_amdgcn_readlane((int)tk, 1) + 1;
        for (int tries = 0; got != call && tries < 7; ++tries) {
            unsigned o2 = 0;
            if (lane == 0) {
                __hip_atomic_fetch_add((gu32*)(ws + CLAIM + (gi * 8 + g) * 64), 0xffffffffu, RLX_AGENT);
                o2 = __hip_atomic_fetch_add((gu32*)(ws + CLAIM + (gi * 8 + ((g + 1) & 7)) * 64), 1u, RLX_AGENT);
            }
            g = (g + 1) & 7;
            got = rfl(o2) + 1;
        }
        if (lane < 4) lds_st(lbase + L_PCALL + lane * 4, lane == 0 ? call : lane == 1 ? (unsigned)g : 0u);
        BAR();
        bool bad = false;
        {
            const bool own = (lane & 7) == rank;
            gu32* fa = own ? (gu32*)(ws + EXPF + rank * 512) + ((lane & 56) + g) : (gu32*)(ws + AFLAG + g * 1024) + lane;
            gu32* fb = own ? (gu32*)(ws + EXPF + rank * 512) + ((lane & 56) + SELFG) : fa;
            gu32* fc = lane < 16 ? (gu32*)(ws + AFLAG + g * 1024) + (128 + (lane & 7)) : fb;
            unsigned long long done0 = 0, done1 = 0;
            unsigned cnt = 0, spins = 0;
            const unsigned ent = (lane & 7) * KBS + (lane >> 3);
            const unsigned long long t0 = rt();
            while ((done0 & done1) != ~0ull) {
                const bool h = done0 == ~0ull;
                const unsigned a0 = __hip_atomic_load(h ? fa + 64 : fa, RLX_SYS), a1 = __hip_atomic_load(fa + 64, RLX_SYS);
                const unsigned b0 = __hip_atomic_load(h ? fb + 64 : fc, RLX_SYS), b1 = __hip_atomic_load(fb + 64, RLX_SYS);
                const unsigned long long n0 = __ballot(bad || (a0 >= call && b0 >= call)) & ~done0;
                const unsigned long long n1 = __ballot(bad || (a1 >= call && b1 >= call)) & ~done1;
                if (n0 | n1) {
                    const unsigned long long lt = (1ull << lane) - 1;
                    const unsigned c0 = __popcll(n0), c1 = __popcll(n1);
                    if ((n0 >> lane) & 1) lds_st(lbase + L_SCHED + (cnt + __popcll(n0 & lt)) * 4, ent);
                    if ((n1 >> lane) & 1) lds_st(lbase + L_SCHED + (cnt + c0 + __popcll(n1 & lt)) * 4, ent + 8);
                    cnt += c0 + c1;
                    done0 |= n0; done1 |= n1;
                    if (cnt < SLOWCNT) for (int j = 0; j < POLL_REP; ++j) __builtin_amdgcn_s_sleep(127);
                    else if (cnt < FASTCNT) __builtin_amdgcn_s_sleep(POLL_SLOW);
                    else if (cnt < FASTCNT2) __builtin_amdgcn_s_sleep(POLL_MID);
                } else {
                    if (cnt < FASTCNT) __builtin_amdgcn_s_sleep(POLL_MISS);
                    else if (cnt < FASTCNT2) __builtin_amdgcn_s_sleep(POLL_MID);
                    else __builtin_amdgcn_s_sleep(1);
                    if ((++spins & 63u) == 0u && spin_expired(tmo, call, t0)) bad = true;
                }
            }
        }
        STAMP(3);
        char* pw = (char*)d_ws_all[lane >> 3] + ws_off;
        asm volatile("" : "+v"(pw));
        BAR();
        {
            unsigned m = wave_umax(lds_ld(lbase + L_WMAX + (lane & 7) * 4));
            if (lane == 0) __hip_atomic_store((gu64*)(ws + GMAX + g * 256) + gi, ((unsigned long long)call << 32) | m, RLX_AGENT);
            const unsigned long long t0 = rt();
            unsigned spins = 0;
            if (gi == 0) {
                unsigned long long v;
                for (;;) {
                    v = __hip_atomic_load((gu64*)(ws + GMAX + g * 256) + (lane & 31), RLX_AGENT);
                    if (__all((unsigned)(v >> 32) == call)) break;
                    if ((++spins & 63u) == 0u && spin_expired(tmo, call, t0)) { bad = true; break; }
                }
                m = wave_umax((unsigned)v);
                __hip_atomic_store((gu64*)(pw + RMAX + (lane & 7) * 512) + (rank * 8 + g), ((unsigned long long)call << 32) | m, RLX_SYS);
            }
            unsigned long long v;
            for (;;) {
                v = __hip_atomic_load((gu64*)(ws + RMAX + g * 512) + lane, RLX_SYS);
                if (rfl(__hip_atomic_load(tmo, RLX_AGENT)) == call) bad = true;
                if (bad || __all((unsigned)(v >> 32) == call)) break;
                if ((++spins & 63u) == 0u && spin_expired(tmo, call, t0)) { bad = true; break; }
            }
            m = wave_umax((unsigned)v);
            const float inv = 448.0f / __builtin_bit_cast(float, m);
            if (lane < 3) lds_st(lbase + L_MISC + lane * 4, lane == 0 ? __builtin_bit_cast(unsigned, inv) : (bad ? 1u : 0u));
        }
        STAMP(5);
        BAR();
        BAR();
    } else if (wave == 11) {
        char* pwl = (char*)d_ws_all[lane & 7] + ws_off;
        char* pol = (char*)d_out_all[lane & 7];
        f32x4 buf[8];
        v4u o[4];
#define SLOT_TAG(T) do { lds_st(lbase + L_PTAG + sl * 4, ((n + 1) << 16) | (T)); ++n; sl = sl + 1 == NPS ? 0 : sl + 1; } while (0)
#define PUT_H(H) _Pragma("unroll") for (int u = 0; u < 2; ++u) \
            *(LAS v4u*)(smem + L_PSLOT + sl * 4096 + (2 * (H) + u) * 1024 + lane * 16) = o[2 * (H) + u]
        unsigned n = 0, sl = 0;
        bool bad = false;
        f32x4 buf2[8];
        const unsigned xcc = xcc_id();
        load_item(buf, x, w, 0u, 64u + dest_of(rank, xcc & 7u) * 32 + gi, lane);
        load_item(buf2, x, w, 1u, 64u + dest_of(rank, xcc & 7u) * 32 + gi, lane);
        BAR();
        GET_GROUP();
        const unsigned p = dest_of(rank, g);
        if ((unsigned)g != (xcc & 7u)) {
            load_item(buf, x, w, 0u, 64u + p * 32 + gi, lane);
            load_item(buf2, x, w, 1u, 64u + p * 32 + gi, lane);
        }
        cvt_item(o, buf, false); PUT_H(0); PUT_H(1);
        SLOT_TAG(64u + gi);
        cvt_item(o, buf2, false); PUT_H(0); PUT_H(1);
        SLOT_TAG(96u + 64u + gi);
        unsigned gp = 0;
        for (int j = 0; j < 8; ++j) if (dest_of((int)p, j) == (unsigned)rank) gp = j;
        unsigned long long rdy0 = 0, rdy1 = 0;
#define FLAG_SET(B) ((((B) < 64u ? rdy0 : rdy1) >> ((B) & 63u)) & 1u)
#define NEED_FLAG(FP, MASK, B) do { if (!FLAG_SET(B)) { \
                const unsigned long long t0 = rt(); unsigned spins = 0; \
                for (;;) { \
                    const unsigned f0 = __hip_atomic_load((gu32*)(FP) + (lane & (MASK)), RLX_SYS); \
                    const unsigned f1 = __hip_atomic_load((gu32*)(FP) + ((lane + 64) & (MASK)), RLX_SYS); \
                    rdy0 = __ballot(f0 >= call); rdy1 = __ballot(f1 >= call); \
                    if (FLAG_SET(B)) break; \
                    __builtin_amdgcn_s_sleep(2); \
                    if ((++spins & 63u) == 0u && spin_expired(tmo, call, t0)) { bad = true; rdy0 = rdy1 = ~0ull; break; } \
                } \
                asm volatile("" ::: "memory"); } } while (0)
#define PROG(C, H) do {} while (0)
#define STARTST() do {} while (0)
#define PEND_TAG() do { if (pend != 0xffffffffu) { lds_st(lbase + L_PTAG + psl * 4, (n << 16) | pend | (bad ? 0x8000u : 0u)); pend = 0xffffffffu; } } while (0)
#define PULL_LOOP(DRAWP, BASE, NITEM, SRC, HSTEP, PREP, FP, FMASK, W, ON) do { \
            unsigned d = 0, so, fbit, pend = 0xffffffffu, psl = 0, dv; \
            if (ON) { dv = atomic_inc_async(DRAWP); ATOMIC_RESULT(d, dv, 0); } \
            while (ON) { \
                unsigned cur = d - (BASE); \
                if (cur >= (NITEM)) break; \
                PROG(cur, HSTEP); \
                PREP(cur); \
                if (!FLAG_SET(fbit) || rfl(lds_ld(lbase + L_PTAG + 12)) + NPS <= n) { \
                    VMCNT(0); PEND_TAG(); \
                    NEED_FLAG(FP, FMASK, fbit); \
                    while (rfl(lds_ld(lbase + L_PTAG + 12)) + NPS <= n) __builtin_amdgcn_s_sleep(1); \
                } \
                STARTST(); \
                const char* sb = (SRC) + rfl(so); \
                const unsigned dst = lbase + L_PSLOT + sl * 4096; \
                if (W == 2) VMCNT(1); else if (W == 3) VMCNT(2); else VMCNT(0); \
                if (W == 1) PEND_TAG(); \
                glds1<0>(lane * 16, sb, dst); \
                dv = atomic_inc_async(DRAWP); \
                if (W == 2) { VMCNT(2); PEND_TAG(); } else if (W == 3) VMCNT(3); else VMCNT(1); \
                glds1<1024, CUT2 && (HSTEP) != 2048>(lane * 16, sb, dst); \
                if (W == 2) VMCNT(2); else if (W == 3) { VMCNT(3); PEND_TAG(); } else VMCNT(0); \
                glds1<2048>(lane * 16, sb + ((HSTEP) - 2048), dst); \
                if (W == 2) VMCNT(1); else if (W == 3) VMCNT(3); else VMCNT(0); \
                glds1<3072, CUT2 && (HSTEP) != 2048>(lane * 16, sb + ((HSTEP) - 2048), dst); \
                ATOMIC_RESULT(d, dv, 3); \
                pend = cur; psl = sl; ++n; sl = sl + 1 == NPS ? 0 : sl + 1; \
            } \
            VMCNT(0); PEND_TAG(); \
            while (rfl(lds_ld(lbase + L_PTAG + 12)) + NPS <= n) __builtin_amdgcn_s_sleep(1); \
            SLOT_TAG(0xffffu); } while (0)
        if (p != (unsigned)rank) {
            if (GATE1) while (rfl(lds_ld(lbase + L_PCALL + 12)) == 0u) __builtin_amdgcn_s_sleep(2);
            const char* const src1 = lane_ptr(pwl, p) + STAGE_OFF;
#define PREP1(T) do { T = item_of(T < 64u ? T : T + 64u, ROT); const unsigned kb_ = T / 96u, z_ = T - kb_ * 96u; \
                fbit = kb_ * 8 + (z_ < 64u ? z_ >> 3 : gp); \
                so = z_ < 64u ? p * SRC_BYTES + T * 4096 : EXPW + ((rank * KBS + kb_) * 32 + z_ - 64u) * 4096; } while (0)
            PULL_LOOP(ws + DRAW1 + g * 128, (call - 1) * (NITEM1 - 64 + NPULL1), NITEM1 - 64, src1, 2048, PREP1,
                      ws + EXPF + p * 512, 127, PW1, ((gi * 5) & 31) < NPULL1);
        }
        BAR();
        BAR();
        BAR();
        if (p != (unsigned)rank) {
            const char* const src2 = lane_ptr(pol, p);
            const unsigned skip = ((gp - SELFG) & 7u) * 256;
            rdy0 = rdy1 = bad ? ~0ull : 0ull;
            if (GATE2) while (rfl(lds_ld(lbase + L_PCALL + 8)) == 0u) __builtin_amdgcn_s_sleep(2);
#define PREP2(T) do { T = T < 1792u ? (SELFG * 256 + T + (T >= skip ? 256u : 0u)) & 2047u \
                                    : gp * 256 + ((T - 1792u) / BLK_ITEMS) * 64 + 8 * DPUSH + (T - 1792u) % BLK_ITEMS; \
                fbit = T >> 8; \
                so = p * NS + T * 2 * NN; } while (0)
            PULL_LOOP(ws + DRAW2 + g * 128, (call - 1) * (NITEM2 + NPULL2), NITEM2, src2, NN, PREP2, ws + RDY + p * 128, 7, PW2, ((gi * 5) & 31) < NPULL2);
        }
    } else {
        char* pwl = (char*)d_ws_all[lane & 7] + ws_off;
        char* pol = (char*)d_out_all[lane & 7];
        asm volatile("" : "+v"(pwl), "+v"(pol));
        BAR();
        GET_GROUP();
        const unsigned p = dest_of(rank, g);
        char* const pws = lane_ptr(pwl, p);
        const int r16 = lane & 15, q = lane >> 4;
        v4u o[4];
        unsigned n = 0, sl = 0, t;
#define TAG_WAIT() do { unsigned tg = rfl(lds_ld(lbase + L_PTAG + sl * 4)); \
            while ((tg >> 16) != n + 1) { __builtin_amdgcn_s_sleep(1); tg = rfl(lds_ld(lbase + L_PTAG + sl * 4)); } \
            t = tg & 0xffffu; } while (0)
#define SLOT_NEXT() do { ++n; lds_st(lbase + L_PTAG + 12, n); sl = sl + 1 == NPS ? 0 : sl + 1; } while (0)
#define SLOT_GET() do { _Pragma("unroll") for (int f = 0; f < 4; ++f) o[f] = *(const LAS v4u*)(smem + L_PSLOT + sl * 4096 + f * 1024 + lane * 16); \
            LGKM0(); PIN_O(); SLOT_NEXT(); } while (0)
        {
            const __amdgpu_buffer_rsrc_t rs = mk_rsrc(pws + STAGE_OFF + (size_t)rank * SRC_BYTES, 2 * BLK_BYTES);
#pragma unroll
            for (int j = 0; j < 2; ++j) {
                TAG_WAIT(); SLOT_GET();
#pragma unroll
                for (int f = 0; f < 4; ++f) {
                    VMCNT(1);
                    __builtin_amdgcn_raw_buffer_store_b128(o[f], rs, t * 4096 + lane * 16 + f * 1024, 0, AUX_SYS);
                    if (j * 4 + f + 1 == GATE1) lds_st(lbase + L_PCALL + 12, 1u);
                }
            }
            VMCNT(0);
            const unsigned cv = atomic_inc_async(ws + CNT1 + (g * 17 + 16) * 128);
            unsigned old;
            ATOMIC_RESULT(old, cv, 0);
            if (old + 1 == 32u * call && lane < 8) __hip_atomic_store((gu32*)(pws + AFLAG + lane * 1024) + (128 + rank), call, RLX_SYS);
        }
        if (p != (unsigned)rank) {
            const __amdgpu_buffer_rsrc_t rl = mk_rsrc(ws + STAGE_OFF + (size_t)p * SRC_BYTES, SRC_BYTES);
            for (;;) {
                TAG_WAIT();
                if (t == 0xffffu) { SLOT_NEXT(); break; }
                SLOT_GET();
                t &= 0x7fffu;
#pragma unroll
                for (int f = 0; f < 4; ++f) __builtin_amdgcn_raw_buffer_store_b128(o[f], rl, t * 4096 + lane * 16 + f * 1024, 0, AUX_DEV);
                const unsigned kb = t / 96u, z = t - kb * 96u;
                VMCNT(0);
                unsigned cv = atomic_inc_masked_async(ws + CNT1 + (g * 17 + kb) * 128, z < 64u ? 1u << (z >> 3) : 0xffu, lane * 4);
                ATOMIC_WAIT(cv, 0);
                if (cv + 1 == (kb < 2u ? 8u : 40u) * call)
                    __hip_atomic_store((gu32*)(ws + AFLAG + lane * 1024) + (kb * 8 + p), call, RLX_AGENT);
            }
        }
        STAMP(2);
        BAR();
        BAR();
        BAR();
        STAMP(6);
        if (p != (unsigned)rank) {
            const __amdgpu_buffer_rsrc_t rd = mk_rsrc(lane_ptr(pol, p), OUT_BYTES);
            for (int it = 0; it < DPUSH; ++it) {
#pragma unroll
                for (int u = 0; u < 4; ++u) {
                    const int row = it * 16 + u * 4 + q;
                    o[u] = *(const LAS v4u*)(smem + row * 256 + ((r16 ^ (row & 15)) * 16));
                }
                PIN_O();
#pragma unroll
                for (int u = 0; u < 4; ++u) {
                    VMCNT(PUSHW);
                    __builtin_amdgcn_raw_buffer_store_b128(o[u], rd, (u * 4 + q) * NN + r16 * 16 + (tm * 128 + it * 16) * NN + rank * NS + tn * 256, 0, AUX_SYS);
                }
            }
            lds_st(lbase + L_PCALL + 8, 1u);
            VMCNT(0);
            if (lane == 0) __hip_atomic_store((gu32*)(pws + FIN + rank * 128) + gi, call, RLX_SYS);
            const __amdgpu_buffer_rsrc_t rl = mk_rsrc(lane_ptr(pol, rank), OUT_BYTES);
            const unsigned vo = lane * 16 + p * NS;
            for (;;) {
                TAG_WAIT();
                if (t == 0xffffu) break;
                SLOT_GET();
                if (t & 0x8000u) { t &= 0x7fffu; o[0] = o[1] = o[2] = o[3] = (v4u){0x7f7f7f7fu, 0x7f7f7f7fu, 0x7f7f7f7fu, 0x7f7f7f7fu}; }
#pragma unroll
                for (int u = 0; u < 4; ++u)
                    __builtin_amdgcn_raw_buffer_store_b128(o[u], rl, vo + (u >> 1) * NN + (u & 1) * 1024 + t * 2 * NN, 0, AUX_DEV);
            }
            STAMP(8);
        } else if (gi == 0) {
            const unsigned long long t0 = rt();
            unsigned spins = 0;
            const __amdgpu_buffer_rsrc_t rw = mk_rsrc(ws + FIN, 1024);
            for (;;) {
                const v4u f = __builtin_amdgcn_raw_buffer_load_b128(rw, lane * 16, 0, AUX_SYS);
                if (__all((lane >> 3) == rank || (f[0] >= call && f[1] >= call && f[2] >= call && f[3] >= call))) break;
                __builtin_amdgcn_s_sleep(8);
                if ((++spins & 63u) == 0u && spin_expired(tmo, call, t0)) {
                    if (lane == 0) ((unsigned char*)d_out_all[rank])[(size_t)tm * 128 * NN + rank * NS + tn * 256] = 0x7f;
                    break;
                }
                asm volatile("" ::: "memory");
            }
            __builtin_amdgcn_fence(__ATOMIC_ACQUIRE, "");
            STAMP(9);
        }
    }
}

extern "C" void kernel_launch(int rank, int nranks, ncclComm_t comm, hipStream_t stream,
                              void* const* d_in, const int* in_sizes, int n_in,
                              void* d_out, int out_size,
                              const int* in_dtypes, int out_dtype,
                              void* const* d_out_all,
                              void* const* d_ws_all, size_t ws_bytes) {
    (void)comm; (void)d_out; (void)in_dtypes; (void)out_dtype;
    if (nranks != NR || (unsigned)rank >= (unsigned)NR || n_in < 2 || ws_bytes < WS_NEED ||
        in_sizes[0] != MM * KS || in_sizes[1] != KS * NN || out_size != MM * NN) {
        fprintf(stderr, "kernel_launch: unexpected problem shape\n");
        return;
    }
    const size_t ws_off = (ws_bytes - WS_NEED) & ~(size_t)((2u << 20) - 1);
    hipLaunchKernelGGL(gemm_ar_q8, dim3(NWG), dim3(NTHREADS), 0, stream,
                       (const float*)d_in[0], (const float*)d_in[1], d_out_all, d_ws_all, ws_off, rank);
    HIP_CHECK(hipGetLastError());
}
```

```cpp
#include <hip/hip_runtime.h>
#include <rccl/rccl.h>
#include <cstdio>
#include <cstdint>

#define HIP_CHECK(expr) do { hipError_t _e = (expr); if (_e != hipSuccess) { \
    fprintf(stderr, "HIP error %s at %s:%d\n", hipGetErrorString(_e), __FILE__, __LINE__); \
    return; } } while (0)

#define GAS __attribute__((address_space(1)))
#define LAS __attribute__((address_space(3)))
typedef unsigned v4u __attribute__((ext_vector_type(4)));
typedef unsigned v2u __attribute__((ext_vector_type(2)));
typedef float f32x4 __attribute__((ext_vector_type(4)));
typedef short bf16x8 __attribute__((ext_vector_type(8)));
typedef __bf16 bf2 __attribute__((ext_vector_type(2)));
typedef GAS unsigned gu32;
typedef GAS unsigned long long gu64;

namespace {

constexpr int NR = 8;
constexpr int MM = 4096, NN = 16384;
constexpr int KS = 512;
constexpr int NS = 2048;
constexpr int KBS = 16;
constexpr unsigned A_BYTES = 256u * 1024u;
constexpr unsigned B_BYTES = 128u * 1024u;
constexpr unsigned BLK_BYTES = A_BYTES + B_BYTES;
constexpr unsigned SRC_BYTES = KBS * BLK_BYTES;
constexpr size_t STAGE_OFF = 1u << 20;
constexpr unsigned EXPW = NR * SRC_BYTES;
constexpr unsigned XSTG = EXPW + NR * KBS * B_BYTES;
constexpr unsigned SIDE = XSTG + NR * KBS * NR * 4096;
constexpr unsigned DATA_BYTES = SIDE + MM * 64;
constexpr size_t WS_NEED = STAGE_OFF + DATA_BYTES;
constexpr unsigned OUT_BYTES = (unsigned)MM * NN;

constexpr unsigned DRAW1 = 0x0000;
constexpr unsigned DRAW2 = 0x0400;
constexpr unsigned TMO   = 0x0800;
constexpr unsigned XRDY  = 0x0900;
constexpr unsigned DRAW3 = 0x0C00;
constexpr unsigned XCNT  = 0x4000;
constexpr unsigned EXPF  = 0x1000;
constexpr unsigned AFLAG = 0x2000;
constexpr unsigned GMAX  = 0x5000;
constexpr unsigned RMAX  = 0x6000;
constexpr unsigned RCNT  = 0x8000;
constexpr unsigned RDY   = 0x8400;
constexpr unsigned FIN   = 0x9000;
constexpr unsigned CNT1  = 0x28000;
constexpr unsigned ECNT  = 0x30000;
constexpr unsigned PRIV  = 0x20000;
constexpr unsigned CLAIM = 0x24000;

constexpr int NWG = 256, NTHREADS = 896;
constexpr int NSLOT = 6;
constexpr unsigned SLOT_BYTES = 24576;
constexpr unsigned L_FULL = NSLOT * SLOT_BYTES;
constexpr unsigned L_FREE = L_FULL + 16;
constexpr unsigned L_WMAX = L_FREE + 32;
constexpr unsigned L_MISC = L_WMAX + 32;
constexpr unsigned L_PTAG = L_MISC + 16;
constexpr unsigned L_PCALL = L_PTAG + 16;
constexpr unsigned L_SYNC = L_PCALL + 16;
constexpr unsigned L_SCHED = L_SYNC + 16;
constexpr unsigned L_PSLOT = L_SCHED + 512;
constexpr unsigned NPS = 3;
constexpr unsigned LDS_BYTES = L_PSLOT + NPS * 4096;
static_assert(LDS_BYTES <= 163840 && L_PSLOT % 16 == 0 && L_MISC % 16 == 0, "LDS layout");
constexpr int NSTEP = NR * KBS;
constexpr unsigned long long DEADLINE = 100000000ull;

#define DPUSH 1
#define NPULL1 22
#define NPULL2 28
#define ROT (p + rank)
#define SLOWCNT 92u
#define POLL_REP 3
#define PUSHW 1
#define NPULL3 16
#define CSLEEP 2
#define PW1 2
#define PW2 2
#define CONV_T0 400u
#define CONV_TAU 384u
#define POLL_SLOW 127
#define POLL_MISS 16
#define POLL_MID 32
#define FASTCNT 112u
#define FASTCNT2 124u
constexpr int SELFG = 3;
constexpr unsigned NITEM1 = KBS * 96;
constexpr unsigned NCONV = 14 * 40;
constexpr unsigned BLK_ITEMS = 64 - 8 * DPUSH;
constexpr unsigned NROWP = 7 * 256 + 4 * BLK_ITEMS;
constexpr unsigned NITEM2 = NROWP + 64;
constexpr unsigned NITEM3 = 7 * KBS;

__device__ __forceinline__ unsigned dest_of(int rank, int g) {
    const unsigned set = (0xB2u >> rank) & 1u;
    const unsigned pos = (0x33212100u >> (4 * rank)) & 3u;
    const unsigned members = ((g < 4) == (set == 0)) ? 0x6320u : 0x7541u;
    return (members >> (4 * ((pos + 1 + g) & 3))) & 7u;
}

#define VMCNT_(n) asm volatile("s_waitcnt vmcnt(" #n ")" ::: "memory")
#define VMCNT(n) VMCNT_(n)
#define LGKM0() asm volatile("s_waitcnt lgkmcnt(0)" ::: "memory")
#define BAR() do { asm volatile("s_waitcnt lgkmcnt(0)" ::: "memory"); __builtin_amdgcn_s_barrier(); asm volatile("" ::: "memory"); } while (0)
#define RLX_AGENT __ATOMIC_RELAXED, __HIP_MEMORY_SCOPE_AGENT
#define RLX_SYS __ATOMIC_RELAXED, __HIP_MEMORY_SCOPE_SYSTEM

__device__ __forceinline__ unsigned long long rt() { return __builtin_amdgcn_s_memrealtime(); }
__device__ __forceinline__ unsigned rfl(unsigned v) { return (unsigned)__builtin_amdgcn_readfirstlane((int)v); }
__device__ __forceinline__ unsigned xcc_id() {
    unsigned v;
    asm volatile("s_getreg_b32 %0, hwreg(HW_REG_XCC_ID)" : "=s"(v));
    return v & 7u;
}
__device__ __forceinline__ unsigned pk_bf16(float lo, float hi) {
    bf2 v; v.x = (__bf16)lo; v.y = (__bf16)hi; return __builtin_bit_cast(unsigned, v);
}
__device__ __forceinline__ unsigned wave_umax(unsigned v) {
    v = max(v, (unsigned)__builtin_amdgcn_update_dpp(0, (int)v, 0x128, 0xf, 0xf, false));
    v = max(v, (unsigned)__builtin_amdgcn_update_dpp(0, (int)v, 0x124, 0xf, 0xf, false));
    v = max(v, (unsigned)__builtin_amdgcn_update_dpp(0, (int)v, 0x122, 0xf, 0xf, false));
    v = max(v, (unsigned)__builtin_amdgcn_update_dpp(0, (int)v, 0x121, 0xf, 0xf, false));
    const unsigned a = (unsigned)__builtin_amdgcn_readlane((int)v, 0), b = (unsigned)__builtin_amdgcn_readlane((int)v, 16);
    const unsigned c = (unsigned)__builtin_amdgcn_readlane((int)v, 32), d = (unsigned)__builtin_amdgcn_readlane((int)v, 48);
    return max(max(a, b), max(c, d));
}
__device__ __forceinline__ char* lane_ptr(const void* v, unsigned l) {
    const unsigned long long u = (unsigned long long)v;
    return (char*)(((unsigned long long)(unsigned)__builtin_amdgcn_readlane((int)(u >> 32), (int)l) << 32) |
                   (unsigned)__builtin_amdgcn_readlane((int)u, (int)l));
}
__device__ __forceinline__ void lds_st(unsigned addr, unsigned v) { asm volatile("ds_write_b32 %0, %1" :: "v"(addr), "v"(v) : "memory"); }
__device__ __forceinline__ unsigned lds_ld(unsigned addr) {
    unsigned v; asm volatile("ds_read_b32 %0, %1\n\ts_waitcnt lgkmcnt(0)" : "=&v"(v) : "v"(addr) : "memory"); return v;
}
__device__ __forceinline__ void glds4(unsigned voff, const char* sbase, unsigned m0v) {
    unsigned keep;
    asm volatile("s_mov_b32 %0, m0\n\ts_mov_b32 m0, %3\n\ts_nop 4\n\t"
                 "global_load_lds_dwordx4 %1, %2 sc0 sc1\n\t"
                 "global_load_lds_dwordx4 %1, %2 offset:1024 sc0 sc1\n\t"
                 "global_load_lds_dwordx4 %1, %2 offset:2048 sc0 sc1\n\t"
                 "global_load_lds_dwordx4 %1, %2 offset:3072 sc0 sc1\n\t"
                 "s_mov_b32 m0, %0"
                 : "=&s"(keep) : "v"(voff), "s"(sbase), "s"(m0v) : "memory");
}
template <int IMM> __device__ __forceinline__ void glds1(unsigned voff, const char* sbase, unsigned m0v, unsigned exec_hi) {
    unsigned keep;
    asm volatile("s_mov_b32 %0, m0\n\ts_mov_b32 m0, %3\n\ts_mov_b32 exec_hi, %5\n\ts_nop 3\n\t"
                 "global_load_lds_dwordx4 %1, %2 offset:%4 sc0 sc1\n\t"
                 "s_mov_b32 exec_hi, -1\n\ts_mov_b32 m0, %0"
                 : "=&s"(keep) : "v"(voff), "s"(sbase), "s"(m0v), "i"(IMM), "s"(rfl(exec_hi)) : "memory");
}
__device__ __forceinline__ __amdgpu_buffer_rsrc_t mk_rsrc(void* base, unsigned bytes) {
    return __builtin_amdgcn_make_buffer_rsrc(base, 0, (int)bytes, 0x00020000);
}
__device__ __forceinline__ unsigned atomic_inc_async(const void* p) {
    unsigned old; unsigned long long sv;
    asm volatile("s_mov_b64 %1, exec\n\ts_mov_b64 exec, 1\n\ts_nop 3\n\t"
                 "global_atomic_add %0, %2, %3, %4 sc0\n\t"
                 "s_mov_b64 exec, %1"
                 : "=&v"(old), "=&s"(sv) : "v"(0u), "v"(1u), "s"(p) : "memory");
    return old;
}
#define ATOMIC_RESULT(dst, old, n) ATOMIC_RESULT_(dst, old, n)
#define ATOMIC_RESULT_(dst, old, n) asm volatile("s_waitcnt vmcnt(" #n ")\n\tv_readfirstlane_b32 %0, %1\n\ts_nop 4" : "=s"(dst) : "v"(old) : "memory")
__device__ __forceinline__ unsigned atomic_inc_masked_async(const void* p, unsigned mask, unsigned lane4) {
    unsigned old = 0; unsigned long long sv;
    asm volatile("s_mov_b64 %1, exec\n\ts_mov_b32 exec_lo, %5\n\ts_mov_b32 exec_hi, 0\n\ts_nop 2\n\t"
                 "global_atomic_add %0, %2, %3, %4 sc0\n\t"
                 "s_mov_b64 exec, %1"
                 : "+v"(old), "=&s"(sv) : "v"(lane4), "v"(1u), "s"(p), "s"(rfl(mask)) : "memory");
    return old;
}
#define ATOMIC_WAIT(old, n) asm volatile("s_waitcnt vmcnt(" #n ")" : "+v"(old) :: "memory")
constexpr int AUX_SYS = 17;
constexpr int AUX_DEV = 16;

__device__ __forceinline__ bool spin_expired(gu32* tmo, unsigned call, unsigned long long t0) {
    if (rfl(__hip_atomic_load(tmo, RLX_AGENT)) == call || rt() - t0 > DEADLINE) {
        __hip_atomic_store(tmo, call, RLX_AGENT);
        return true;
    }
    return false;
}

__device__ __forceinline__ void load_item(f32x4 (&buf)[8], const float* x, const float* w, unsigned kb, unsigned z, int lane) {
    const int r16 = lane & 15, q = lane >> 4;
    if (z < 64u) {
        const float* a = x + (size_t)(z * 64 + r16) * KS + kb * 32 + 4 * q;
#pragma unroll
        for (int e = 0; e < 8; ++e) buf[e] = *(const f32x4*)(a + (e >> 1) * 16 * KS + (e & 1) * 16);
    } else {
        const float* b = w + (size_t)(kb * 32 + 4 * q) * NN + (z - 64u) * 64 + 4 * r16;
#pragma unroll
        for (int e = 0; e < 8; ++e) buf[e] = *(const f32x4*)(b + (size_t)((e >> 2) * 16 + (e & 3)) * NN);
    }
}
__device__ __forceinline__ void cvt_item(v4u (&o)[4], const f32x4 (&buf)[8], bool is_x) {
    if (is_x) {
#pragma unroll
        for (int f = 0; f < 4; ++f) {
            o[f][0] = pk_bf16(buf[2*f][0], buf[2*f][1]); o[f][1] = pk_bf16(buf[2*f][2], buf[2*f][3]);
            o[f][2] = pk_bf16(buf[2*f+1][0], buf[2*f+1][1]); o[f][3] = pk_bf16(buf[2*f+1][2], buf[2*f+1][3]);
        }
    } else {
#pragma unroll
        for (int j = 0; j < 4; ++j) {
            o[j][0] = pk_bf16(buf[0][j], buf[1][j]); o[j][1] = pk_bf16(buf[2][j], buf[3][j]);
            o[j][2] = pk_bf16(buf[4][j], buf[5][j]); o[j][3] = pk_bf16(buf[6][j], buf[7][j]);
        }
    }
}
__device__ __forceinline__ unsigned item_of(unsigned t, unsigned src) {
    if (t < 128u) return t < 64u ? (t + src * 8) & 63u : (t >= 96u ? 96u : 0u) + 64u + (t & 31u);
    const unsigned k = (t - 128u) / 96u, u = t - 128u - k * 96u, j = u / 12u, v = u - j * 12u;
    if (k == 14u) return 15u * 96u + ((u + src * 8) & 63u);
    if (v < 8u) return (k + 1u) * 96u + ((j + src) & 7u) * 8u + v;
    return (k + 2u) * 96u + 64u + j * 4u + v - 8u;
}


}

__global__ void __launch_bounds__(NTHREADS, 4)
gemm_ar_q8(const float* __restrict__ x, const float* __restrict__ w, void* const* __restrict__ d_out_all,
           void* const* __restrict__ d_ws_all, size_t ws_off, int rank) {
    __shared__ __attribute__((aligned(1024))) unsigned char smem[LDS_BYTES];
    const unsigned lbase = (unsigned)(size_t)(LAS unsigned char*)smem;
    const int lane = threadIdx.x & 63;
    const int wave = (int)rfl(threadIdx.x >> 6);
    const int gi = blockIdx.x >> 3;
    char* const ws = (char*)d_ws_all[rank] + ws_off;
    gu32* const tmo = (gu32*)(ws + TMO);
    const int tn = gi & 7;
#define GET_GROUP() const v2u gc_ = *(volatile LAS v2u*)(smem + L_PCALL); \
    const unsigned call = rfl(gc_[0]); const int g = (int)(rfl(gc_[1]) & 7u); const int tm = g * 4 + (gi >> 3); (void)call; (void)tm
#define PIN_O() asm volatile("" : "+v"(o[0]), "+v"(o[1]), "+v"(o[2]), "+v"(o[3]) :: "memory")

#define CONVERT(C) CONVERT_(2u + (C) / 40u, (C) % 40u, 2u + (C) / 40u)
#define CONVERT_(KB, U, CK) do { \
        f32x4 buf[8]; v4u o[4]; \
        const unsigned d = dest_of(rank, g), kb = (KB), u = (U), ck = (CK); \
        load_item(buf, x, w, kb, u < 32u ? 64u + d * 32 + u : g * 8 + u - 32u, lane); \
        cvt_item(o, buf, u >= 32u); \
        const unsigned off = u >= 32u ? (rank * KBS + kb) * BLK_BYTES + (g * 8 + u - 32u) * 4096 \
                           : d == (unsigned)rank ? (rank * KBS + kb) * BLK_BYTES + A_BYTES + u * 4096 \
                           : EXPW + ((d * KBS + kb) * 32 + u) * 4096; \
        const __amdgpu_buffer_rsrc_t rs = mk_rsrc(ws + STAGE_OFF, DATA_BYTES); \
        if (kb >= 2u) { _Pragma("unroll") for (int f = 0; f < 4; ++f) __builtin_amdgcn_raw_buffer_store_b128(o[f], rs, off + lane * 16 + f * 1024, 0, AUX_SYS); } \
        if (u == 31u) { _Pragma("unroll") for (int f = 0; f < 4; ++f) \
            __builtin_amdgcn_raw_buffer_store_b128(o[f], rs, XSTG + ((rank * KBS + kb) * NR + d) * 4096 + lane * 16 + f * 1024, 0, AUX_SYS); } \
        VMCNT(0); \
        const unsigned cv = atomic_inc_async(ws + ECNT + (g * 16 + ck) * 128); \
        unsigned old; \
        ATOMIC_RESULT(old, cv, 0); \
        if (old + 1 == (ck == 3u ? 42u : 40u) * call && lane < 8) __hip_atomic_store((gu32*)(pw + EXPF + rank * 512) + (ck * 8 + g), call, RLX_SYS); \
    } while (0)

    if (wave == 13) {
        char* pw = (char*)d_ws_all[lane & 7] + ws_off;
        BAR();
        GET_GROUP();
        const unsigned long long t0 = rt();
        for (unsigned j = 0; j * 32 + gi < NCONV; j += j == 0 ? 2 : 1) {
            while (rt() - t0 < (j == 0 ? 0u : CONV_T0 + (j - 2) * CONV_TAU + gi * (CONV_TAU / 32))) __builtin_amdgcn_s_sleep(4);
            CONVERT(j * 32 + gi);
        }
    } else if (wave < 8) {
        char* pw = (char*)d_ws_all[lane & 7] + ws_off;
        {
            f32x4 buf[8];
            v4u o[4];
            const unsigned xcc = xcc_id();
            const unsigned kb = (wave >> 1) & 1, z = (wave & 1) * 32 + gi;
            const bool ahead = wave < 4 && (xcc & 7u) == (unsigned)SELFG;
            if (ahead) load_item(buf, x, w, kb, z, lane);
            BAR();
            const v2u gc = *(volatile LAS v2u*)(smem + L_PCALL);
            if (wave < 4 && (rfl(gc[1]) & 7u) == (unsigned)SELFG) {
                const unsigned call = rfl(gc[0]);
                if (!ahead) load_item(buf, x, w, kb, z, lane);
                cvt_item(o, buf, true);
                const __amdgpu_buffer_rsrc_t rs = mk_rsrc(ws + STAGE_OFF, DATA_BYTES);
#pragma unroll
                for (int f = 0; f < 4; ++f)
                    __builtin_amdgcn_raw_buffer_store_b128(o[f], rs, (rank * KBS + kb) * BLK_BYTES + z * 4096 + lane * 16 + f * 1024, 0, AUX_SYS);
                VMCNT(0);
                const unsigned cv = atomic_inc_async(ws + ECNT + (SELFG * 16 + kb) * 128);
                unsigned old;
                ATOMIC_RESULT(old, cv, 0);
                if (old + 1 == 64u * call) __hip_atomic_store((gu32*)(pw + EXPF + rank * 512) + (kb * 8 + (lane >> 3)), call, RLX_SYS);
            }
        }
        GET_GROUP();
        if (wave == 4) CONVERT(32u + gi);
        if (wave == 5 && gi < 2) CONVERT_((unsigned)gi, 31u, 3u);
        const int wm = wave >> 2, wn = wave & 3;
        const LAS unsigned char* abase = (const LAS unsigned char*)smem + wm * 4096 + lane * 16;
        const LAS unsigned char* bbase = (const LAS unsigned char*)smem + 8192 + wn * 4096 + lane * 16;
        volatile LAS v2u* full = (volatile LAS v2u*)(smem + L_FULL);
        unsigned slot_off = 0, s = 0, bad = 0;
        float inv = 0.f;
        const __amdgpu_buffer_rsrc_t ro = mk_rsrc(d_out_all[rank], OUT_BYTES);
#define QUANT(O, I) _Pragma("unroll") for (int r = 0; r < 4; ++r) { \
            int d_ = __builtin_amdgcn_cvt_pk_fp8_f32(acc[I][0][r] * inv, acc[I][1][r] * inv, 0, false); \
            d_ = __builtin_amdgcn_cvt_pk_fp8_f32(acc[I][2][r] * inv, acc[I][3][r] * inv, d_, true); \
            O[r] = bad ? 0x7f7f7f7fu : (unsigned)d_; }
#define LDS_ARRIVE(OFF) do { LGKM0(); if (lane == 0) __hip_atomic_fetch_add((LAS unsigned*)(smem + (OFF)), 1u, __ATOMIC_RELAXED, __HIP_MEMORY_SCOPE_WORKGROUP); } while (0)
        for (unsigned pass = 0; pass < (tn < 2 ? 2u : 1u); ++pass) {
        f32x4 acc[4][4];
#pragma unroll
        for (int i = 0; i < 4; ++i)
#pragma unroll
            for (int j = 0; j < 4; ++j) acc[i][j] = (f32x4){0.f, 0.f, 0.f, 0.f};
        for (const unsigned send = s + NSTEP; s < send; ++s) {
            for (;;) {
                const v2u f = *full;
                if (rfl(f.x) > s && rfl(f.y) > s) break;
                __builtin_amdgcn_s_sleep(1);
            }
            asm volatile("" ::: "memory");
            v4u a[4], b[4];
#pragma unroll
            for (int i = 0; i < 4; ++i) b[i] = *(const LAS v4u*)(bbase + slot_off + i * 1024);
#pragma unroll
            for (int i = 0; i < 4; ++i) a[i] = *(const LAS v4u*)(abase + slot_off + i * 1024);
            LGKM0();
            lds_st(lbase + L_FREE + wave * 4, s + 1);
            asm volatile("" : "+v"(a[0]), "+v"(a[1]), "+v"(a[2]), "+v"(a[3]), "+v"(b[0]), "+v"(b[1]), "+v"(b[2]), "+v"(b[3]) :: "memory");
#pragma unroll
            for (int i = 0; i < 4; ++i)
#pragma unroll
                for (int j = 0; j < 4; ++j)
                    acc[i][j] = __builtin_amdgcn_mfma_f32_16x16x32_bf16(__builtin_bit_cast(bf16x8, b[j]), __builtin_bit_cast(bf16x8, a[i]), acc[i][j], 0, 0, 0);
            slot_off += SLOT_BYTES;
            if (slot_off == NSLOT * SLOT_BYTES) slot_off = 0;
        }
        int ln = lane;
        asm volatile("" : "+v"(ln));
        const int r16 = ln & 15, q = ln >> 4;
        if (pass > 0) {
            const unsigned d = tn * 4 + wn;
            bad |= rfl(lds_ld(lbase + L_SYNC + 8));
#pragma unroll
            for (int i = 0; i < 4; ++i) {
                v4u o;
                QUANT(o, i);
                if (d != (unsigned)rank)
                    __builtin_amdgcn_raw_buffer_store_b128(o, ro, (tm * 128 + wm * 64 + i * 16 + r16) * NN + d * NS + (NS - 64) + q * 16, 0, AUX_DEV);
            }
            continue;
        }
        float mx = 0.f;
#pragma unroll
        for (int i = 0; i < 4; ++i)
#pragma unroll
            for (int j = 0; j < 4; ++j)
#pragma unroll
                for (int r = 0; r < 4; ++r) mx = fmaxf(mx, fabsf(acc[i][j][r]));
        lds_st(lbase + L_WMAX + wave * 4, wave_umax(__builtin_bit_cast(unsigned, mx)));
        LDS_ARRIVE(L_SYNC);
        char* pw2 = (char*)d_ws_all[ln & 7] + ws_off;
        v4u misc;
        for (;;) { misc = *(volatile LAS v4u*)(smem + L_MISC); if (rfl(misc[1]) != 0u) break; __builtin_amdgcn_s_sleep(CSLEEP); }
        inv = __builtin_bit_cast(float, misc[0]);
        bad = misc[2];
#pragma unroll
        for (int i = 0; i < 4; ++i) {
            v4u o;
            QUANT(o, i);
            *(LAS v4u*)(smem + (wm * 64 + i * 16 + r16) * 256 + (((wn * 4 + q) ^ r16) * 16)) = o;
        }
        LDS_ARRIVE(L_SYNC + 4);
        while (rfl(lds_ld(lbase + L_SYNC + 4)) < 8u) {}
        if (g != SELFG) while (rfl(lds_ld(lbase + L_PCALL + 8)) == 0u) __builtin_amdgcn_s_sleep(4);
#pragma unroll
        for (int u = 0; u < 4; ++u) {
            const int row = wave * 16 + u * 4 + q;
            const v4u v = *(const LAS v4u*)(smem + row * 256 + ((r16 ^ (row & 15)) * 16));
            __builtin_amdgcn_raw_buffer_store_b128(v, ro, (tm * 128 + row) * NN + rank * NS + tn * 256 + r16 * 16, 0, AUX_SYS);
        }
        if (tn == 7) {
            const v4u v = *(const LAS v4u*)(smem + (wave * 16 + (ln >> 2)) * 256 + (((8 + (ln & 3)) ^ (ln >> 2)) * 16));
            __builtin_amdgcn_raw_buffer_store_b128(v, mk_rsrc(ws + STAGE_OFF + SIDE, MM * 64), (tm * 128 + wave * 16) * 64 + ln * 16, 0, AUX_SYS);
        }
        VMCNT(0);
        unsigned old = 0;
        if (lane == 0) old = __hip_atomic_fetch_add((LAS unsigned*)(smem + L_MISC + 12), 1u, __ATOMIC_RELAXED, __HIP_MEMORY_SCOPE_WORKGROUP);
        if (rfl(old) == 7) {
            const unsigned cv = atomic_inc_async(ws + RCNT + g * 128);
            unsigned oc;
            ATOMIC_RESULT(oc, cv, 0);
            if (oc + 1 == 32u * call && lane < 8) __hip_atomic_store((gu32*)(pw2 + RDY + rank * 128) + g, call, RLX_SYS);
        }
        }
    } else if (wave < 10) {
        const int lw = wave - 8;
        BAR();
        GET_GROUP();
        const char* stage = ws + STAGE_OFF;
        const unsigned voff = lane * 16;
        unsigned slot_off = 0, pub = 0, s = 0;
        const unsigned a_full = lbase + L_FULL + lw * 4;
        const unsigned nstep = tn < 2 ? 2u * NSTEP : NSTEP;
        while (pub < nstep) {
            bool go = false;
            unsigned e = 0;
            if (s == (unsigned)NSTEP && pub == s) {
                while (rfl(lds_ld(lbase + L_MISC + 12)) < 8u || (g != SELFG && rfl(lds_ld(lbase + L_PCALL + 8)) == 0u)) __builtin_amdgcn_s_sleep(8);
                const unsigned long long t0 = rt();
                for (unsigned spins = 0;;) {
                    const unsigned f0 = __hip_atomic_load((gu32*)(ws + EXPF + rank * 512) + lane, RLX_SYS);
                    const unsigned f1 = __hip_atomic_load((gu32*)(ws + EXPF + rank * 512) + lane + 64, RLX_SYS);
                    const unsigned f2 = __hip_atomic_load((gu32*)(ws + XRDY) + (lane & 7), RLX_SYS);
                    if (rfl(__hip_atomic_load(tmo, RLX_AGENT)) == call) { lds_st(lbase + L_SYNC + 8, 1u); break; }
                    if (__all(f0 >= call && f1 >= call && (f2 >= call || (lane & 7) == rank))) break;
                    __builtin_amdgcn_s_sleep(32);
                    if ((++spins & 63u) == 0u) (void)spin_expired(tmo, call, t0);
                }
            }
            if (s < nstep && s - pub < 5u && (s != (unsigned)NSTEP || pub == s)) {
                const unsigned v = lds_ld(lane < 8 ? lbase + L_FREE + lane * 4 : lbase + L_SCHED + (s & 127u) * 4);
                e = (unsigned)__builtin_amdgcn_readlane((int)v, 8);
                if (s >= (unsigned)NSTEP) e = s & 127u;
                go = e != 0xffffffffu && __all(lane >= 8 || v + NSLOT > s);
            }
            if (go) {
                const char* blk = stage + (size_t)(e & 127u) * BLK_BYTES;
                const unsigned dst = lbase + slot_off;
                if (s >= (unsigned)NSTEP) {
                    const char* xb = stage + XSTG + ((e & 127u) * NR + tn * 4) * 4096;
                    const unsigned st = 4096u;
                    if (lw == 0) {
                        glds4(voff, blk + tm * 8192, dst);
                        glds4(voff, blk + tm * 8192 + 4096, dst + 4096);
                        glds4(voff, xb, dst + 8192);
                    } else {
                        glds4(voff, xb + st, dst + 12288);
                        glds4(voff, xb + 2 * st, dst + 16384);
                        glds4(voff, xb + 3 * st, dst + 20480);
                    }
                } else if (lw == 0) {
                    glds4(voff, blk + tm * 8192, dst);
                    glds4(voff, blk + tm * 8192 + 4096, dst + 4096);
                    glds4(voff, blk + A_BYTES + tn * 16384, dst + 8192);
                } else {
                    glds4(voff, blk + A_BYTES + tn * 16384 + 4096, dst + 12288);
                    glds4(voff, blk + A_BYTES + tn * 16384 + 8192, dst + 16384);
                    glds4(voff, blk + A_BYTES + tn * 16384 + 12288, dst + 20480);
                }
                ++s;
                slot_off += SLOT_BYTES;
                if (slot_off == NSLOT * SLOT_BYTES) slot_off = 0;
            } else if (pub < s) {
                const unsigned infl = s - pub;
                if (infl == 1) VMCNT(0); else if (infl == 2) VMCNT(12); else if (infl == 3) VMCNT(24);
                else if (infl == 4) VMCNT(36); else VMCNT(48);
                ++pub;
                lds_st(a_full, pub);
            } else {
                __builtin_amdgcn_s_sleep(1);
            }
        }
    } else if (wave == 10) {
        const unsigned xcc = xcc_id();
        int g = (int)(xcc & 7u);
        unsigned tk = 0;
        if (lane < 2) tk = __hip_atomic_fetch_add((gu32*)(ws + (lane == 0 ? PRIV + blockIdx.x * 64 : CLAIM + (gi * 8 + g) * 64)), 1u, RLX_AGENT);
        lds_st(lbase + L_SCHED + lane * 4, 0xffffffffu);
        lds_st(lbase + L_SCHED + 256 + lane * 4, 0xffffffffu);
        if (lane < 28) lds_st(lbase + L_FULL + lane * 4, 0u);
        const unsigned call = rfl(tk) + 1;
        unsigned got = (unsigned)__builtin_amdgcn_readlane((int)tk, 1) + 1;
        for (int tries = 0; got != call && tries < 7; ++tries) {
            unsigned o2 = 0;
            if (lane == 0) {
                __hip_atomic_fetch_add((gu32*)(ws + CLAIM + (gi * 8 + g) * 64), 0xffffffffu, RLX_AGENT);
                o2 = __hip_atomic_fetch_add((gu32*)(ws + CLAIM + (gi * 8 + ((g + 1) & 7)) * 64), 1u, RLX_AGENT);
            }
            g = (g + 1) & 7;
            got = rfl(o2) + 1;
        }
        if (lane < 8) lds_st(lbase + L_PCALL + lane * 4, lane == 0 ? call : lane == 1 ? (unsigned)g : 0u);
        BAR();
        bool bad = false;
        {
            const bool own = (lane & 7) == rank;
            gu32* fa = own ? (gu32*)(ws + EXPF + rank * 512) + ((lane & 56) + g) : (gu32*)(ws + AFLAG + g * 1024) + lane;
            gu32* fb = own ? (gu32*)(ws + EXPF + rank * 512) + ((lane & 56) + SELFG) : fa;
            gu32* fc = lane < 16 ? (gu32*)(ws + AFLAG + g * 1024) + (128 + (lane & 7)) : fb;
            unsigned long long done0 = 0, done1 = 0;
            unsigned cnt = 0, spins = 0;
            const unsigned ent = (lane & 7) * KBS + (lane >> 3);
            const unsigned long long t0 = rt();
            while ((done0 & done1) != ~0ull) {
                const bool h = done0 == ~0ull;
                const unsigned a0 = __hip_atomic_load(h ? fa + 64 : fa, RLX_SYS), a1 = __hip_atomic_load(fa + 64, RLX_SYS);
                const unsigned b0 = __hip_atomic_load(h ? fb + 64 : fc, RLX_SYS), b1 = __hip_atomic_load(fb + 64, RLX_SYS);
                const unsigned long long n0 = __ballot(bad || (a0 >= call && b0 >= call)) & ~done0;
                const unsigned long long n1 = __ballot(bad || (a1 >= call && b1 >= call)) & ~done1;
                if (n0 | n1) {
                    const unsigned long long lt = (1ull << lane) - 1;
                    const unsigned c0 = __popcll(n0), c1 = __popcll(n1);
                    if ((n0 >> lane) & 1) lds_st(lbase + L_SCHED + (cnt + __popcll(n0 & lt)) * 4, ent);
                    if ((n1 >> lane) & 1) lds_st(lbase + L_SCHED + (cnt + c0 + __popcll(n1 & lt)) * 4, ent + 8);
                    cnt += c0 + c1;
                    done0 |= n0; done1 |= n1;
                    if (cnt < SLOWCNT) for (int j = 0; j < POLL_REP; ++j) __builtin_amdgcn_s_sleep(127);
                    else if (cnt < FASTCNT) __builtin_amdgcn_s_sleep(POLL_SLOW);
                    else if (cnt < FASTCNT2) __builtin_amdgcn_s_sleep(POLL_MID);
                } else {
                    if (cnt < FASTCNT) __builtin_amdgcn_s_sleep(POLL_MISS);
                    else if (cnt < FASTCNT2) __builtin_amdgcn_s_sleep(POLL_MID);
                    else __builtin_amdgcn_s_sleep(1);
                    if ((++spins & 63u) == 0u && spin_expired(tmo, call, t0)) bad = true;
                }
            }
        }
        char* pw = (char*)d_ws_all[lane >> 3] + ws_off;
        asm volatile("" : "+v"(pw));
        while (rfl(lds_ld(lbase + L_SYNC)) < 8u) {}
        {
            unsigned m = wave_umax(lds_ld(lbase + L_WMAX + (lane & 7) * 4));
            if (lane == 0) __hip_atomic_store((gu64*)(ws + GMAX + g * 256) + gi, ((unsigned long long)call << 32) | m, RLX_AGENT);
            const unsigned long long t0 = rt();
            unsigned spins = 0;
            if (gi == 0 && g == SELFG) {
                const __amdgpu_buffer_rsrc_t rg = mk_rsrc(ws + GMAX, 2048);
                v4u a, b;
                for (;;) {
                    a = __builtin_amdgcn_raw_buffer_load_b128(rg, lane * 16, 0, AUX_DEV);
                    b = __builtin_amdgcn_raw_buffer_load_b128(rg, lane * 16 + 1024, 0, AUX_DEV);
                    if (__all(a[1] == call && a[3] == call && b[1] == call && b[3] == call)) break;
                    if ((++spins & 63u) == 0u && spin_expired(tmo, call, t0)) { bad = true; break; }
                    asm volatile("" ::: "memory");
                }
                m = wave_umax(max(max(a[0], a[2]), max(b[0], b[2])));
                __hip_atomic_store((gu64*)(pw + RMAX + (lane & 7) * 512) + rank, ((unsigned long long)call << 32) | m, RLX_SYS);
            }
            unsigned long long v;
            for (;;) {
                v = __hip_atomic_load((gu64*)(ws + RMAX + g * 512) + (lane & 7), RLX_SYS);
                if (rfl(__hip_atomic_load(tmo, RLX_AGENT)) == call) bad = true;
                if (bad || __all((unsigned)(v >> 32) == call)) break;
                if ((++spins & 63u) == 0u && spin_expired(tmo, call, t0)) { bad = true; break; }
            }
            m = wave_umax((unsigned)v);
            const float inv = 448.0f / __builtin_bit_cast(float, m);
            if (lane < 3) lds_st(lbase + L_MISC + lane * 4, lane == 0 ? __builtin_bit_cast(unsigned, inv) : lane == 1 ? 1u : (bad ? 1u : 0u));
        }
    } else if (wave == 11) {
        char* pwl = (char*)d_ws_all[lane & 7] + ws_off;
        char* pol = (char*)d_out_all[lane & 7];
        f32x4 buf[8];
        v4u o[4];
#define SLOT_TAG(T) do { lds_st(lbase + L_PTAG + sl * 4, ((n + 1) << 16) | (T)); ++n; sl = sl + 1 == NPS ? 0 : sl + 1; } while (0)
#define PUT_H(H) _Pragma("unroll") for (int u = 0; u < 2; ++u) \
            *(LAS v4u*)(smem + L_PSLOT + sl * 4096 + (2 * (H) + u) * 1024 + lane * 16) = o[2 * (H) + u]
        unsigned n = 0, sl = 0;
        bool bad = false;
        f32x4 buf2[8];
        const unsigned xcc = xcc_id();
        load_item(buf, x, w, 0u, 64u + dest_of(rank, xcc & 7u) * 32 + gi, lane);
        load_item(buf2, x, w, 1u, 64u + dest_of(rank, xcc & 7u) * 32 + gi, lane);
        BAR();
        GET_GROUP();
        const unsigned p = dest_of(rank, g);
        if ((unsigned)g != (xcc & 7u)) {
            load_item(buf, x, w, 0u, 64u + p * 32 + gi, lane);
            load_item(buf2, x, w, 1u, 64u + p * 32 + gi, lane);
        }
        cvt_item(o, buf, false); PUT_H(0); PUT_H(1);
        SLOT_TAG(64u + gi);
        cvt_item(o, buf2, false); PUT_H(0); PUT_H(1);
        SLOT_TAG(96u + 64u + gi);
        unsigned gp = 0;
        for (int j = 0; j < 8; ++j) if (dest_of((int)p, j) == (unsigned)rank) gp = j;
        unsigned long long rdy0 = 0, rdy1 = 0;
#define FLAG_SET(B) ((((B) < 64u ? rdy0 : rdy1) >> ((B) & 63u)) & 1u)
#define NEED_FLAG(FP, MASK, B) do { if (!FLAG_SET(B)) { \
                const unsigned long long t0 = rt(); unsigned spins = 0; \
                for (;;) { \
                    const unsigned f0 = __hip_atomic_load((gu32*)(FP) + (lane & (MASK)), RLX_SYS); \
                    const unsigned f1 = __hip_atomic_load((gu32*)(FP) + ((lane + 64) & (MASK)), RLX_SYS); \
                    rdy0 = __ballot(f0 >= call); rdy1 = __ballot(f1 >= call); \
                    if (FLAG_SET(B)) break; \
                    __builtin_amdgcn_s_sleep(2); \
                    if ((++spins & 63u) == 0u && spin_expired(tmo, call, t0)) { bad = true; rdy0 = rdy1 = ~0ull; break; } \
                } \
                asm volatile("" ::: "memory"); } } while (0)
#define PEND_TAG() do { if (pend != 0xffffffffu) { lds_st(lbase + L_PTAG + psl * 4, (n << 16) | pend | (bad ? 0x8000u : 0u)); pend = 0xffffffffu; } } while (0)
#define PULL_LOOP(DRAWP, BASE, NITEM, SRC, HSTEP, PREP, FP, FMASK, W, ON) do { \
            unsigned d = 0, so, fbit, pend = 0xffffffffu, psl = 0, dv; \
            if ((ON)) { dv = atomic_inc_async(DRAWP); ATOMIC_RESULT(d, dv, 0); } \
            while ((ON)) { \
                unsigned cur = d - (BASE); \
                if (cur >= (NITEM)) break; \
                PREP(cur); \
                if (!FLAG_SET(fbit) || rfl(lds_ld(lbase + L_PTAG + 12)) + NPS <= n) { \
                    VMCNT(0); PEND_TAG(); \
                    NEED_FLAG(FP, FMASK, fbit); \
                    while (rfl(lds_ld(lbase + L_PTAG + 12)) + NPS <= n) __builtin_amdgcn_s_sleep(1); \
                } \
                const bool rows = (HSTEP) != 2048 && cur < 2048u;         \
                const char* sb = (rows ? (SRC) : src1) + rfl(so); \
                const char* sb2 = sb + (rows ? (HSTEP) - 2048 : 0); \
                const unsigned hm = rows ? 0x00ffffffu : 0xffffffffu; \
                const unsigned dst = lbase + L_PSLOT + sl * 4096; \
                if (W == 2) VMCNT(1); else if (W == 3) VMCNT(2); else VMCNT(0); \
                if (W == 1) PEND_TAG(); \
                glds1<0>(lane * 16, sb, dst, 0xffffffffu); \
                dv = atomic_inc_async(DRAWP); \
                if (W == 2) { VMCNT(2); PEND_TAG(); } else if (W == 3) VMCNT(3); else VMCNT(1); \
                glds1<1024>(lane * 16, sb, dst, hm); \
                if (W == 2) VMCNT(2); else if (W == 3) { VMCNT(3); PEND_TAG(); } else VMCNT(0); \
                glds1<2048>(lane * 16, sb2, dst, 0xffffffffu); \
                if (W == 2) VMCNT(1); else if (W == 3) VMCNT(3); else VMCNT(0); \
                glds1<3072>(lane * 16, sb2, dst, hm); \
                ATOMIC_RESULT(d, dv, 3); \
                pend = cur; psl = sl; ++n; sl = sl + 1 == NPS ? 0 : sl + 1; \
            } \
            VMCNT(0); PEND_TAG(); \
            while (rfl(lds_ld(lbase + L_PTAG + 12)) + NPS <= n) __builtin_amdgcn_s_sleep(1); \
            SLOT_TAG(0xffffu); } while (0)
        if (p != (unsigned)rank) {
            const char* const src1 = lane_ptr(pwl, p) + STAGE_OFF;
#define PREP1(T) do { T = item_of(T < 64u ? T : T + 64u, ROT); const unsigned kb_ = T / 96u, z_ = T - kb_ * 96u; \
                fbit = kb_ * 8 + (z_ < 64u ? z_ >> 3 : gp); \
                so = z_ < 64u ? p * SRC_BYTES + T * 4096 : EXPW + ((rank * KBS + kb_) * 32 + z_ - 64u) * 4096; } while (0)
            PULL_LOOP(ws + DRAW1 + g * 128, (call - 1) * (NITEM1 - 64 + NPULL1), NITEM1 - 64, src1, 2048, PREP1,
                      ws + EXPF + p * 512, 127, PW1, ((gi * 5) & 31) < NPULL1);
#define PREP3(T) do { const unsigned k0_ = T / 7u, kb_ = (k0_ + 2 * rank) & 15u, d_ = (rank + 1 + T - k0_ * 7u) & 7u; unsigned gd_ = 0; \
                for (int j = 0; j < 8; ++j) if (dest_of((int)p, j) == d_) gd_ = j; \
                fbit = (kb_ < 2u ? 3u : kb_) * 8 + gd_; \
                T = kb_ * 8 + d_; so = XSTG + (p * KBS * NR + T) * 4096; } while (0)
            PULL_LOOP(ws + DRAW3 + g * 128, (call - 1) * (NITEM3 + NPULL3), NITEM3, src1, 2048, PREP3,
                      ws + EXPF + p * 512, 127, PW1, ((gi * 5) & 31) < NPULL3);
            const char* const src2 = lane_ptr(pol, p);
            rdy0 = rdy1 = bad ? ~0ull : 0ull;
            const unsigned skip = ((gp - SELFG) & 7u) * 256;
            while (rfl(lds_ld(lbase + L_PCALL + 8)) == 0u) __builtin_amdgcn_s_sleep(2);
#define PREP2(T) do { if (T < NROWP) { \
                    T = T < 1792u ? (SELFG * 256 + T + (T >= skip ? 256u : 0u)) & 2047u \
                                  : gp * 256 + ((T - 1792u) / BLK_ITEMS) * 64 + 8 * DPUSH + (T - 1792u) % BLK_ITEMS; \
                    fbit = T >> 8; so = p * NS + T * 2 * NN; \
                } else { T += 2048u - NROWP; fbit = (T - 2048u) >> 3; so = SIDE + (T - 2048u) * 4096; } } while (0)
            PULL_LOOP(ws + DRAW2 + g * 128, (call - 1) * (NITEM2 + NPULL2), NITEM2, src2, NN, PREP2, ws + RDY + p * 128, 7, PW2, ((gi * 5) & 31) < NPULL2);
        }
    } else {
        char* pwl = (char*)d_ws_all[lane & 7] + ws_off;
        char* pol = (char*)d_out_all[lane & 7];
        asm volatile("" : "+v"(pwl), "+v"(pol));
        BAR();
        GET_GROUP();
        const unsigned p = dest_of(rank, g);
        char* const pws = lane_ptr(pwl, p);
        v4u o[4];
        unsigned n = 0, sl = 0, t;
#define TAG_WAIT() do { unsigned tg = rfl(lds_ld(lbase + L_PTAG + sl * 4)); \
            while ((tg >> 16) != n + 1) { __builtin_amdgcn_s_sleep(1); tg = rfl(lds_ld(lbase + L_PTAG + sl * 4)); } \
            t = tg & 0xffffu; } while (0)
#define SLOT_NEXT() do { ++n; lds_st(lbase + L_PTAG + 12, n); sl = sl + 1 == NPS ? 0 : sl + 1; } while (0)
#define SLOT_GET() do { _Pragma("unroll") for (int f = 0; f < 4; ++f) o[f] = *(const LAS v4u*)(smem + L_PSLOT + sl * 4096 + f * 1024 + lane * 16); \
            LGKM0(); PIN_O(); SLOT_NEXT(); } while (0)
        {
            const __amdgpu_buffer_rsrc_t rs = mk_rsrc(pws + STAGE_OFF + (size_t)rank * SRC_BYTES, 2 * BLK_BYTES);
#pragma unroll
            for (int j = 0; j < 2; ++j) {
                TAG_WAIT(); SLOT_GET();
#pragma unroll
                for (int f = 0; f < 4; ++f) {
                    VMCNT(1);
                    __builtin_amdgcn_raw_buffer_store_b128(o[f], rs, t * 4096 + lane * 16 + f * 1024, 0, AUX_SYS);
                }
            }
            VMCNT(0);
            const unsigned cv = atomic_inc_async(ws + CNT1 + (g * 17 + 16) * 128);
            unsigned old;
            ATOMIC_RESULT(old, cv, 0);
            if (old + 1 == 32u * call && lane < 8) __hip_atomic_store((gu32*)(pws + AFLAG + lane * 1024) + (128 + rank), call, RLX_SYS);
        }
        if (p != (unsigned)rank) {
            const __amdgpu_buffer_rsrc_t rl = mk_rsrc(ws + STAGE_OFF + (size_t)p * SRC_BYTES, SRC_BYTES);
            for (;;) {
                TAG_WAIT();
                if (t == 0xffffu) { SLOT_NEXT(); break; }
                SLOT_GET();
                t &= 0x7fffu;
#pragma unroll
                for (int f = 0; f < 4; ++f) __builtin_amdgcn_raw_buffer_store_b128(o[f], rl, t * 4096 + lane * 16 + f * 1024, 0, AUX_DEV);
                const unsigned kb = t / 96u, z = t - kb * 96u;
                VMCNT(0);
                unsigned cv = atomic_inc_masked_async(ws + CNT1 + (g * 17 + kb) * 128, z < 64u ? 1u << (z >> 3) : 0xffu, lane * 4);
                ATOMIC_WAIT(cv, 0);
                if (cv + 1 == (kb < 2u ? 8u : 40u) * call)
                    __hip_atomic_store((gu32*)(ws + AFLAG + lane * 1024) + (kb * 8 + p), call, RLX_AGENT);
            }
        }
        if (p != (unsigned)rank) {
            const __amdgpu_buffer_rsrc_t rx = mk_rsrc(ws + STAGE_OFF + XSTG + (size_t)p * KBS * NR * 4096, KBS * NR * 4096);
            for (;;) {
                TAG_WAIT();
                if (t == 0xffffu) { SLOT_NEXT(); break; }
                SLOT_GET();
#pragma unroll
                for (int f = 0; f < 4; ++f) __builtin_amdgcn_raw_buffer_store_b128(o[f], rx, (t & 0x7fffu) * 4096 + lane * 16 + f * 1024, 0, AUX_DEV);
                VMCNT(0);
                const unsigned cv = atomic_inc_async(ws + XCNT + g * 128);
                unsigned old;
                ATOMIC_RESULT(old, cv, 0);
                if (old + 1 == NITEM3 * call && lane == 0) __hip_atomic_store((gu32*)(ws + XRDY) + p, call, RLX_AGENT);
            }
            const int r16 = lane & 15, q = lane >> 4;
            while (rfl(lds_ld(lbase + L_SYNC + 4)) < 8u) __builtin_amdgcn_s_sleep(1);
            const __amdgpu_buffer_rsrc_t rd = mk_rsrc(lane_ptr(pol, p), OUT_BYTES);
            for (int it = 0; it < DPUSH; ++it) {
#pragma unroll
                for (int u = 0; u < 4; ++u) {
                    const int row = it * 16 + u * 4 + q;
                    o[u] = *(const LAS v4u*)(smem + row * 256 + ((r16 ^ (row & 15)) * 16));
                }
                PIN_O();
#pragma unroll
                for (int u = 0; u < 4; ++u) {
                    VMCNT(PUSHW);
                    __builtin_amdgcn_raw_buffer_store_b128(o[u], rd, (u * 4 + q) * NN + r16 * 16 + (tm * 128 + it * 16) * NN + rank * NS + tn * 256, 0, AUX_SYS);
                }
            }
            lds_st(lbase + L_PCALL + 8, 1u);
            VMCNT(0);
            if (lane == 0) __hip_atomic_store((gu32*)(pws + FIN + rank * 128) + gi, call, RLX_SYS);
            const __amdgpu_buffer_rsrc_t rl = mk_rsrc(lane_ptr(pol, rank), OUT_BYTES);
            const unsigned vo = lane * 16 + p * NS;
            for (;;) {
                TAG_WAIT();
                if (t == 0xffffu) break;
                SLOT_GET();
                if (t & 0x8000u) { t &= 0x7fffu; o[0] = o[1] = o[2] = o[3] = (v4u){0x7f7f7f7fu, 0x7f7f7f7fu, 0x7f7f7f7fu, 0x7f7f7f7fu}; }
#pragma unroll
                for (int u = 0; u < 4; ++u) {
                    const unsigned oa = ((u & 1) && lane >= 56 ? 0x80000000u : vo) + (u >> 1) * NN + (u & 1) * 1024 + t * 2 * NN;
                    const unsigned ob = ((t - 2048u) * 64 + u * 16 + (lane >> 2)) * NN + p * NS + (NS - 128) + (lane & 3) * 16;
                    __builtin_amdgcn_raw_buffer_store_b128(o[u], rl, t < 2048u ? oa : ob, 0, AUX_DEV);
                }
            }
        } else if (gi == 0) {
            const unsigned long long t0 = rt();
            unsigned spins = 0;
            const __amdgpu_buffer_rsrc_t rw = mk_rsrc(ws + FIN, 1024);
            for (;;) {
                const v4u f = __builtin_amdgcn_raw_buffer_load_b128(rw, lane * 16, 0, AUX_SYS);
                if (__all((lane >> 3) == rank || (f[0] >= call && f[1] >= call && f[2] >= call && f[3] >= call))) break;
                __builtin_amdgcn_s_sleep(8);
                if ((++spins & 63u) == 0u && spin_expired(tmo, call, t0)) {
                    if (lane == 0) ((unsigned char*)d_out_all[rank])[(size_t)tm * 128 * NN + rank * NS + tn * 256] = 0x7f;
                    break;
                }
                asm volatile("" ::: "memory");
            }
            __builtin_amdgcn_fence(__ATOMIC_ACQUIRE, "");
        }
    }
}

extern "C" void kernel_launch(int rank, int nranks, ncclComm_t comm, hipStream_t stream,
                              void* const* d_in, const int* in_sizes, int n_in,
                              void* d_out, int out_size,
                              const int* in_dtypes, int out_dtype,
                              void* const* d_out_all,
                              void* const* d_ws_all, size_t ws_bytes) {
    (void)comm; (void)d_out; (void)in_dtypes; (void)out_dtype;
    if (nranks != NR || (unsigned)rank >= (unsigned)NR || n_in < 2 || ws_bytes < WS_NEED ||
        in_sizes[0] != MM * KS || in_sizes[1] != KS * NN || out_size != MM * NN) {
        fprintf(stderr, "kernel_launch: unexpected problem shape\n");
        return;
    }
    const size_t ws_off = (ws_bytes - WS_NEED) & ~(size_t)((2u << 20) - 1);
    hipLaunchKernelGGL(gemm_ar_q8, dim3(NWG), dim3(NTHREADS), 0, stream,
                       (const float*)d_in[0], (const float*)d_in[1], d_out_all, d_ws_all, ws_off, rank);
    HIP_CHECK(hipGetLastError());
}
```

```cpp
#include <hip/hip_runtime.h>
#include <rccl/rccl.h>
#include <cstdio>
#include <cstdint>

#define HIP_CHECK(expr) do { hipError_t _e = (expr); if (_e != hipSuccess) { \
    fprintf(stderr, "HIP error %s at %s:%d\n", hipGetErrorString(_e), __FILE__, __LINE__); \
    return; } } while (0)

#define GAS __attribute__((address_space(1)))
#define LAS __attribute__((address_space(3)))
typedef unsigned v4u __attribute__((ext_vector_type(4)));
typedef unsigned v2u __attribute__((ext_vector_type(2)));
typedef float f32x4 __attribute__((ext_vector_type(4)));
typedef short bf16x8 __attribute__((ext_vector_type(8)));
typedef __bf16 bf2 __attribute__((ext_vector_type(2)));
typedef GAS unsigned gu32;
typedef GAS unsigned long long gu64;

namespace {

constexpr int NR = 8;
constexpr int MM = 4096, NN = 16384;
constexpr int KS = 512;
constexpr int NS = 2048;
constexpr int KBS = 16;
constexpr unsigned A_BYTES = 256u * 1024u;
constexpr unsigned B_BYTES = 128u * 1024u;
constexpr unsigned BLK_BYTES = A_BYTES + B_BYTES;
constexpr unsigned SRC_BYTES = KBS * BLK_BYTES;
constexpr size_t STAGE_OFF = 1u << 20;
constexpr unsigned EXPW = NR * SRC_BYTES;
constexpr unsigned XSTG = EXPW + NR * KBS * B_BYTES;
constexpr unsigned SIDE = XSTG + NR * KBS * NR * 4096;
constexpr unsigned DATA_BYTES = SIDE + MM * 64;
constexpr size_t WS_NEED = STAGE_OFF + DATA_BYTES;
constexpr unsigned OUT_BYTES = (unsigned)MM * NN;

constexpr unsigned DRAW1 = 0x0000;
constexpr unsigned DRAW2 = 0x0400;
constexpr unsigned TMO   = 0x0800;
constexpr unsigned XRDY  = 0x0900;
constexpr unsigned DRAW3 = 0x0C00;
constexpr unsigned XCNT  = 0x4000;
constexpr unsigned EXPF  = 0x1000;
constexpr unsigned AFLAG = 0x2000;
constexpr unsigned GMAX  = 0x5000;
constexpr unsigned RMAX  = 0x6000;
constexpr unsigned RCNT  = 0x8000;
constexpr unsigned RDY   = 0x8400;
constexpr unsigned FIN   = 0x9000;
constexpr unsigned CNT1  = 0x28000;
constexpr unsigned ECNT  = 0x30000;
constexpr unsigned PRIV  = 0x20000;
constexpr unsigned CLAIM = 0x24000;

constexpr int NWG = 256, NTHREADS = 896;
constexpr int NSLOT = 6;
constexpr unsigned SLOT_BYTES = 24576;
constexpr unsigned L_FULL = NSLOT * SLOT_BYTES;
constexpr unsigned L_FREE = L_FULL + 16;
constexpr unsigned L_WMAX = L_FREE + 32;
constexpr unsigned L_MISC = L_WMAX + 32;
constexpr unsigned L_PTAG = L_MISC + 16;
constexpr unsigned L_PCALL = L_PTAG + 16;
constexpr unsigned L_SYNC = L_PCALL + 16;
constexpr unsigned L_SCHED = L_SYNC + 16;
constexpr unsigned L_PSLOT = L_SCHED + 512;
constexpr unsigned NPS = 3;
constexpr unsigned LDS_BYTES = L_PSLOT + NPS * 4096;
static_assert(LDS_BYTES <= 163840 && L_PSLOT % 16 == 0 && L_MISC % 16 == 0, "LDS layout");
constexpr int NSTEP = NR * KBS;
constexpr unsigned long long DEADLINE = 100000000ull;

#define DPUSH 1
#define NPULL1 22
#define NPULL2 28
#define ROT (p + rank)
#define SLOWCNT 64u
#define POLL_REP 3
#define PUSHW 1
#define NPULL3 16
#define CSLEEP 2
#define PW1 2
#define PW2 2
#define CONV_T0 400u
#define CONV_TAU 384u
#define POLL_SLOW 127
#define POLL_MISS 16
#define POLL_MID 48
#define FASTCNT 96u
#define FASTCNT2 116u
constexpr int SELFG = 3;
constexpr unsigned NITEM1 = KBS * 96;
constexpr unsigned NCONV = 14 * 40;
constexpr unsigned BLK_ITEMS = 64 - 8 * DPUSH;
constexpr unsigned NROWP = 7 * 256 + 4 * BLK_ITEMS;
constexpr unsigned NITEM2 = NROWP + 64;
constexpr unsigned NITEM3 = 7 * KBS;

__device__ __forceinline__ unsigned dest_of(int rank, int g) {
    const unsigned set = (0xB2u >> rank) & 1u;
    const unsigned pos = (0x33212100u >> (4 * rank)) & 3u;
    const unsigned members = ((g < 4) == (set == 0)) ? 0x6320u : 0x7541u;
    return (members >> (4 * ((pos + 1 + g) & 3))) & 7u;
}

#define VMCNT_(n) asm volatile("s_waitcnt vmcnt(" #n ")" ::: "memory")
#define VMCNT(n) VMCNT_(n)
#define LGKM0() asm volatile("s_waitcnt lgkmcnt(0)" ::: "memory")
#define BAR() do { asm volatile("s_waitcnt lgkmcnt(0)" ::: "memory"); __builtin_amdgcn_s_barrier(); asm volatile("" ::: "memory"); } while (0)
#define RLX_AGENT __ATOMIC_RELAXED, __HIP_MEMORY_SCOPE_AGENT
#define RLX_SYS __ATOMIC_RELAXED, __HIP_MEMORY_SCOPE_SYSTEM

__device__ __forceinline__ unsigned long long rt() { return __builtin_amdgcn_s_memrealtime(); }
__device__ __forceinline__ unsigned rfl(unsigned v) { return (unsigned)__builtin_amdgcn_readfirstlane((int)v); }
__device__ __forceinline__ unsigned xcc_id() {
    unsigned v;
    asm volatile("s_getreg_b32 %0, hwreg(HW_REG_XCC_ID)" : "=s"(v));
    return v & 7u;
}
__device__ __forceinline__ unsigned pk_bf16(float lo, float hi) {
    bf2 v; v.x = (__bf16)lo; v.y = (__bf16)hi; return __builtin_bit_cast(unsigned, v);
}
__device__ __forceinline__ unsigned wave_umax(unsigned v) {
    v = max(v, (unsigned)__builtin_amdgcn_update_dpp(0, (int)v, 0x128, 0xf, 0xf, false));
    v = max(v, (unsigned)__builtin_amdgcn_update_dpp(0, (int)v, 0x124, 0xf, 0xf, false));
    v = max(v, (unsigned)__builtin_amdgcn_update_dpp(0, (int)v, 0x122, 0xf, 0xf, false));
    v = max(v, (unsigned)__builtin_amdgcn_update_dpp(0, (int)v, 0x121, 0xf, 0xf, false));
    const unsigned a = (unsigned)__builtin_amdgcn_readlane((int)v, 0), b = (unsigned)__builtin_amdgcn_readlane((int)v, 16);
    const unsigned c = (unsigned)__builtin_amdgcn_readlane((int)v, 32), d = (unsigned)__builtin_amdgcn_readlane((int)v, 48);
    return max(max(a, b), max(c, d));
}
__device__ __forceinline__ char* lane_ptr(const void* v, unsigned l) {
    const unsigned long long u = (unsigned long long)v;
    return (char*)(((unsigned long long)(unsigned)__builtin_amdgcn_readlane((int)(u >> 32), (int)l) << 32) |
                   (unsigned)__builtin_amdgcn_readlane((int)u, (int)l));
}
__device__ __forceinline__ void lds_st(unsigned addr, unsigned v) { asm volatile("ds_write_b32 %0, %1" :: "v"(addr), "v"(v) : "memory"); }
__device__ __forceinline__ unsigned lds_ld(unsigned addr) {
    unsigned v; asm volatile("ds_read_b32 %0, %1\n\ts_waitcnt lgkmcnt(0)" : "=&v"(v) : "v"(addr) : "memory"); return v;
}
__device__ __forceinline__ void glds4(unsigned voff, const char* sbase, unsigned m0v) {
    unsigned keep;
    asm volatile("s_mov_b32 %0, m0\n\ts_mov_b32 m0, %3\n\ts_nop 4\n\t"
                 "global_load_lds_dwordx4 %1, %2 sc0 sc1\n\t"
                 "global_load_lds_dwordx4 %1, %2 offset:1024 sc0 sc1\n\t"
                 "global_load_lds_dwordx4 %1, %2 offset:2048 sc0 sc1\n\t"
                 "global_load_lds_dwordx4 %1, %2 offset:3072 sc0 sc1\n\t"
                 "s_mov_b32 m0, %0"
                 : "=&s"(keep) : "v"(voff), "s"(sbase), "s"(m0v) : "memory");
}
template <int IMM> __device__ __forceinline__ void glds1(unsigned voff, const char* sbase, unsigned m0v, unsigned exec_hi) {
    unsigned keep;
    asm volatile("s_mov_b32 %0, m0\n\ts_mov_b32 m0, %3\n\ts_mov_b32 exec_hi, %5\n\ts_nop 3\n\t"
                 "global_load_lds_dwordx4 %1, %2 offset:%4 sc0 sc1\n\t"
                 "s_mov_b32 exec_hi, -1\n\ts_mov_b32 m0, %0"
                 : "=&s"(keep) : "v"(voff), "s"(sbase), "s"(m0v), "i"(IMM), "s"(rfl(exec_hi)) : "memory");
}
__device__ __forceinline__ __amdgpu_buffer_rsrc_t mk_rsrc(void* base, unsigned bytes) {
    return __builtin_amdgcn_make_buffer_rsrc(base, 0, (int)bytes, 0x00020000);
}
__device__ __forceinline__ unsigned atomic_inc_async(const void* p) {
    unsigned old; unsigned long long sv;
    asm volatile("s_mov_b64 %1, exec\n\ts_mov_b64 exec, 1\n\ts_nop 3\n\t"
                 "global_atomic_add %0, %2, %3, %4 sc0\n\t"
                 "s_mov_b64 exec, %1"
                 : "=&v"(old), "=&s"(sv) : "v"(0u), "v"(1u), "s"(p) : "memory");
    return old;
}
#define ATOMIC_RESULT(dst, old, n) ATOMIC_RESULT_(dst, old, n)
#define ATOMIC_RESULT_(dst, old, n) asm volatile("s_waitcnt vmcnt(" #n ")\n\tv_readfirstlane_b32 %0, %1\n\ts_nop 4" : "=s"(dst) : "v"(old) : "memory")
__device__ __forceinline__ unsigned atomic_inc_masked_async(const void* p, unsigned mask, unsigned lane4) {
    unsigned old = 0; unsigned long long sv;
    asm volatile("s_mov_b64 %1, exec\n\ts_mov_b32 exec_lo, %5\n\ts_mov_b32 exec_hi, 0\n\ts_nop 2\n\t"
                 "global_atomic_add %0, %2, %3, %4 sc0\n\t"
                 "s_mov_b64 exec, %1"
                 : "+v"(old), "=&s"(sv) : "v"(lane4), "v"(1u), "s"(p), "s"(rfl(mask)) : "memory");
    return old;
}
#define ATOMIC_WAIT(old, n) asm volatile("s_waitcnt vmcnt(" #n ")" : "+v"(old) :: "memory")
constexpr int AUX_SYS = 17;
constexpr int AUX_DEV = 16;

__device__ __forceinline__ bool spin_expired(gu32* tmo, unsigned call, unsigned long long t0) {
    if (rfl(__hip_atomic_load(tmo, RLX_AGENT)) == call || rt() - t0 > DEADLINE) {
        __hip_atomic_store(tmo, call, RLX_AGENT);
        return true;
    }
    return false;
}

__device__ __forceinline__ void load_item(f32x4 (&buf)[8], const float* x, const float* w, unsigned kb, unsigned z, int lane) {
    const int r16 = lane & 15, q = lane >> 4;
    if (z < 64u) {
        const float* a = x + (size_t)(z * 64 + r16) * KS + kb * 32 + 4 * q;
#pragma unroll
        for (int e = 0; e < 8; ++e) buf[e] = *(const f32x4*)(a + (e >> 1) * 16 * KS + (e & 1) * 16);
    } else {
        const float* b = w + (size_t)(kb * 32 + 4 * q) * NN + (z - 64u) * 64 + 4 * r16;
#pragma unroll
        for (int e = 0; e < 8; ++e) buf[e] = *(const f32x4*)(b + (size_t)((e >> 2) * 16 + (e & 3)) * NN);
    }
}
__device__ __forceinline__ void cvt_item(v4u (&o)[4], const f32x4 (&buf)[8], bool is_x) {
    if (is_x) {
#pragma unroll
        for (int f = 0; f < 4; ++f) {
            o[f][0] = pk_bf16(buf[2*f][0], buf[2*f][1]); o[f][1] = pk_bf16(buf[2*f][2], buf[2*f][3]);
            o[f][2] = pk_bf16(buf[2*f+1][0], buf[2*f+1][1]); o[f][3] = pk_bf16(buf[2*f+1][2], buf[2*f+1][3]);
        }
    } else {
#pragma unroll
        for (int j = 0; j < 4; ++j) {
            o[j][0] = pk_bf16(buf[0][j], buf[1][j]); o[j][1] = pk_bf16(buf[2][j], buf[3][j]);
            o[j][2] = pk_bf16(buf[4][j], buf[5][j]); o[j][3] = pk_bf16(buf[6][j], buf[7][j]);
        }
    }
}
__device__ __forceinline__ unsigned item_of(unsigned t, unsigned src) {
    if (t < 128u) return t < 64u ? (t + src * 8) & 63u : (t >= 96u ? 96u : 0u) + 64u + (t & 31u);
    const unsigned k = (t - 128u) / 96u, u = t - 128u - k * 96u, j = u / 12u, v = u - j * 12u;
    if (k == 14u) return 15u * 96u + ((u + src * 8) & 63u);
    if (v < 8u) return (k + 1u) * 96u + ((j + src) & 7u) * 8u + v;
    return (k + 2u) * 96u + 64u + j * 4u + v - 8u;
}


}

__global__ void __launch_bounds__(NTHREADS, 4)
gemm_ar_q8(const float* __restrict__ x, const float* __restrict__ w, void* const* __restrict__ d_out_all,
           void* const* __restrict__ d_ws_all, size_t ws_off, int rank) {
    __shared__ __attribute__((aligned(1024))) unsigned char smem[LDS_BYTES];
    const unsigned lbase = (unsigned)(size_t)(LAS unsigned char*)smem;
    const int lane = threadIdx.x & 63;
    const int wave = (int)rfl(threadIdx.x >> 6);
    const int gi = blockIdx.x >> 3;
    char* const ws = (char*)d_ws_all[rank] + ws_off;
    gu32* const tmo = (gu32*)(ws + TMO);
    const int tn = gi & 7;
#define GET_GROUP() const v2u gc_ = *(volatile LAS v2u*)(smem + L_PCALL); \
    const unsigned call = rfl(gc_[0]); const int g = (int)(rfl(gc_[1]) & 7u); const int tm = g * 4 + (gi >> 3); (void)call; (void)tm
#define PIN_O() asm volatile("" : "+v"(o[0]), "+v"(o[1]), "+v"(o[2]), "+v"(o[3]) :: "memory")

#define CONVERT(C) CONVERT_(2u + (C) / 40u, (C) % 40u, 2u + (C) / 40u)
#define CONVERT_(KB, U, CK) do { \
        f32x4 buf[8]; v4u o[4]; \
        const unsigned d = dest_of(rank, g), kb = (KB), u = (U), ck = (CK); \
        load_item(buf, x, w, kb, u < 32u ? 64u + d * 32 + u : g * 8 + u - 32u, lane); \
        cvt_item(o, buf, u >= 32u); \
        const unsigned off = u >= 32u ? (rank * KBS + kb) * BLK_BYTES + (g * 8 + u - 32u) * 4096 \
                           : d == (unsigned)rank ? (rank * KBS + kb) * BLK_BYTES + A_BYTES + u * 4096 \
                           : EXPW + ((d * KBS + kb) * 32 + u) * 4096; \
        const __amdgpu_buffer_rsrc_t rs = mk_rsrc(ws + STAGE_OFF, DATA_BYTES); \
        if (kb >= 2u) { _Pragma("unroll") for (int f = 0; f < 4; ++f) __builtin_amdgcn_raw_buffer_store_b128(o[f], rs, off + lane * 16 + f * 1024, 0, AUX_SYS); } \
        if (u == 31u) { _Pragma("unroll") for (int f = 0; f < 4; ++f) \
            __builtin_amdgcn_raw_buffer_store_b128(o[f], rs, XSTG + ((rank * KBS + kb) * NR + d) * 4096 + lane * 16 + f * 1024, 0, AUX_SYS); } \
        VMCNT(0); \
        const unsigned cv = atomic_inc_async(ws + ECNT + (g * 16 + ck) * 128); \
        unsigned old; \
        ATOMIC_RESULT(old, cv, 0); \
        if (old + 1 == (ck == 3u ? 42u : 40u) * call && lane < 8) __hip_atomic_store((gu32*)(pw + EXPF + rank * 512) + (ck * 8 + g), call, RLX_SYS); \
    } while (0)

    if (wave == 13) {
        char* pw = (char*)d_ws_all[lane & 7] + ws_off;
        BAR();
        GET_GROUP();
        const unsigned long long t0 = rt();
        for (unsigned j = 0; j * 32 + gi < NCONV; j += j == 0 ? 2 : 1) {
            while (rt() - t0 < (j == 0 ? 0u : CONV_T0 + (j - 2) * CONV_TAU + gi * (CONV_TAU / 32))) __builtin_amdgcn_s_sleep(4);
            CONVERT(j * 32 + gi);
        }
    } else if (wave < 8) {
        char* pw = (char*)d_ws_all[lane & 7] + ws_off;
        {
            f32x4 buf[8];
            v4u o[4];
            const unsigned xcc = xcc_id();
            const unsigned kb = (wave >> 1) & 1, z = (wave & 1) * 32 + gi;
            const bool ahead = wave < 4 && (xcc & 7u) == (unsigned)SELFG;
            if (ahead) load_item(buf, x, w, kb, z, lane);
            BAR();
            const v2u gc = *(volatile LAS v2u*)(smem + L_PCALL);
            if (wave < 4 && (rfl(gc[1]) & 7u) == (unsigned)SELFG) {
                const unsigned call = rfl(gc[0]);
                if (!ahead) load_item(buf, x, w, kb, z, lane);
                cvt_item(o, buf, true);
                const __amdgpu_buffer_rsrc_t rs = mk_rsrc(ws + STAGE_OFF, DATA_BYTES);
#pragma unroll
                for (int f = 0; f < 4; ++f)
                    __builtin_amdgcn_raw_buffer_store_b128(o[f], rs, (rank * KBS + kb) * BLK_BYTES + z * 4096 + lane * 16 + f * 1024, 0, AUX_SYS);
                VMCNT(0);
                const unsigned cv = atomic_inc_async(ws + ECNT + (SELFG * 16 + kb) * 128);
                unsigned old;
                ATOMIC_RESULT(old, cv, 0);
                if (old + 1 == 64u * call) __hip_atomic_store((gu32*)(pw + EXPF + rank * 512) + (kb * 8 + (lane >> 3)), call, RLX_SYS);
            }
        }
        GET_GROUP();
        if (wave == 4) CONVERT(32u + gi);
        if (wave == 5 && gi < 2) CONVERT_((unsigned)gi, 31u, 3u);
        const int wm = wave >> 2, wn = wave & 3;
        const LAS unsigned char* abase = (const LAS unsigned char*)smem + wm * 4096 + lane * 16;
        const LAS unsigned char* bbase = (const LAS unsigned char*)smem + 8192 + wn * 4096 + lane * 16;
        volatile LAS v2u* full = (volatile LAS v2u*)(smem + L_FULL);
        unsigned slot_off = 0, s = 0, bad = 0;
        float inv = 0.f;
        const __amdgpu_buffer_rsrc_t ro = mk_rsrc(d_out_all[rank], OUT_BYTES);
#define QUANT(O, I) _Pragma("unroll") for (int r = 0; r < 4; ++r) { \
            int d_ = __builtin_amdgcn_cvt_pk_fp8_f32(acc[I][0][r] * inv, acc[I][1][r] * inv, 0, false); \
            d_ = __builtin_amdgcn_cvt_pk_fp8_f32(acc[I][2][r] * inv, acc[I][3][r] * inv, d_, true); \
            O[r] = bad ? 0x7f7f7f7fu : (unsigned)d_; }
#define LDS_ARRIVE(OFF) do { LGKM0(); if (lane == 0) __hip_atomic_fetch_add((LAS unsigned*)(smem + (OFF)), 1u, __ATOMIC_RELAXED, __HIP_MEMORY_SCOPE_WORKGROUP); } while (0)
        for (unsigned pass = 0; pass < (tn < 2 ? 2u : 1u); ++pass) {
        f32x4 acc[4][4];
#pragma unroll
        for (int i = 0; i < 4; ++i)
#pragma unroll
            for (int j = 0; j < 4; ++j) acc[i][j] = (f32x4){0.f, 0.f, 0.f, 0.f};
        for (const unsigned send = s + NSTEP; s < send; ++s) {
            for (;;) {
                const v2u f = *full;
                if (rfl(f.x) > s && rfl(f.y) > s) break;
                __builtin_amdgcn_s_sleep(1);
            }
            asm volatile("" ::: "memory");
            v4u a[4], b[4];
#pragma unroll
            for (int i = 0; i < 4; ++i) b[i] = *(const LAS v4u*)(bbase + slot_off + i * 1024);
#pragma unroll
            for (int i = 0; i < 4; ++i) a[i] = *(const LAS v4u*)(abase + slot_off + i * 1024);
            LGKM0();
            lds_st(lbase + L_FREE + wave * 4, s + 1);
            asm volatile("" : "+v"(a[0]), "+v"(a[1]), "+v"(a[2]), "+v"(a[3]), "+v"(b[0]), "+v"(b[1]), "+v"(b[2]), "+v"(b[3]) :: "memory");
#pragma unroll
            for (int i = 0; i < 4; ++i)
#pragma unroll
                for (int j = 0; j < 4; ++j)
                    acc[i][j] = __builtin_amdgcn_mfma_f32_16x16x32_bf16(__builtin_bit_cast(bf16x8, b[j]), __builtin_bit_cast(bf16x8, a[i]), acc[i][j], 0, 0, 0);
            slot_off += SLOT_BYTES;
            if (slot_off == NSLOT * SLOT_BYTES) slot_off = 0;
        }
        int ln = lane;
        asm volatile("" : "+v"(ln));
        const int r16 = ln & 15, q = ln >> 4;
        if (pass > 0) {
            const unsigned d = tn * 4 + wn;
            bad |= rfl(lds_ld(lbase + L_SYNC + 8));
#pragma unroll
            for (int i = 0; i < 4; ++i) {
                v4u o;
                QUANT(o, i);
                if (d != (unsigned)rank)
                    __builtin_amdgcn_raw_buffer_store_b128(o, ro, (tm * 128 + wm * 64 + i * 16 + r16) * NN + d * NS + (NS - 64) + q * 16, 0, AUX_DEV);
            }
            continue;
        }
        float mx = 0.f;
#pragma unroll
        for (int i = 0; i < 4; ++i)
#pragma unroll
            for (int j = 0; j < 4; ++j)
#pragma unroll
                for (int r = 0; r < 4; ++r) mx = fmaxf(mx, fabsf(acc[i][j][r]));
        lds_st(lbase + L_WMAX + wave * 4, wave_umax(__builtin_bit_cast(unsigned, mx)));
        LDS_ARRIVE(L_SYNC);
        char* pw2 = (char*)d_ws_all[ln & 7] + ws_off;
        v4u misc;
        for (;;) { misc = *(volatile LAS v4u*)(smem + L_MISC); if (rfl(misc[1]) != 0u) break; __builtin_amdgcn_s_sleep(CSLEEP); }
        inv = __builtin_bit_cast(float, misc[0]);
        bad = misc[2];
#pragma unroll
        for (int i = 0; i < 4; ++i) {
            v4u o;
            QUANT(o, i);
            *(LAS v4u*)(smem + (wm * 64 + i * 16 + r16) * 256 + (((wn * 4 + q) ^ r16) * 16)) = o;
        }
        LDS_ARRIVE(L_SYNC + 4);
        while (rfl(lds_ld(lbase + L_SYNC + 4)) < 8u) {}
        if (g != SELFG) while (rfl(lds_ld(lbase + L_PCALL + 8)) == 0u) __builtin_amdgcn_s_sleep(4);
#pragma unroll
        for (int u = 0; u < 4; ++u) {
            const int row = wave * 16 + u * 4 + q;
            const v4u v = *(const LAS v4u*)(smem + row * 256 + ((r16 ^ (row & 15)) * 16));
            __builtin_amdgcn_raw_buffer_store_b128(v, ro, (tm * 128 + row) * NN + rank * NS + tn * 256 + r16 * 16, 0, AUX_SYS);
        }
        if (tn == 7) {
            const v4u v = *(const LAS v4u*)(smem + (wave * 16 + (ln >> 2)) * 256 + (((8 + (ln & 3)) ^ (ln >> 2)) * 16));
            __builtin_amdgcn_raw_buffer_store_b128(v, mk_rsrc(ws + STAGE_OFF + SIDE, MM * 64), (tm * 128 + wave * 16) * 64 + ln * 16, 0, AUX_SYS);
        }
        VMCNT(0);
        unsigned old = 0;
        if (lane == 0) old = __hip_atomic_fetch_add((LAS unsigned*)(smem + L_MISC + 12), 1u, __ATOMIC_RELAXED, __HIP_MEMORY_SCOPE_WORKGROUP);
        if (rfl(old) == 7) {
            const unsigned cv = atomic_inc_async(ws + RCNT + g * 128);
            unsigned oc;
            ATOMIC_RESULT(oc, cv, 0);
            if (oc + 1 == 32u * call && lane < 8) __hip_atomic_store((gu32*)(pw2 + RDY + rank * 128) + g, call, RLX_SYS);
        }
        }
    } else if (wave < 10) {
        const int lw = wave - 8;
        BAR();
        GET_GROUP();
        const char* stage = ws + STAGE_OFF;
        const unsigned voff = lane * 16;
        unsigned slot_off = 0, pub = 0, s = 0;
        const unsigned a_full = lbase + L_FULL + lw * 4;
        const unsigned nstep = tn < 2 ? 2u * NSTEP : NSTEP;
        while (pub < nstep) {
            bool go = false;
            unsigned e = 0;
            if (s == (unsigned)NSTEP && pub == s) {
                while (rfl(lds_ld(lbase + L_MISC + 12)) < 8u || (g != SELFG && rfl(lds_ld(lbase + L_PCALL + 8)) == 0u)) __builtin_amdgcn_s_sleep(8);
                const unsigned long long t0 = rt();
                for (unsigned spins = 0;;) {
                    const unsigned f0 = __hip_atomic_load((gu32*)(ws + EXPF + rank * 512) + lane, RLX_SYS);
                    const unsigned f1 = __hip_atomic_load((gu32*)(ws + EXPF + rank * 512) + lane + 64, RLX_SYS);
                    const unsigned f2 = __hip_atomic_load((gu32*)(ws + XRDY) + (lane & 7), RLX_SYS);
                    if (rfl(__hip_atomic_load(tmo, RLX_AGENT)) == call) { lds_st(lbase + L_SYNC + 8, 1u); break; }
                    if (__all(f0 >= call && f1 >= call && (f2 >= call || (lane & 7) == rank))) break;
                    __builtin_amdgcn_s_sleep(32);
                    if ((++spins & 63u) == 0u) (void)spin_expired(tmo, call, t0);
                }
            }
            if (s < nstep && s - pub < 5u && (s != (unsigned)NSTEP || pub == s)) {
                const unsigned v = lds_ld(lane < 8 ? lbase + L_FREE + lane * 4 : lbase + L_SCHED + (s & 127u) * 4);
                e = (unsigned)__builtin_amdgcn_readlane((int)v, 8);
                if (s >= (unsigned)NSTEP) e = s & 127u;
                go = e != 0xffffffffu && __all(lane >= 8 || v + NSLOT > s);
            }
            if (go) {
                const char* blk = stage + (size_t)(e & 127u) * BLK_BYTES;
                const unsigned dst = lbase + slot_off;
                if (s >= (unsigned)NSTEP) {
                    const char* xb = stage + XSTG + ((e & 127u) * NR + tn * 4) * 4096;
                    const unsigned st = 4096u;
                    if (lw == 0) {
                        glds4(voff, blk + tm * 8192, dst);
                        glds4(voff, blk + tm * 8192 + 4096, dst + 4096);
                        glds4(voff, xb, dst + 8192);
                    } else {
                        glds4(voff, xb + st, dst + 12288);
                        glds4(voff, xb + 2 * st, dst + 16384);
                        glds4(voff, xb + 3 * st, dst + 20480);
                    }
                } else if (lw == 0) {
                    glds4(voff, blk + tm * 8192, dst);
                    glds4(voff, blk + tm * 8192 + 4096, dst + 4096);
                    glds4(voff, blk + A_BYTES + tn * 16384, dst + 8192);
                } else {
                    glds4(voff, blk + A_BYTES + tn * 16384 + 4096, dst + 12288);
                    glds4(voff, blk + A_BYTES + tn * 16384 + 8192, dst + 16384);
                    glds4(voff, blk + A_BYTES + tn * 16384 + 12288, dst + 20480);
                }
                ++s;
                slot_off += SLOT_BYTES;
                if (slot_off == NSLOT * SLOT_BYTES) slot_off = 0;
            } else if (pub < s) {
                const unsigned infl = s - pub;
                if (infl == 1) VMCNT(0); else if (infl == 2) VMCNT(12); else if (infl == 3) VMCNT(24);
                else if (infl == 4) VMCNT(36); else VMCNT(48);
                ++pub;
                lds_st(a_full, pub);
            } else {
                __builtin_amdgcn_s_sleep(1);
            }
        }
    } else if (wave == 10) {
        const unsigned xcc = xcc_id();
        int g = (int)(xcc & 7u);
        unsigned tk = 0;
        if (lane < 2) tk = __hip_atomic_fetch_add((gu32*)(ws + (lane == 0 ? PRIV + blockIdx.x * 64 : CLAIM + (gi * 8 + g) * 64)), 1u, RLX_AGENT);
        lds_st(lbase + L_SCHED + lane * 4, 0xffffffffu);
        lds_st(lbase + L_SCHED + 256 + lane * 4, 0xffffffffu);
        if (lane < 28) lds_st(lbase + L_FULL + lane * 4, 0u);
        const unsigned call = rfl(tk) + 1;
        unsigned got = (unsigned)__builtin_amdgcn_readlane((int)tk, 1) + 1;
        for (int tries = 0; got != call && tries < 7; ++tries) {
            unsigned o2 = 0;
            if (lane == 0) {
                __hip_atomic_fetch_add((gu32*)(ws + CLAIM + (gi * 8 + g) * 64), 0xffffffffu, RLX_AGENT);
                o2 = __hip_atomic_fetch_add((gu32*)(ws + CLAIM + (gi * 8 + ((g + 1) & 7)) * 64), 1u, RLX_AGENT);
            }
            g = (g + 1) & 7;
            got = rfl(o2) + 1;
        }
        if (lane < 8) lds_st(lbase + L_PCALL + lane * 4, lane == 0 ? call : lane == 1 ? (unsigned)g : 0u);
        BAR();
        bool bad = false;
        {
            const bool own = (lane & 7) == rank;
            gu32* fa = own ? (gu32*)(ws + EXPF + rank * 512) + ((lane & 56) + g) : (gu32*)(ws + AFLAG + g * 1024) + lane;
            gu32* fb = own ? (gu32*)(ws + EXPF + rank * 512) + ((lane & 56) + SELFG) : fa;
            gu32* fc = lane < 16 ? (gu32*)(ws + AFLAG + g * 1024) + (128 + (lane & 7)) : fb;
            unsigned long long done0 = 0, done1 = 0;
            unsigned cnt = 0, spins = 0;
            const unsigned ent = (lane & 7) * KBS + (lane >> 3);
            const unsigned long long t0 = rt();
            while ((done0 & done1) != ~0ull) {
                const bool h = done0 == ~0ull;
                const unsigned a0 = __hip_atomic_load(h ? fa + 64 : fa, RLX_SYS), a1 = __hip_atomic_load(fa + 64, RLX_SYS);
                const unsigned b0 = __hip_atomic_load(h ? fb + 64 : fc, RLX_SYS), b1 = __hip_atomic_load(fb + 64, RLX_SYS);
                const unsigned long long n0 = __ballot(bad || (a0 >= call && b0 >= call)) & ~done0;
                const unsigned long long n1 = __ballot(bad || (a1 >= call && b1 >= call)) & ~done1;
                if (n0 | n1) {
                    const unsigned long long lt = (1ull << lane) - 1;
                    const unsigned c0 = __popcll(n0), c1 = __popcll(n1);
                    if ((n0 >> lane) & 1) lds_st(lbase + L_SCHED + (cnt + __popcll(n0 & lt)) * 4, ent);
                    if ((n1 >> lane) & 1) lds_st(lbase + L_SCHED + (cnt + c0 + __popcll(n1 & lt)) * 4, ent + 8);
                    cnt += c0 + c1;
                    done0 |= n0; done1 |= n1;
                    if (cnt < SLOWCNT) for (int j = 0; j < POLL_REP; ++j) __builtin_amdgcn_s_sleep(127);
                    else if (cnt < FASTCNT) __builtin_amdgcn_s_sleep(POLL_SLOW);
                    else if (cnt < FASTCNT2) __builtin_amdgcn_s_sleep(POLL_MID);
                } else {
                    if (cnt < FASTCNT) __builtin_amdgcn_s_sleep(POLL_MISS);
                    else if (cnt < FASTCNT2) __builtin_amdgcn_s_sleep(POLL_MID);
                    else __builtin_amdgcn_s_sleep(1);
                    if ((++spins & 63u) == 0u && spin_expired(tmo, call, t0)) bad = true;
                }
            }
        }
        char* pw = (char*)d_ws_all[lane >> 3] + ws_off;
        asm volatile("" : "+v"(pw));
        while (rfl(lds_ld(lbase + L_SYNC)) < 8u) {}
        {
            unsigned m = wave_umax(lds_ld(lbase + L_WMAX + (lane & 7) * 4));
            if (lane == 0) __hip_atomic_store((gu64*)(ws + GMAX + g * 256) + gi, ((unsigned long long)call << 32) | m, RLX_AGENT);
            const unsigned long long t0 = rt();
            unsigned spins = 0;
            if (gi == 0 && g == SELFG) {
                const __amdgpu_buffer_rsrc_t rg = mk_rsrc(ws + GMAX, 2048);
                v4u a, b;
                for (;;) {
                    a = __builtin_amdgcn_raw_buffer_load_b128(rg, lane * 16, 0, AUX_DEV);
                    b = __builtin_amdgcn_raw_buffer_load_b128(rg, lane * 16 + 1024, 0, AUX_DEV);
                    if (__all(a[1] == call && a[3] == call && b[1] == call && b[3] == call)) break;
                    if ((++spins & 63u) == 0u && spin_expired(tmo, call, t0)) { bad = true; break; }
                    asm volatile("" ::: "memory");
                }
                m = wave_umax(max(max(a[0], a[2]), max(b[0], b[2])));
                __hip_atomic_store((gu64*)(pw + RMAX + (lane & 7) * 512) + rank, ((unsigned long long)call << 32) | m, RLX_SYS);
            }
            unsigned long long v;
            for (;;) {
                v = __hip_atomic_load((gu64*)(ws + RMAX + g * 512) + (lane & 7), RLX_SYS);
                if (rfl(__hip_atomic_load(tmo, RLX_AGENT)) == call) bad = true;
                if (bad || __all((unsigned)(v >> 32) == call)) break;
                if ((++spins & 63u) == 0u && spin_expired(tmo, call, t0)) { bad = true; break; }
            }
            m = wave_umax((unsigned)v);
            const float inv = 448.0f / __builtin_bit_cast(float, m);
            if (lane < 3) lds_st(lbase + L_MISC + lane * 4, lane == 0 ? __builtin_bit_cast(unsigned, inv) : lane == 1 ? 1u : (bad ? 1u : 0u));
        }
    } else if (wave == 11) {
        char* pwl = (char*)d_ws_all[lane & 7] + ws_off;
        char* pol = (char*)d_out_all[lane & 7];
        f32x4 buf[8];
        v4u o[4];
#define SLOT_TAG(T) do { lds_st(lbase + L_PTAG + sl * 4, ((n + 1) << 16) | (T)); ++n; sl = sl + 1 == NPS ? 0 : sl + 1; } while (0)
#define PUT_H(H) _Pragma("unroll") for (int u = 0; u < 2; ++u) \
            *(LAS v4u*)(smem + L_PSLOT + sl * 4096 + (2 * (H) + u) * 1024 + lane * 16) = o[2 * (H) + u]
        unsigned n = 0, sl = 0;
        bool bad = false;
        f32x4 buf2[8];
        const unsigned xcc = xcc_id();
        load_item(buf, x, w, 0u, 64u + dest_of(rank, xcc & 7u) * 32 + gi, lane);
        load_item(buf2, x, w, 1u, 64u + dest_of(rank, xcc & 7u) * 32 + gi, lane);
        BAR();
        GET_GROUP();
        const unsigned p = dest_of(rank, g);
        if ((unsigned)g != (xcc & 7u)) {
            load_item(buf, x, w, 0u, 64u + p * 32 + gi, lane);
            load_item(buf2, x, w, 1u, 64u + p * 32 + gi, lane);
        }
        cvt_item(o, buf, false); PUT_H(0); PUT_H(1);
        SLOT_TAG(64u + gi);
        cvt_item(o, buf2, false); PUT_H(0); PUT_H(1);
        SLOT_TAG(96u + 64u + gi);
        unsigned gp = 0;
        for (int j = 0; j < 8; ++j) if (dest_of((int)p, j) == (unsigned)rank) gp = j;
        unsigned long long rdy0 = 0, rdy1 = 0;
#define FLAG_SET(B) ((((B) < 64u ? rdy0 : rdy1) >> ((B) & 63u)) & 1u)
#define NEED_FLAG(FP, MASK, B) do { if (!FLAG_SET(B)) { \
                const unsigned long long t0 = rt(); unsigned spins = 0; \
                for (;;) { \
                    const unsigned f0 = __hip_atomic_load((gu32*)(FP) + (lane & (MASK)), RLX_SYS); \
                    const unsigned f1 = __hip_atomic_load((gu32*)(FP) + ((lane + 64) & (MASK)), RLX_SYS); \
                    rdy0 = __ballot(f0 >= call); rdy1 = __ballot(f1 >= call); \
                    if (FLAG_SET(B)) break; \
                    __builtin_amdgcn_s_sleep(2); \
                    if ((++spins & 63u) == 0u && spin_expired(tmo, call, t0)) { bad = true; rdy0 = rdy1 = ~0ull; break; } \
                } \
                asm volatile("" ::: "memory"); } } while (0)
#define PEND_TAG() do { if (pend != 0xffffffffu) { lds_st(lbase + L_PTAG + psl * 4, (n << 16) | pend | (bad ? 0x8000u : 0u)); pend = 0xffffffffu; } } while (0)
#define PULL_LOOP(DRAWP, BASE, NITEM, SRC, HSTEP, PREP, FP, FMASK, W, ON) do { \
            unsigned d = 0, so, fbit, pend = 0xffffffffu, psl = 0, dv; \
            if ((ON)) { dv = atomic_inc_async(DRAWP); ATOMIC_RESULT(d, dv, 0); } \
            while ((ON)) { \
                unsigned cur = d - (BASE); \
                if (cur >= (NITEM)) break; \
                PREP(cur); \
                if (!FLAG_SET(fbit) || rfl(lds_ld(lbase + L_PTAG + 12)) + NPS <= n) { \
                    VMCNT(0); PEND_TAG(); \
                    NEED_FLAG(FP, FMASK, fbit); \
                    while (rfl(lds_ld(lbase + L_PTAG + 12)) + NPS <= n) __builtin_amdgcn_s_sleep(1); \
                } \
                const bool rows = (HSTEP) != 2048 && cur < 2048u;         \
                const char* sb = (rows ? (SRC) : src1) + rfl(so); \
                const char* sb2 = sb + (rows ? (HSTEP) - 2048 : 0); \
                const unsigned hm = rows ? 0x00ffffffu : 0xffffffffu; \
                const unsigned dst = lbase + L_PSLOT + sl * 4096; \
                if (W == 2) VMCNT(1); else if (W == 3) VMCNT(2); else VMCNT(0); \
                if (W == 1) PEND_TAG(); \
                glds1<0>(lane * 16, sb, dst, 0xffffffffu); \
                dv = atomic_inc_async(DRAWP); \
                if (W == 2) { VMCNT(2); PEND_TAG(); } else if (W == 3) VMCNT(3); else VMCNT(1); \
                glds1<1024>(lane * 16, sb, dst, hm); \
                if (W == 2) VMCNT(2); else if (W == 3) { VMCNT(3); PEND_TAG(); } else VMCNT(0); \
                glds1<2048>(lane * 16, sb2, dst, 0xffffffffu); \
                if (W == 2) VMCNT(1); else if (W == 3) VMCNT(3); else VMCNT(0); \
                glds1<3072>(lane * 16, sb2, dst, hm); \
                ATOMIC_RESULT(d, dv, 3); \
                pend = cur; psl = sl; ++n; sl = sl + 1 == NPS ? 0 : sl + 1; \
            } \
            VMCNT(0); PEND_TAG(); \
            while (rfl(lds_ld(lbase + L_PTAG + 12)) + NPS <= n) __builtin_amdgcn_s_sleep(1); \
            SLOT_TAG(0xffffu); } while (0)
        if (p != (unsigned)rank) {
            const char* const src1 = lane_ptr(pwl, p) + STAGE_OFF;
#define PREP1(T) do { T = item_of(T < 64u ? T : T + 64u, ROT); const unsigned kb_ = T / 96u, z_ = T - kb_ * 96u; \
                fbit = kb_ * 8 + (z_ < 64u ? z_ >> 3 : gp); \
                so = z_ < 64u ? p * SRC_BYTES + T * 4096 : EXPW + ((rank * KBS + kb_) * 32 + z_ - 64u) * 4096; } while (0)
            PULL_LOOP(ws + DRAW1 + g * 128, (call - 1) * (NITEM1 - 64 + NPULL1), NITEM1 - 64, src1, 2048, PREP1,
                      ws + EXPF + p * 512, 127, PW1, ((gi * 5) & 31) < NPULL1);
#define PREP3(T) do { const unsigned k0_ = T / 7u, kb_ = (k0_ + 2 * rank) & 15u, d_ = (rank + 1 + T - k0_ * 7u) & 7u; unsigned gd_ = 0; \
                for (int j = 0; j < 8; ++j) if (dest_of((int)p, j) == d_) gd_ = j; \
                fbit = (kb_ < 2u ? 3u : kb_) * 8 + gd_; \
                T = kb_ * 8 + d_; so = XSTG + (p * KBS * NR + T) * 4096; } while (0)
            PULL_LOOP(ws + DRAW3 + g * 128, (call - 1) * (NITEM3 + NPULL3), NITEM3, src1, 2048, PREP3,
                      ws + EXPF + p * 512, 127, PW1, ((gi * 5) & 31) < NPULL3);
            const char* const src2 = lane_ptr(pol, p);
            rdy0 = rdy1 = bad ? ~0ull : 0ull;
            const unsigned skip = ((gp - SELFG) & 7u) * 256;
            while (rfl(lds_ld(lbase + L_PCALL + 8)) == 0u) __builtin_amdgcn_s_sleep(2);
#define PREP2(T) do { if (T < NROWP) { \
                    T = T < 1792u ? (SELFG * 256 + T + (T >= skip ? 256u : 0u)) & 2047u \
                                  : gp * 256 + ((T - 1792u) / BLK_ITEMS) * 64 + 8 * DPUSH + (T - 1792u) % BLK_ITEMS; \
                    fbit = T >> 8; so = p * NS + T * 2 * NN; \
                } else { T += 2048u - NROWP; fbit = (T - 2048u) >> 3; so = SIDE + (T - 2048u) * 4096; } } while (0)
            PULL_LOOP(ws + DRAW2 + g * 128, (call - 1) * (NITEM2 + NPULL2), NITEM2, src2, NN, PREP2, ws + RDY + p * 128, 7, PW2, ((gi * 5) & 31) < NPULL2);
        }
    } else {
        char* pwl = (char*)d_ws_all[lane & 7] + ws_off;
        char* pol = (char*)d_out_all[lane & 7];
        asm volatile("" : "+v"(pwl), "+v"(pol));
        BAR();
        GET_GROUP();
        const unsigned p = dest_of(rank, g);
        char* const pws = lane_ptr(pwl, p);
        v4u o[4];
        unsigned n = 0, sl = 0, t;
#define TAG_WAIT() do { unsigned tg = rfl(lds_ld(lbase + L_PTAG + sl * 4)); \
            while ((tg >> 16) != n + 1) { __builtin_amdgcn_s_sleep(1); tg = rfl(lds_ld(lbase + L_PTAG + sl * 4)); } \
            t = tg & 0xffffu; } while (0)
#define SLOT_NEXT() do { ++n; lds_st(lbase + L_PTAG + 12, n); sl = sl + 1 == NPS ? 0 : sl + 1; } while (0)
#define SLOT_GET() do { _Pragma("unroll") for (int f = 0; f < 4; ++f) o[f] = *(const LAS v4u*)(smem + L_PSLOT + sl * 4096 + f * 1024 + lane * 16); \
            LGKM0(); PIN_O(); SLOT_NEXT(); } while (0)
        {
            const __amdgpu_buffer_rsrc_t rs = mk_rsrc(pws + STAGE_OFF + (size_t)rank * SRC_BYTES, 2 * BLK_BYTES);
#pragma unroll
            for (int j = 0; j < 2; ++j) {
                TAG_WAIT(); SLOT_GET();
#pragma unroll
                for (int f = 0; f < 4; ++f) {
                    VMCNT(1);
                    __builtin_amdgcn_raw_buffer_store_b128(o[f], rs, t * 4096 + lane * 16 + f * 1024, 0, AUX_SYS);
                }
            }
            VMCNT(0);
            const unsigned cv = atomic_inc_async(ws + CNT1 + (g * 17 + 16) * 128);
            unsigned old;
            ATOMIC_RESULT(old, cv, 0);
            if (old + 1 == 32u * call && lane < 8) __hip_atomic_store((gu32*)(pws + AFLAG + lane * 1024) + (128 + rank), call, RLX_SYS);
        }
        if (p != (unsigned)rank) {
            const __amdgpu_buffer_rsrc_t rl = mk_rsrc(ws + STAGE_OFF + (size_t)p * SRC_BYTES, SRC_BYTES);
            for (;;) {
                TAG_WAIT();
                if (t == 0xffffu) { SLOT_NEXT(); break; }
                SLOT_GET();
                t &= 0x7fffu;
#pragma unroll
                for (int f = 0; f < 4; ++f) __builtin_amdgcn_raw_buffer_store_b128(o[f], rl, t * 4096 + lane * 16 + f * 1024, 0, AUX_DEV);
                const unsigned kb = t / 96u, z = t - kb * 96u;
                VMCNT(0);
                unsigned cv = atomic_inc_masked_async(ws + CNT1 + (g * 17 + kb) * 128, z < 64u ? 1u << (z >> 3) : 0xffu, lane * 4);
                ATOMIC_WAIT(cv, 0);
                if (cv + 1 == (kb < 2u ? 8u : 40u) * call)
                    __hip_atomic_store((gu32*)(ws + AFLAG + lane * 1024) + (kb * 8 + p), call, RLX_AGENT);
            }
        }
        if (p != (unsigned)rank) {
            const __amdgpu_buffer_rsrc_t rx = mk_rsrc(ws + STAGE_OFF + XSTG + (size_t)p * KBS * NR * 4096, KBS * NR * 4096);
            for (;;) {
                TAG_WAIT();
                if (t == 0xffffu) { SLOT_NEXT(); break; }
                SLOT_GET();
#pragma unroll
                for (int f = 0; f < 4; ++f) __builtin_amdgcn_raw_buffer_store_b128(o[f], rx, (t & 0x7fffu) * 4096 + lane * 16 + f * 1024, 0, AUX_DEV);
                VMCNT(0);
                const unsigned cv = atomic_inc_async(ws + XCNT + g * 128);
                unsigned old;
                ATOMIC_RESULT(old, cv, 0);
                if (old + 1 == NITEM3 * call && lane == 0) __hip_atomic_store((gu32*)(ws + XRDY) + p, call, RLX_AGENT);
            }
            const int r16 = lane & 15, q = lane >> 4;
            while (rfl(lds_ld(lbase + L_SYNC + 4)) < 8u) __builtin_amdgcn_s_sleep(1);
            const __amdgpu_buffer_rsrc_t rd = mk_rsrc(lane_ptr(pol, p), OUT_BYTES);
            for (int it = 0; it < DPUSH; ++it) {
#pragma unroll
                for (int u = 0; u < 4; ++u) {
                    const int row = it * 16 + u * 4 + q;
                    o[u] = *(const LAS v4u*)(smem + row * 256 + ((r16 ^ (row & 15)) * 16));
                }
                PIN_O();
#pragma unroll
                for (int u = 0; u < 4; ++u) {
                    VMCNT(PUSHW);
                    __builtin_amdgcn_raw_buffer_store_b128(o[u], rd, (u * 4 + q) * NN + r16 * 16 + (tm * 128 + it * 16) * NN + rank * NS + tn * 256, 0, AUX_SYS);
                }
            }
            lds_st(lbase + L_PCALL + 8, 1u);
            VMCNT(0);
            if (lane == 0) __hip_atomic_store((gu32*)(pws + FIN + rank * 128) + gi, call, RLX_SYS);
            const __amdgpu_buffer_rsrc_t rl = mk_rsrc(lane_ptr(pol, rank), OUT_BYTES);
            const unsigned vo = lane * 16 + p * NS;
            for (;;) {
                TAG_WAIT();
                if (t == 0xffffu) break;
                SLOT_GET();
                if (t & 0x8000u) { t &= 0x7fffu; o[0] = o[1] = o[2] = o[3] = (v4u){0x7f7f7f7fu, 0x7f7f7f7fu, 0x7f7f7f7fu, 0x7f7f7f7fu}; }
#pragma unroll
                for (int u = 0; u < 4; ++u) {
                    const unsigned oa = ((u & 1) && lane >= 56 ? 0x80000000u : vo) + (u >> 1) * NN + (u & 1) * 1024 + t * 2 * NN;
                    const unsigned ob = ((t - 2048u) * 64 + u * 16 + (lane >> 2)) * NN + p * NS + (NS - 128) + (lane & 3) * 16;
                    __builtin_amdgcn_raw_buffer_store_b128(o[u], rl, t < 2048u ? oa : ob, 0, AUX_DEV);
                }
            }
        } else if (gi == 0) {
            const unsigned long long t0 = rt();
            unsigned spins = 0;
            const __amdgpu_buffer_rsrc_t rw = mk_rsrc(ws + FIN, 1024);
            for (;;) {
                const v4u f = __builtin_amdgcn_raw_buffer_load_b128(rw, lane * 16, 0, AUX_SYS);
                if (__all((lane >> 3) == rank || (f[0] >= call && f[1] >= call && f[2] >= call && f[3] >= call))) break;
                __builtin_amdgcn_s_sleep(8);
                if ((++spins & 63u) == 0u && spin_expired(tmo, call, t0)) {
                    if (lane == 0) ((unsigned char*)d_out_all[rank])[(size_t)tm * 128 * NN + rank * NS + tn * 256] = 0x7f;
                    break;
                }
                asm volatile("" ::: "memory");
            }
            __builtin_amdgcn_fence(__ATOMIC_ACQUIRE, "");
        }
    }
}

extern "C" void kernel_launch(int rank, int nranks, ncclComm_t comm, hipStream_t stream,
                              void* const* d_in, const int* in_sizes, int n_in,
                              void* d_out, int out_size,
                              const int* in_dtypes, int out_dtype,
                              void* const* d_out_all,
                              void* const* d_ws_all, size_t ws_bytes) {
    (void)comm; (void)d_out; (void)in_dtypes; (void)out_dtype;
    if (nranks != NR || (unsigned)rank >= (unsigned)NR || n_in < 2 || ws_bytes < WS_NEED ||
        in_sizes[0] != MM * KS || in_sizes[1] != KS * NN || out_size != MM * NN) {
        fprintf(stderr, "kernel_launch: unexpected problem shape\n");
        return;
    }
    const size_t ws_off = (ws_bytes - WS_NEED) & ~(size_t)((2u << 20) - 1);
    hipLaunchKernelGGL(gemm_ar_q8, dim3(NWG), dim3(NTHREADS), 0, stream,
                       (const float*)d_in[0], (const float*)d_in[1], d_out_all, d_ws_all, ws_off, rank);
    HIP_CHECK(hipGetLastError());
}
```
